```python
import numpy as np

kernel_name = "gpu_gemm_ar_m4096_k4096_n4096_e4m3e4m3_dequant"


ABSMAX_RTOL = 0.03

COMPUTE_DTYPE = "float32"
EPILOGUE = "dequant"
IN_QUANT = None
OUT_QUANT = None
IN_FP8 = "e4m3e4m3"
IN_DTYPES = {"x": "e4m3", "w_mat": "e4m3",
             "scale_x": "float32", "scale_w": "float32"}
WORLD_SIZE = 8

def _snap_fp8(x, mant_bits, max_abs, min_normal):
    a = np.abs(x.astype(np.float64))
    _m, _e = np.frexp(a)
    q = np.where(a >= min_normal, np.ldexp(1.0, _e - 1 - mant_bits),
                 min_normal * 2.0 ** -mant_bits)
    snapped = np.minimum(np.round(a / q) * q, max_abs)
    return (np.sign(x) * snapped).astype(np.float32)

def setup_inputs():

    x = _snap_fp8(np.random.randn(4096, 4096).astype(np.float32), 3, 448.0, 2.0 ** -6)
    w_mat = _snap_fp8(np.random.randn(4096, 4096).astype(np.float32), 3, 448.0, 2.0 ** -6)
    scale_x = (0.02 * (1.0 + np.random.rand(1))).astype(np.float32)
    scale_w = (0.02 * (1.0 + np.random.rand(1))).astype(np.float32)
    return {"x": x, "w_mat": w_mat, "scale_x": scale_x, "scale_w": scale_w}

def reference(x, w_mat, scale_x, scale_w):
    acc = (x @ w_mat).astype(np.float32)
    y = (acc * (scale_x * scale_w)).astype(np.float32)
    return y
```

```cpp
#include <hip/hip_runtime.h>
#include <rccl/rccl.h>
#include <cstdio>
#include <cstdint>

#define HIP_CHECK(expr) do { hipError_t _e = (expr); if (_e != hipSuccess) { \
    fprintf(stderr, "HIP error %s at %s:%d\n", hipGetErrorString(_e), __FILE__, __LINE__); \
    return; } } while (0)

namespace {

#define LAS __attribute__((address_space(3)))
#define GAS __attribute__((address_space(1)))
typedef int v4i __attribute__((ext_vector_type(4)));
typedef int v8i __attribute__((ext_vector_type(8)));
typedef unsigned v4u __attribute__((ext_vector_type(4)));
typedef float v4f __attribute__((ext_vector_type(4)));
typedef float v16f __attribute__((ext_vector_type(16)));

constexpr int NR = 8;
constexpr int MM = 4096, NN = 4096, KS = 512;
constexpr int ROUNDS = 16;
constexpr int NPANEL = 32;
constexpr int BLK = 8192;
constexpr int NWG = 256, NTHREADS = 768;
constexpr int NSTEP = 64;
constexpr size_t WS_CTR = 0;
constexpr size_t WS_FLAG = 16384;
constexpr size_t WS_TMO = 16384 + 4096;
constexpr size_t WS_DBG = 32768;
constexpr size_t WS_DATA = 1u << 20;
constexpr size_t SRC_BYTES = (size_t)ROUNDS * NPANEL * BLK;
constexpr size_t PAR_BYTES = (size_t)NR * SRC_BYTES;
constexpr size_t WS_NEED = WS_DATA + 2 * PAR_BYTES;
constexpr unsigned long long DEADLINE = 40000000ull;
constexpr int NSLOT = 4, SLOT = 32768;
constexpr int L_CTL = NSLOT * SLOT;
constexpr int L_FULL = L_CTL;
constexpr int L_PROG = L_CTL + 64;
constexpr int L_READY = L_CTL + 128;
constexpr int L_BAD = L_CTL + 192;
constexpr int LDS_BYTES = L_CTL + 256;

struct Params {
    const unsigned char* x; const unsigned char* w; const float* sx; const float* sw;
    float* out; unsigned char* ws[NR]; int rank;
};

__device__ __forceinline__ unsigned long long rt() { return __builtin_amdgcn_s_memrealtime(); }
__device__ __forceinline__ unsigned rfl(unsigned v) { return __builtin_amdgcn_readfirstlane(v); }
__device__ __forceinline__ unsigned lds_ld(LAS unsigned char* L, int off) { return *(volatile LAS unsigned*)(L + off); }
__device__ __forceinline__ void lds_st(LAS unsigned char* L, int off, unsigned v) { *(volatile LAS unsigned*)(L + off) = v; }
#define FENCE() asm volatile("" ::: "memory")
#define WAIT_V(n) asm volatile("s_waitcnt vmcnt(" #n ")" ::: "memory")

#define STAMP(i) do {} while (0)

__device__ __forceinline__ void unit_of(int u, int& d, int& r) { d = u & 7; r = u >> 3; }

__device__ __forceinline__ void dma8k(const unsigned char* g, unsigned ldsaddr, unsigned voff) {
    asm volatile(
        "s_mov_b32 m0, %2\n\t"
        "s_nop 0\n\t"
        "global_load_lds_dwordx4 %0, %1 offset:-4096\n\t"
        "global_load_lds_dwordx4 %0, %1 offset:-3072\n\t"
        "global_load_lds_dwordx4 %0, %1 offset:-2048\n\t"
        "global_load_lds_dwordx4 %0, %1 offset:-1024\n\t"
        "global_load_lds_dwordx4 %0, %1 offset:0\n\t"
        "global_load_lds_dwordx4 %0, %1 offset:1024\n\t"
        "global_load_lds_dwordx4 %0, %1 offset:2048\n\t"
        "global_load_lds_dwordx4 %0, %1 offset:3072\n\t"
        :: "v"(voff), "s"(g + 4096), "s"(ldsaddr + 4096) : "memory");
}
__device__ __forceinline__ void st16_sys(unsigned char* sbase, unsigned voff, v4u v) {
    asm volatile("global_store_dwordx4 %0, %1, %2 sc0 sc1\n\ts_nop 1" :: "v"(voff), "v"(v), "s"(sbase) : "memory");
}

__global__ void __launch_bounds__(NTHREADS) fused_gemm_ag(Params p) {
    __shared__ __attribute__((aligned(1024))) unsigned char lds_raw[LDS_BYTES];
    LAS unsigned char* L = (LAS unsigned char*)lds_raw;
    const int tid = threadIdx.x, lane = tid & 63;
    const int wave = rfl(tid >> 6);
    const int b = blockIdx.x, xcd = b & 7, idx = b >> 3;
    const int ti = (xcd >> 1) * 4 + (idx >> 3), tj = (xcd & 1) * 8 + (idx & 7);
    unsigned char* myws = p.ws[p.rank & 7];

    if (wave < 8) {
        if (wave == 0) lds_st(L, L_CTL + lane * 4, 0u);
        asm volatile("s_waitcnt lgkmcnt(0)" ::: "memory");
        __builtin_amdgcn_s_barrier();
        FENCE();
        const int wr = wave >> 1, wc = wave & 1, i = lane & 31, h = lane >> 5;
        const int abase = h * 8192 + (64 * wr + i) * 16;
        const int bbase = 16384 + h * 8192 + (32 * wc + i) * 16;
        v16f acc[2][4];
#pragma unroll
        for (int m = 0; m < 2; ++m)
#pragma unroll
            for (int c = 0; c < 4; ++c)
#pragma unroll
                for (int e = 0; e < 16; ++e) acc[m][c][e] = 0.f;
        if (wave == 0) STAMP(0);
        for (int s = 0; s < NSTEP; ++s) {
            for (unsigned spin = 0;; ++spin) {
                typedef unsigned v2u __attribute__((ext_vector_type(2)));
                v2u f = *(volatile LAS v2u*)(L + L_FULL);
                unsigned mn = rfl(f.x < f.y ? f.x : f.y);
                if (mn > (unsigned)s || spin > (1u << 26)) break;
                __builtin_amdgcn_s_sleep(1);
            }
            FENCE();
            LAS unsigned char* sl = L + (s & (NSLOT - 1)) * SLOT;
            v8i a[2], bf[4];
#pragma unroll
            for (int m = 0; m < 2; ++m) {
                v4i lo = *(LAS v4i*)(sl + abase + m * 512), hi = *(LAS v4i*)(sl + abase + m * 512 + 4096);
                a[m] = __builtin_shufflevector(lo, hi, 0, 1, 2, 3, 4, 5, 6, 7);
            }
#pragma unroll
            for (int c = 0; c < 4; ++c) {
                v4i lo = *(LAS v4i*)(sl + bbase + c * 1024), hi = *(LAS v4i*)(sl + bbase + c * 1024 + 4096);
                bf[c] = __builtin_shufflevector(lo, hi, 0, 1, 2, 3, 4, 5, 6, 7);
                if (c == 3) {
                    asm volatile("s_waitcnt lgkmcnt(0)" : "+v"(bf[3]) :: "memory");
                    lds_st(L, L_PROG + wave * 4, (unsigned)(s + 1));
                    FENCE();
                }
#pragma unroll
                for (int m = 0; m < 2; ++m)
                    acc[m][c] = __builtin_amdgcn_mfma_scale_f32_32x32x64_f8f6f4(a[m], bf[c], acc[m][c], 0, 0, 0, 0, 0, 0);
            }
        }
        if (wave == 0) STAMP(1);
        const float scale = p.sx[0] * p.sw[0];
        const bool bad = lds_ld(L, L_BAD) != 0u;
        const float mul = bad ? __builtin_nanf("") : scale;
        float* o = p.out + (size_t)(ti * 256 + wr * 64 + 4 * h) * NN + tj * 256 + wc * 128 + 4 * i;
#pragma unroll
        for (int m = 0; m < 2; ++m)
#pragma unroll
            for (int e = 0; e < 16; ++e) {
                v4f v = {acc[m][0][e] * mul, acc[m][1][e] * mul, acc[m][2][e] * mul, acc[m][3][e] * mul};
                *(v4f*)(o + (size_t)(m * 32 + (e & 3) + 8 * (e >> 2)) * NN) = v;
            }
        return;
    }

    unsigned* ctr = (unsigned*)(myws + WS_CTR + (size_t)b * 64);
    const unsigned prev = rfl(__hip_atomic_load(ctr, __ATOMIC_RELAXED, __HIP_MEMORY_SCOPE_AGENT));
    const unsigned call = prev + 1u;
    const unsigned fbase = prev * ROUNDS;
    const size_t par = WS_DATA + (size_t)(call & 1u) * PAR_BYTES;

    if (wave == 10) {
        const int q = xcd * 4 + (idx & 3), d = idx >> 2;
        const unsigned dst = (unsigned)(p.rank + NR - d) & 7u;
        unsigned char* dws = p.ws[dst];
        unsigned char* dblk = dws + par + (size_t)p.rank * SRC_BYTES + (size_t)q * BLK;
        unsigned* dflag = (unsigned*)(dws + WS_FLAG + q * 128 + p.rank * 4);
        __builtin_amdgcn_s_barrier();
        STAMP(8);
        if (q < 16) {
            const unsigned char* src = p.x + (size_t)(q * 256 + (lane >> 1)) * KS + (lane & 1) * 16;
            const unsigned voff = (lane & 1) * 4096 + (lane >> 1) * 16;
            v4u v[2][8];
#pragma unroll
            for (int t = 0; t < 8; ++t) v[0][t] = *(const v4u*)(src + (size_t)t * 32 * KS);
#pragma unroll 1
            for (int r2 = 0; r2 < ROUNDS; r2 += 2) {
#pragma unroll
                for (int z = 0; z < 2; ++z) {
                    const int r = r2 + z;
                    const int rn = r + 1 < ROUNDS ? r + 1 : r;
#pragma unroll
                    for (int t = 0; t < 8; ++t) v[z ^ 1][t] = *(const v4u*)(src + (size_t)t * 32 * KS + rn * 32);
                    unsigned char* db = dblk + (size_t)r * NPANEL * BLK;
#pragma unroll
                    for (int t = 0; t < 8; ++t) st16_sys(db + t * 512, voff, v[z][t]);
                    WAIT_V(0);
                    if (lane == 0) __hip_atomic_store(dflag, fbase + r + 1, __ATOMIC_RELAXED, __HIP_MEMORY_SCOPE_SYSTEM);
                }
            }
        } else {
            const unsigned char* src = p.w + (size_t)(q - 16) * 256 + lane * 4;
            const unsigned voff = lane * 16;
#pragma unroll 1
            for (int r = 0; r < ROUNDS; ++r) {
                unsigned u[32];
#pragma unroll
                for (int k = 0; k < 32; ++k) u[k] = *(const unsigned*)(src + (size_t)(r * 32 + k) * NN);
                unsigned char* db = dblk + (size_t)r * NPANEL * BLK;
#pragma unroll
                for (int j = 0; j < 2; ++j) {
                    v4u o[4];
#pragma unroll
                    for (int t = 0; t < 4; ++t) {
                        const unsigned a0 = u[16 * j + 4 * t], a1 = u[16 * j + 4 * t + 1], a2 = u[16 * j + 4 * t + 2], a3 = u[16 * j + 4 * t + 3];
                        const unsigned x0 = __builtin_amdgcn_perm(a1, a0, 0x05010400u), x1 = __builtin_amdgcn_perm(a1, a0, 0x07030602u);
                        const unsigned y0 = __builtin_amdgcn_perm(a3, a2, 0x05010400u), y1 = __builtin_amdgcn_perm(a3, a2, 0x07030602u);
                        o[0][t] = __builtin_amdgcn_perm(y0, x0, 0x05040100u);
                        o[1][t] = __builtin_amdgcn_perm(y0, x0, 0x07060302u);
                        o[2][t] = __builtin_amdgcn_perm(y1, x1, 0x05040100u);
                        o[3][t] = __builtin_amdgcn_perm(y1, x1, 0x07060302u);
                    }
#pragma unroll
                    for (int c = 0; c < 4; ++c) st16_sys(db + j * 4096 + c * 1024, voff, o[c]);
                }
                WAIT_V(0);
                if (lane == 0) __hip_atomic_store(dflag, fbase + r + 1, __ATOMIC_RELAXED, __HIP_MEMORY_SCOPE_SYSTEM);
            }
        }
        STAMP(9);
        return;
    }

    if (wave == 11) {
        __builtin_amdgcn_fence(__ATOMIC_ACQUIRE, "");
        WAIT_V(0);
        __builtin_amdgcn_s_barrier();
        if (lane == 0) __hip_atomic_store(ctr, call, __ATOMIC_RELAXED, __HIP_MEMORY_SCOPE_AGENT);
        const int d = lane & 7;
        const unsigned srcr = (unsigned)(p.rank + d) & 7u;
        const unsigned* addr = lane < 8 ? (const unsigned*)(myws + WS_FLAG + ti * 128 + srcr * 4)
                             : lane < 16 ? (const unsigned*)(myws + WS_FLAG + (16 + tj) * 128 + srcr * 4)
                             : (const unsigned*)(myws + WS_TMO);
        const unsigned long long t0 = rt();
        for (;;) {
            unsigned v = __hip_atomic_load(addr, __ATOMIC_RELAXED, __HIP_MEMORY_SCOPE_SYSTEM);
            const unsigned tmo = __builtin_amdgcn_readlane(v, 16);
            bool dead = tmo == call;
            if (!dead && rt() - t0 > DEADLINE) {
                if (lane == 0) __hip_atomic_store((unsigned*)(myws + WS_TMO), call, __ATOMIC_RELAXED, __HIP_MEMORY_SCOPE_AGENT);
                dead = true;
            }
            int n = (int)(v - fbase);
            n = n < 0 ? 0 : n;
            const int other = __shfl_xor(n, 8);
            n = n < other ? n : other;
            if (dead) { n = ROUNDS; lds_st(L, L_BAD, 1u); }
            FENCE();
            if (lane < 8) lds_st(L, L_READY + lane * 4, (unsigned)n);
            const bool all = __ballot(lane < 8 && n < ROUNDS) == 0ull;
            if (all) break;
            __builtin_amdgcn_s_sleep(4);
        }
        STAMP(10);
        return;
    }

    {
        const int ld = wave - 8;
        __builtin_amdgcn_s_barrier();
        FENCE();
        const unsigned voff = lane * 16;
        const unsigned char* dbase = myws + par;
        const unsigned lbase = (unsigned)(uintptr_t)L;
        int pub = 0;
        for (int s = 0; s < NSTEP; ++s) {
            int d, r; unit_of(2 * s + ld, d, r);
            for (unsigned spin = 0;; ++spin) {
                const unsigned pr = lds_ld(L, L_PROG + (lane & 7) * 4);
                const unsigned rd = rfl(lds_ld(L, L_READY + d * 4));
                const bool freeok = __ballot((int)pr < s - (NSLOT - 1)) == 0ull;
                if ((freeok && rd > (unsigned)r) || spin > (1u << 26)) break;
                if (pub < s) { WAIT_V(0); pub = s; lds_st(L, L_FULL + ld * 4, (unsigned)pub); }
                else __builtin_amdgcn_s_sleep(1);
            }
            FENCE();
            const unsigned srcr = (unsigned)(p.rank + d) & 7u;
            const unsigned char* g = dbase + (size_t)srcr * SRC_BYTES + (size_t)r * NPANEL * BLK;
            const unsigned la = lbase + (s & (NSLOT - 1)) * SLOT + ld * 8192;
            dma8k(g + ti * BLK, la, voff);
            dma8k(g + (16 + tj) * BLK, la + 16384, voff);
            if (pub < s) { WAIT_V(16); pub = s; lds_st(L, L_FULL + ld * 4, (unsigned)pub); }
        }
        WAIT_V(0);
        lds_st(L, L_FULL + ld * 4, (unsigned)NSTEP);
        if (ld == 0) STAMP(11);
    }
}

}

extern "C" void kernel_launch(int rank, int nranks, ncclComm_t comm, hipStream_t stream,
                              void* const* d_in, const int* in_sizes, int n_in,
                              void* d_out, int out_size, const int* in_dtypes, int out_dtype,
                              void* const* d_out_all, void* const* d_ws_all, size_t ws_bytes) {
    if (nranks != NR || n_in != 4 || ws_bytes < WS_NEED || in_sizes[0] != MM * KS || in_sizes[1] != KS * NN ||
        out_size != MM * NN || (unsigned)rank >= (unsigned)NR) {
        fprintf(stderr, "kernel_launch: unexpected problem geometry\n");
        return;
    }
    Params p;
    p.x = (const unsigned char*)d_in[0]; p.w = (const unsigned char*)d_in[1];
    p.sx = (const float*)d_in[2]; p.sw = (const float*)d_in[3];
    p.out = (float*)d_out; p.rank = rank;
    for (int r = 0; r < NR; ++r) p.ws[r] = (unsigned char*)d_ws_all[r];
    hipLaunchKernelGGL(fused_gemm_ag, dim3(NWG), dim3(NTHREADS), 0, stream, p);
    HIP_CHECK(hipGetLastError());
}
```

```cpp
#include <hip/hip_runtime.h>
#include <rccl/rccl.h>
#include <cstdio>
#include <cstdint>
#include <type_traits>

#define HIP_CHECK(expr) do { hipError_t _e = (expr); if (_e != hipSuccess) { \
    fprintf(stderr, "HIP error %s at %s:%d\n", hipGetErrorString(_e), __FILE__, __LINE__); \
    return; } } while (0)

namespace {

#define LAS __attribute__((address_space(3)))
typedef int v4i __attribute__((ext_vector_type(4)));
typedef int v8i __attribute__((ext_vector_type(8)));
typedef unsigned v2u __attribute__((ext_vector_type(2)));
typedef unsigned v4u __attribute__((ext_vector_type(4)));
typedef float v4f __attribute__((ext_vector_type(4)));
typedef float v16f __attribute__((ext_vector_type(16)));
typedef unsigned long long u64;

constexpr int NR = 8;
constexpr int MM = 4096, NN = 4096, KS = 512;
constexpr int NCH = 32;
constexpr int NPANEL = 32;
constexpr int CH = 4096;
constexpr int NWG = 256, NTHREADS = 768;
constexpr int NSTEP = 64;
constexpr int NQ = NR * NCH;
constexpr size_t WS_CTR = 0;
constexpr size_t WS_FLAG = 16384;
constexpr size_t WS_TMO = 16384 + 4096;
constexpr size_t WS_DBG = 32768;
constexpr size_t WS_DATA = 1u << 20;
constexpr size_t ROW_BYTES = (size_t)NPANEL * CH;
constexpr size_t SRC_BYTES = (size_t)NCH * ROW_BYTES;
constexpr size_t PAR_BYTES = (size_t)NR * SRC_BYTES;
constexpr size_t WS_NEED = WS_DATA + 2 * PAR_BYTES;
constexpr u64 DEADLINE = 40000000ull;
constexpr int NSLOT = 4, SLOT = 32768;
constexpr int L_CTL = NSLOT * SLOT;
constexpr int L_FULL = L_CTL;
constexpr int L_PROG = L_CTL + 64;
constexpr int L_BAD = L_CTL + 128;
constexpr int L_QUEUE = L_CTL + 256;
constexpr int L_CALL = L_QUEUE + (NQ + 64) * 4;
constexpr int LDS_BYTES = L_CALL + 64;

#define WIN 2
#define PERIOD 56
#define MAP 0
#define DISP 0
#define LEAD 200
#define POLLSLEEP 2
#define ROT 0
#define LDMODE 1
#define OST 1
#define FGRAN 2
#define FTAIL 4

struct Params {
    const unsigned char* x; const unsigned char* w; const float* sx; const float* sw;
    float* out; unsigned char* ws[NR]; int rank;
};

__device__ __forceinline__ u64 rt() { return __builtin_amdgcn_s_memrealtime(); }
__device__ __forceinline__ unsigned rfl(unsigned v) { return __builtin_amdgcn_readfirstlane(v); }
__device__ __forceinline__ unsigned lds_ld(LAS unsigned char* L, int off) { return *(volatile LAS unsigned*)(L + off); }
__device__ __forceinline__ void lds_st(LAS unsigned char* L, int off, unsigned v) { *(volatile LAS unsigned*)(L + off) = v; }
#define FENCE() asm volatile("" ::: "memory")
template <int N> __device__ __forceinline__ void wait_vm() { asm volatile("s_waitcnt vmcnt(%0)" :: "n"(N) : "memory"); }

__device__ __forceinline__ unsigned vm_count() {
    unsigned r;
    asm volatile("s_getreg_b32 %0, hwreg(HW_REG_IB_STS)" : "=s"(r) :: "memory");
    return (r & 15u) | ((r >> 18) & 48u);
}

#define STAMP(i) do {} while (0)

__device__ __forceinline__ void dma4k(const unsigned char* g, unsigned ldsaddr, unsigned voff) {
    asm volatile(
        "s_mov_b32 m0, %2\n\t"
        "s_nop 0\n\t"
        "global_load_lds_dwordx4 %0, %1 offset:0\n\t"
        "global_load_lds_dwordx4 %0, %1 offset:1024\n\t"
        "global_load_lds_dwordx4 %0, %1 offset:2048\n\t"
        "global_load_lds_dwordx4 %0, %1 offset:3072\n\t"
        :: "v"(voff), "s"(g), "s"(ldsaddr) : "memory");
}
template <int OFF> __device__ __forceinline__ void st16_sys(unsigned char* sbase, unsigned voff, v4u v) {
    asm volatile("global_store_dwordx4 %0, %1, %2 offset:%3 sc0 sc1\n\ts_nop 1" :: "v"(voff), "v"(v), "s"(sbase), "n"(OFF) : "memory");
}
__device__ __forceinline__ void st4_sys_lane0(unsigned* p, unsigned v, int lane) {
    if (lane == 0) asm volatile("global_store_dword %0, %1, %2 sc0 sc1" :: "v"(0u), "v"(v), "s"(p) : "memory");
}
__device__ __forceinline__ void ld4(unsigned& d, unsigned voff, const unsigned char* sbase) {
    asm volatile("global_load_dword %0, %1, %2" : "=v"(d) : "v"(voff), "s"(sbase) : "memory");
}
__device__ __forceinline__ void ld16(v4u& d, unsigned voff, const unsigned char* sbase) {
    asm volatile("global_load_dwordx4 %0, %1, %2" : "=v"(d) : "v"(voff), "s"(sbase) : "memory");
}
__device__ __forceinline__ void landed(unsigned (&g)[16]) {
    asm volatile("" : "+v"(g[0]), "+v"(g[1]), "+v"(g[2]), "+v"(g[3]), "+v"(g[4]), "+v"(g[5]), "+v"(g[6]), "+v"(g[7]),
                      "+v"(g[8]), "+v"(g[9]), "+v"(g[10]), "+v"(g[11]), "+v"(g[12]), "+v"(g[13]), "+v"(g[14]), "+v"(g[15]) :: "memory");
}
__device__ __forceinline__ void landed(v4u& a, v4u& b, v4u& c, v4u& d) {
    asm volatile("" : "+v"(a), "+v"(b), "+v"(c), "+v"(d) :: "memory");
}
template <int N> using ic = std::integral_constant<int, N>;

template <bool IS_W>
__device__ __forceinline__ void send_panel(const Params& p, LAS unsigned char* L, unsigned char* myws, int q, int d, int lane,
                                           u64 t0, unsigned period) {
    const unsigned char* src = IS_W ? p.w + (size_t)(q - 16) * 256 : p.x + (size_t)q * 256 * KS;
    const unsigned lo = IS_W ? lane * 4 : (lane >> 2) * 4 * KS + (lane & 3) * 16;
    const unsigned so = lane * 16;
    unsigned gw[2][16];
    v4u ow[2][4];
    v4u gx[2][16];
    auto loadw = [&](int i, unsigned (&g)[16]) {
#pragma unroll
        for (int k = 0; k < 16; ++k) ld4(g[k], lo, src + (size_t)(i * 16 + k) * NN);
    };
    auto xformw = [&](unsigned (&g)[16], v4u (&ob)[4]) {
        landed(g);
#pragma unroll
        for (int t = 0; t < 4; ++t) {
            const unsigned a0 = g[4 * t], a1 = g[4 * t + 1], a2 = g[4 * t + 2], a3 = g[4 * t + 3];
            const unsigned x0 = __builtin_amdgcn_perm(a1, a0, 0x05010400u), x1 = __builtin_amdgcn_perm(a1, a0, 0x07030602u);
            const unsigned y0 = __builtin_amdgcn_perm(a3, a2, 0x05010400u), y1 = __builtin_amdgcn_perm(a3, a2, 0x07030602u);
            ob[0][t] = __builtin_amdgcn_perm(y0, x0, 0x05040100u);
            ob[1][t] = __builtin_amdgcn_perm(y0, x0, 0x07060302u);
            ob[2][t] = __builtin_amdgcn_perm(y1, x1, 0x05040100u);
            ob[3][t] = __builtin_amdgcn_perm(y1, x1, 0x07060302u);
        }
        landed(ob[0], ob[1], ob[2], ob[3]);
    };
    auto loadx = [&](int grp, int half, v4u (&g)[16]) {
#pragma unroll
        for (int t = 0; t < 8; ++t) ld16(g[8 * half + t], lo, src + (size_t)(128 * half + 64 * (t >> 2) + (t & 3)) * KS + grp * 64);
    };
    const bool b0 = lane & 1, b1 = lane & 2;
    auto xformx = [&](int half, v4u (&g)[16]) {
#pragma unroll
        for (int j = 2 * half; j < 2 * half + 2; ++j) {
            v4u &r0 = g[4 * j], &r1 = g[4 * j + 1], &r2 = g[4 * j + 2], &r3 = g[4 * j + 3];
            landed(r0, r1, r2, r3);
#pragma unroll
            for (int e = 0; e < 4; ++e) {
                const unsigned a0 = r0[e], a1 = r1[e], a2 = r2[e], a3 = r3[e];
                const unsigned u0 = __builtin_amdgcn_mov_dpp(a0, 0xB1, 0xf, 0xf, true), u1 = __builtin_amdgcn_mov_dpp(a1, 0xB1, 0xf, 0xf, true);
                const unsigned u2 = __builtin_amdgcn_mov_dpp(a2, 0xB1, 0xf, 0xf, true), u3 = __builtin_amdgcn_mov_dpp(a3, 0xB1, 0xf, 0xf, true);
                const unsigned n0 = b0 ? u1 : a0, n1 = b0 ? a1 : u0, n2 = b0 ? u3 : a2, n3 = b0 ? a3 : u2;
                const unsigned w0 = __builtin_amdgcn_mov_dpp(n0, 0x4E, 0xf, 0xf, true), w1 = __builtin_amdgcn_mov_dpp(n1, 0x4E, 0xf, 0xf, true);
                const unsigned w2 = __builtin_amdgcn_mov_dpp(n2, 0x4E, 0xf, 0xf, true), w3 = __builtin_amdgcn_mov_dpp(n3, 0x4E, 0xf, 0xf, true);
                r0[e] = b1 ? w2 : n0; r2[e] = b1 ? n2 : w0; r1[e] = b1 ? w3 : n1; r3[e] = b1 ? n3 : w1;
            }
            landed(r0, r1, r2, r3);
        }
    };
    unsigned* ctr = (unsigned*)(myws + WS_CTR + (size_t)blockIdx.x * 64);
    unsigned prevv;
    asm volatile("global_load_dword %0, %1, %2 sc1" : "=v"(prevv) : "v"(0u), "s"(ctr) : "memory");
    if constexpr (IS_W) { loadw(0, gw[0]); loadw(1, gw[1]); } else { loadx(0, 0, gx[0]); loadx(0, 1, gx[0]); }
    asm volatile("s_waitcnt vmcnt(%1)" : "+v"(prevv) : "n"(IS_W ? 32 : 16) : "memory");
    const unsigned prev = rfl(prevv), call = prev + 1u, fbase = prev * NCH;
    lds_st(L, L_CALL, call);
    asm volatile("s_waitcnt lgkmcnt(0)" ::: "memory");
    __builtin_amdgcn_s_barrier();
    FENCE();
    STAMP(8);
    const unsigned dst = (unsigned)(p.rank + NR - d) & 7u;
    unsigned char* dws = p.ws[dst];
    unsigned char* db = dws + WS_DATA + (size_t)(call & 1u) * PAR_BYTES + (size_t)p.rank * SRC_BYTES + (size_t)q * CH;
    unsigned* dflag = (unsigned*)(dws + WS_FLAG + q * 128 + p.rank * 4);
    unsigned fmask = 0;
    for (int i = 0; i < NCH; ++i) if ((i + d) % FGRAN == FGRAN - 1 || i >= NCH - FTAIL) fmask |= 1u << i;
    fmask = rfl(fmask);
    if constexpr (IS_W) { wait_vm<16>(); xformw(gw[0], ow[0]); } else { wait_vm<0>(); xformx(0, gx[0]); xformx(1, gx[0]); }
    STAMP(12);
    u64 tt = t0;
    auto pace = [&]() {
        while ((long long)(rt() - tt) < 0) __builtin_amdgcn_s_sleep(1);
        tt += period;
    };
    auto piece = [&](auto nl, int i, v4u& s0, v4u& s1, v4u& s2, v4u& s3, auto&& loads, auto&& xform) {
        constexpr int NL = decltype(nl)::value;
        const bool fl = i > 0 && ((fmask >> (i - 1)) & 1u);
        pace(); st16_sys<0>(db, so, s0);
        wait_vm<WIN>();
        if (WIN == 1 && fl) st4_sys_lane0(dflag, fbase + i, lane);
        pace(); st16_sys<1024>(db, so, s1);
        if (WIN == 1 && fl) wait_vm<WIN + 1>(); else wait_vm<WIN>();
        if (WIN == 2 && fl) st4_sys_lane0(dflag, fbase + i, lane);
        loads();
        pace(); st16_sys<2048>(db, so, s2);
        if (WIN == 2 && fl) wait_vm<NL + WIN + 1>(); else wait_vm<NL + WIN>();
        xform();
        pace(); st16_sys<3072>(db, so, s3);
        if (WIN == 1) wait_vm<1>(); else if (fl) wait_vm<NL + WIN + 1>(); else wait_vm<NL + WIN>();
        db += ROW_BYTES;
    };
    if constexpr (IS_W) {
#pragma unroll 1
        for (int i = 0; i < NCH; i += 2) {
            piece(ic<16>{}, i, ow[0][0], ow[0][1], ow[0][2], ow[0][3],
                  [&]() { loadw(i + 2 < NCH ? i + 2 : NCH - 1, gw[0]); }, [&]() { xformw(gw[1], ow[1]); });
            piece(ic<16>{}, i + 1, ow[1][0], ow[1][1], ow[1][2], ow[1][3],
                  [&]() { loadw(i + 3 < NCH ? i + 3 : NCH - 1, gw[1]); }, [&]() { xformw(gw[0], ow[0]); });
        }
    } else {
        auto group = [&](int g, v4u (&c)[16], v4u (&n)[16]) {
            const int gn = g + 1 < NCH / 4 ? g + 1 : g;
            piece(ic<8>{}, 4 * g, c[0], c[4], c[8], c[12], [&]() { loadx(gn, 0, n); }, [&]() {});
            piece(ic<8>{}, 4 * g + 1, c[1], c[5], c[9], c[13], [&]() { loadx(gn, 1, n); }, [&]() {});
            piece(ic<0>{}, 4 * g + 2, c[2], c[6], c[10], c[14], [&]() {}, [&]() { xformx(0, n); });
            piece(ic<0>{}, 4 * g + 3, c[3], c[7], c[11], c[15], [&]() {}, [&]() { xformx(1, n); });
        };
#pragma unroll 1
        for (int g = 0; g < NCH / 4; g += 2) {
            group(g, gx[0], gx[1]);
            group(g + 1, gx[1], gx[0]);
        }
    }
    wait_vm<0>();
    st4_sys_lane0(dflag, fbase + NCH, lane);
    if (lane == 0) __hip_atomic_store(ctr, call, __ATOMIC_RELAXED, __HIP_MEMORY_SCOPE_AGENT);
    STAMP(9);
}

__global__ void __launch_bounds__(NTHREADS) fused_gemm_ag(Params p) {
    __shared__ __attribute__((aligned(1024))) unsigned char lds_raw[LDS_BYTES];
    LAS unsigned char* L = (LAS unsigned char*)lds_raw;
    const int tid = threadIdx.x, lane = tid & 63;
    const int wave = rfl(tid >> 6);
    const int b = blockIdx.x, xcd = b & 7, idx = b >> 3;
    const int ti = (xcd >> 1) * 4 + (idx >> 3), tj = (xcd & 1) * 8 + (idx & 7);
    unsigned char* myws = p.ws[p.rank & 7];

    if (wave < 8) {
        if (wave == 0) STAMP(0);
        if (wave < 6) lds_st(L, L_CTL + wave * 256 + lane * 4, 0u);
        asm volatile("s_waitcnt lgkmcnt(0)" ::: "memory");
        __builtin_amdgcn_s_barrier();
        FENCE();
        const int wr = wave >> 1, wc = wave & 1, i = lane & 31, h = lane >> 5;
        const int abase = h * 8192 + (64 * wr + i) * 16;
        const int bbase = 16384 + h * 8192 + (32 * wc + i) * 16;
        v16f acc[2][4];
#pragma unroll
        for (int m = 0; m < 2; ++m)
#pragma unroll
            for (int c = 0; c < 4; ++c)
#pragma unroll
                for (int e = 0; e < 16; ++e) acc[m][c][e] = 0.f;
        for (int s = 0; s < NSTEP; ++s) {
            for (unsigned spin = 0;; ++spin) {
                v2u f = *(volatile LAS v2u*)(L + L_FULL);
                unsigned mn = rfl(f.x < f.y ? f.x : f.y);
                if (mn >= (unsigned)(4 * s + 4) || spin > (1u << 26)) break;
                __builtin_amdgcn_s_sleep(1);
            }
            FENCE();
            LAS unsigned char* sl = L + (s & (NSLOT - 1)) * SLOT;
            v8i a[2], bf[4];
#pragma unroll
            for (int m = 0; m < 2; ++m) {
                v4i lo = *(LAS v4i*)(sl + abase + m * 512), hi = *(LAS v4i*)(sl + abase + m * 512 + 4096);
                a[m] = __builtin_shufflevector(lo, hi, 0, 1, 2, 3, 4, 5, 6, 7);
            }
#pragma unroll
            for (int c = 0; c < 4; ++c) {
                v4i lo = *(LAS v4i*)(sl + bbase + c * 1024), hi = *(LAS v4i*)(sl + bbase + c * 1024 + 4096);
                bf[c] = __builtin_shufflevector(lo, hi, 0, 1, 2, 3, 4, 5, 6, 7);
                if (c == 3) {
                    asm volatile("s_waitcnt lgkmcnt(0)" : "+v"(bf[3]) :: "memory");
                    lds_st(L, L_PROG + wave * 4, (unsigned)(s + 1));
                    FENCE();
                }
#pragma unroll
                for (int m = 0; m < 2; ++m)
                    acc[m][c] = __builtin_amdgcn_mfma_scale_f32_32x32x64_f8f6f4(a[m], bf[c], acc[m][c], 0, 0, 0, 0, 0, 0);
            }
        }
        if (wave == 0) STAMP(1);
        const float scale = p.sx[0] * p.sw[0];
        const bool bad = lds_ld(L, L_BAD) != 0u;
        const float mul = bad ? __builtin_nanf("") : scale;
        float* o = p.out + (size_t)(ti * 256 + wr * 64 + 4 * h) * NN + tj * 256 + wc * 128 + 4 * i;
#pragma unroll
        for (int m = 0; m < 2; ++m)
#pragma unroll
            for (int e = 0; e < 16; ++e) {
                v4f v = {acc[m][0][e] * mul, acc[m][1][e] * mul, acc[m][2][e] * mul, acc[m][3][e] * mul};
                float* po = o + (size_t)(m * 32 + (e & 3) + 8 * (e >> 2)) * NN;
                asm volatile("global_store_dwordx4 %0, %1, off sc0 sc1\n\ts_nop 1" :: "v"(po), "v"(v) : "memory");
            }
        return;
    }

    if (wave == 10) {
        const u64 tstart = rt();
        const int q = xcd * 4 + (idx & 3), d = ((idx >> 2) + 1) & 7;
        const unsigned period = d == 0 ? 0u : (unsigned)PERIOD;
        const int ord = (q & 15) * 2 + (q >> 4);
        const u64 t0 = tstart - (unsigned)(idx * DISP) + (unsigned)(LEAD + ord * PERIOD / 32);
        if (q < 16) send_panel<false>(p, L, myws, q, d, lane, t0, period);
        else send_panel<true>(p, L, myws, q, d, lane, t0, period);
        return;
    }

    if (wave == 11) {
        __builtin_amdgcn_s_barrier();
        FENCE();
        const unsigned call = rfl(lds_ld(L, L_CALL));
        const unsigned fbase = (call - 1u) * NCH;
        __builtin_amdgcn_fence(__ATOMIC_ACQUIRE, "");
        wait_vm<0>();
        const int d = (lane + (ROT ? (idx & 7) + 2 * (idx >> 3) : 0)) & 7;
        const unsigned srcr = (unsigned)(p.rank + d) & 7u;
        const unsigned* addr = lane < 8 ? (const unsigned*)(myws + WS_FLAG + ti * 128 + srcr * 4)
                             : lane < 16 ? (const unsigned*)(myws + WS_FLAG + (16 + tj) * 128 + srcr * 4)
                             : (const unsigned*)(myws + WS_TMO);
        const unsigned ebase = (call & 1u) * NQ + srcr * NCH + 1u;
        const u64 t0 = rt();
        int seen = 0, qn = 0;
        int sst = 0; (void)sst;
        for (;;) {
            unsigned v = __hip_atomic_load(addr, __ATOMIC_RELAXED, __HIP_MEMORY_SCOPE_SYSTEM);
            const unsigned tmo = __builtin_amdgcn_readlane(v, 16);
            bool dead = tmo == call;
            if (!dead && rt() - t0 > DEADLINE) {
                if (lane == 0) __hip_atomic_store((unsigned*)(myws + WS_TMO), call, __ATOMIC_RELAXED, __HIP_MEMORY_SCOPE_AGENT);
                dead = true;
            }
            int n = (int)(v - fbase);
            n = n < 0 ? 0 : n > NCH ? NCH : n;
            const int other = __shfl_xor(n, 8);
            n = n < other ? n : other;
            if (dead) { n = NCH; lds_st(L, L_BAD, 1u); }
            int nw = lane < 8 ? n - seen : 0;
            nw = nw < 0 ? 0 : nw;
            if (__ballot(nw > 0)) {
                int pre = nw;
#pragma unroll
                for (int o = 1; o < 8; o <<= 1) { const int t = __shfl_up(pre, o); if (lane >= o) pre += t; }
                const int tot = __builtin_amdgcn_readlane(pre, 7);
                int pos = qn + pre - nw;
                FENCE();
                for (int j = 0; j < nw; ++j) lds_st(L, L_QUEUE + (pos + j) * 4, ebase + seen + j);
                seen += nw;
                qn += tot;
            }
            if (qn >= NQ) break;
            __builtin_amdgcn_s_sleep(POLLSLEEP);
        }
        STAMP(10);
        return;
    }

    {
        const int ld = wave - 8;
        __builtin_amdgcn_s_barrier();
        FENCE();
        const unsigned voff = lane * 16;
        const unsigned char* dbase = myws + WS_DATA - ROW_BYTES + (size_t)(ld ? 16 + tj : ti) * CH;
        const unsigned lbase = (unsigned)(uintptr_t)L + ld * 16384;
        int iss = 0, done = 0;
        unsigned spin = 0;
        while (done < NQ) {
            const unsigned v = lds_ld(L, lane < 16 ? L_PROG + (lane & 7) * 4 : L_QUEUE + (iss + (lane & 15)) * 4);
            int lim = iss + __builtin_ctzll(~(__ballot(v != 0u) >> 16));
            lim = lim < iss + 16 ? lim : iss + 16;
            lim = lim < NQ ? lim : NQ;
            lim = lim < done + 15 ? lim : done + 15;
            const int st = iss >> 2;
            int fr = st + NSLOT;
#pragma unroll
            for (int j = NSLOT - 1; j >= 0; --j) if (__ballot(lane < 8 && (int)v < st + j - (NSLOT - 1))) fr = st + j;
            lim = lim < fr * 4 ? lim : fr * 4;
            FENCE();
            const bool any = iss < lim;
            if (any) {
                const int i0 = iss;
                for (; iss < lim; ++iss) {
                    const unsigned e = __builtin_amdgcn_readlane(v, 16 + iss - i0);
                    dma4k(dbase + (size_t)e * ROW_BYTES, lbase + ((iss >> 2) & (NSLOT - 1)) * SLOT + (iss & 3) * CH, voff);
                }
            }
            const int tg = iss - (int)((vm_count() + 3u) >> 2);
            const bool adv = tg > done;
            if (adv) {
                switch (iss - tg) {
                    case 0: wait_vm<0>(); break;   case 1: wait_vm<4>(); break;   case 2: wait_vm<8>(); break;
                    case 3: wait_vm<12>(); break;  case 4: wait_vm<16>(); break;  case 5: wait_vm<20>(); break;
                    case 6: wait_vm<24>(); break;  case 7: wait_vm<28>(); break;  case 8: wait_vm<32>(); break;
                    case 9: wait_vm<36>(); break;  case 10: wait_vm<40>(); break; case 11: wait_vm<44>(); break;
                    case 12: wait_vm<48>(); break; case 13: wait_vm<52>(); break; default: wait_vm<56>(); break;
                }
                done = tg;
                lds_st(L, L_FULL + ld * 4, (unsigned)done);
            } else if (!any) {
                if (++spin > (1u << 26)) { lds_st(L, L_BAD, 1u); lds_st(L, L_FULL + ld * 4, (unsigned)NQ); break; }
                __builtin_amdgcn_s_sleep(1);
            }
        }
        if (ld == 0) STAMP(11);
    }
}

}

extern "C" void kernel_launch(int rank, int nranks, ncclComm_t comm, hipStream_t stream,
                              void* const* d_in, const int* in_sizes, int n_in,
                              void* d_out, int out_size, const int* in_dtypes, int out_dtype,
                              void* const* d_out_all, void* const* d_ws_all, size_t ws_bytes) {
    if (nranks != NR || n_in != 4 || ws_bytes < WS_NEED || in_sizes[0] != MM * KS || in_sizes[1] != KS * NN ||
        out_size != MM * NN || (unsigned)rank >= (unsigned)NR) {
        fprintf(stderr, "kernel_launch: unexpected problem geometry\n");
        return;
    }
    Params p;
    p.x = (const unsigned char*)d_in[0]; p.w = (const unsigned char*)d_in[1];
    p.sx = (const float*)d_in[2]; p.sw = (const float*)d_in[3];
    p.out = (float*)d_out; p.rank = rank;
    for (int r = 0; r < NR; ++r) p.ws[r] = (unsigned char*)d_ws_all[r];
    hipLaunchKernelGGL(fused_gemm_ag, dim3(NWG), dim3(NTHREADS), 0, stream, p);
    HIP_CHECK(hipGetLastError());
}
```

```cpp
#include <hip/hip_runtime.h>
#include <rccl/rccl.h>
#include <cstdio>
#include <cstdint>
#include <type_traits>

#define HIP_CHECK(expr) do { hipError_t _e = (expr); if (_e != hipSuccess) { \
    fprintf(stderr, "HIP error %s at %s:%d\n", hipGetErrorString(_e), __FILE__, __LINE__); \
    return; } } while (0)

namespace {

#define LAS __attribute__((address_space(3)))
typedef int v4i __attribute__((ext_vector_type(4)));
typedef int v8i __attribute__((ext_vector_type(8)));
typedef unsigned v2u __attribute__((ext_vector_type(2)));
typedef unsigned v4u __attribute__((ext_vector_type(4)));
typedef float v4f __attribute__((ext_vector_type(4)));
typedef float v16f __attribute__((ext_vector_type(16)));
typedef unsigned long long u64;

constexpr int NR = 8;
constexpr int MM = 4096, NN = 4096, KS = 512;
constexpr int NCH = 32;
constexpr int NBULK = 28;
constexpr int NPANEL = 32;
constexpr int CH = 4096;
constexpr int NWG = 256, NTHREADS = 768;
constexpr int NQ = (NR - 1) * NBULK + NCH;
constexpr int NSB = NQ / 4;
constexpr int NPAIR = 2 * (NR - 1);
constexpr int NFIN = 4 * NPAIR;
constexpr unsigned FSTRIDE = 64;
constexpr size_t WS_CTR = 0;
constexpr size_t WS_FLAG = 16384;
constexpr size_t WS_TMO = 16384 + 4096;
constexpr size_t WS_DBG = 32768;
constexpr size_t WS_DATA = 1u << 20;
constexpr size_t ROW_BYTES = (size_t)NPANEL * CH;
constexpr size_t SRC_BYTES = (size_t)NCH * ROW_BYTES;
constexpr size_t PAR_BYTES = (size_t)NR * SRC_BYTES;
constexpr size_t WS_NEED = WS_DATA + 2 * PAR_BYTES;
constexpr unsigned DEADLINE = 40000000u;
constexpr unsigned DEADLINE2 = 60000000u;
constexpr int NSLOT = 4, SLOT = 32768;
constexpr int L_XR = NPAIR * 2 * CH;
constexpr int NXB = 11;
constexpr int L_CTL = L_XR + NXB * CH;
constexpr int L_FULL = L_CTL;
constexpr int L_PROG = L_CTL + 64;
constexpr int L_BAD = L_CTL + 128;
constexpr int L_QUEUE = L_CTL + 256;
constexpr int L_FIN = L_CTL + 1536;
constexpr int L_CALL = L_CTL + 2048;
constexpr int LDS_BYTES = L_CALL + 64 + 256;
static_assert(L_QUEUE + (NQ + 32) * 4 <= L_FIN && L_FIN + (NFIN + 32) * 4 <= L_CALL && L_CTL >= NSLOT * SLOT, "LDS map");

#define PER8 448
#define PERX8 456
#define LEAD 200
#define POLLSLEEP 2
#define LATE1 100000
#define LATE2 100000
#define XBATCH 8
#define WBATCH 4
#define OWIN 63
#define XMID 1

struct Params {
    const unsigned char* x; const unsigned char* w; const float* sx; const float* sw;
    float* out; unsigned char* ws[NR]; int rank;
    unsigned dtab;
};

__device__ __forceinline__ u64 rt() { return __builtin_amdgcn_s_memrealtime(); }
__device__ __forceinline__ unsigned rfl(unsigned v) { return __builtin_amdgcn_readfirstlane(v); }
__device__ __forceinline__ unsigned lds_ld(LAS unsigned char* L, int off) { return *(volatile LAS unsigned*)(L + off); }
__device__ __forceinline__ void lds_st(LAS unsigned char* L, int off, unsigned v) { *(volatile LAS unsigned*)(L + off) = v; }
#define FENCE() asm volatile("" ::: "memory")
template <int N> __device__ __forceinline__ void wait_vm() { asm volatile("s_waitcnt vmcnt(%0)" :: "n"(N) : "memory"); }
__device__ __forceinline__ int vm_count() {
    unsigned r;
    asm volatile("s_getreg_b32 %0, hwreg(HW_REG_IB_STS)" : "=s"(r) :: "memory");
    return (int)((r & 15u) | ((r >> 18) & 48u));
}
__device__ __forceinline__ bool expired(unsigned& spin, unsigned t0) {
    return (++spin & 4095u) == 0u && (unsigned)rt() - t0 > DEADLINE2;
}
template <int CTRL> __device__ __forceinline__ int dpp0(int v) { return __builtin_amdgcn_update_dpp(0, v, CTRL, 0xf, 0xf, true); }

#define STAMPL(i) do {} while (0)
#define STAMP(i) do {} while (0)

__device__ __forceinline__ void dma4k(const unsigned char* g, unsigned ldsaddr, unsigned voff) {
    asm volatile("s_mov_b32 m0, %2\n\ts_nop 0\n\t"
                 "global_load_lds_dwordx4 %0, %1 offset:0\n\tglobal_load_lds_dwordx4 %0, %1 offset:1024\n\t"
                 "global_load_lds_dwordx4 %0, %1 offset:2048\n\tglobal_load_lds_dwordx4 %0, %1 offset:3072"
                 :: "v"(voff), "s"(g), "s"(ldsaddr) : "memory");
}
__device__ __forceinline__ void dma2k(const unsigned char* g, unsigned ldsaddr, unsigned voff) {
    asm volatile("s_mov_b32 m0, %2\n\ts_nop 0\n\t"
                 "global_load_lds_dwordx4 %0, %1 offset:0\n\tglobal_load_lds_dwordx4 %0, %1 offset:1024"
                 :: "v"(voff), "s"(g), "s"(ldsaddr) : "memory");
}
__device__ __forceinline__ void st16_sys(unsigned char* sbase, unsigned voff, v4u v) {
    asm volatile("global_store_dwordx4 %0, %1, %2 sc0 sc1\n\ts_nop 1" :: "v"(voff), "v"(v), "s"(sbase) : "memory");
}
__device__ __forceinline__ void st4_sys_lane0(unsigned* p, unsigned v) {
    u64 ex;
    asm volatile("s_mov_b64 %0, exec\n\ts_mov_b64 exec, 1\n\tglobal_store_dword %1, %2, %3 sc0 sc1\n\ts_mov_b64 exec, %0"
                 : "=&s"(ex) : "v"(0u), "v"(v), "s"(p) : "memory");
}
__device__ __forceinline__ void ld4(unsigned& d, unsigned voff, const unsigned char* sbase) {
    asm volatile("global_load_dword %0, %1, %2" : "=v"(d) : "v"(voff), "s"(sbase) : "memory");
}
template <int OFF> __device__ __forceinline__ void ld16(v4u& d, unsigned voff, const unsigned char* sbase) {
    asm volatile("global_load_dwordx4 %0, %1, %2 offset:%3" : "=v"(d) : "v"(voff), "s"(sbase), "n"(OFF) : "memory");
}
__device__ __forceinline__ void landed(unsigned (&g)[16]) {
    asm volatile("" : "+v"(g[0]), "+v"(g[1]), "+v"(g[2]), "+v"(g[3]), "+v"(g[4]), "+v"(g[5]), "+v"(g[6]), "+v"(g[7]),
                      "+v"(g[8]), "+v"(g[9]), "+v"(g[10]), "+v"(g[11]), "+v"(g[12]), "+v"(g[13]), "+v"(g[14]), "+v"(g[15]) :: "memory");
}
__device__ __forceinline__ void landed(v4u& a, v4u& b, v4u& c, v4u& d) {
    asm volatile("" : "+v"(a), "+v"(b), "+v"(c), "+v"(d) :: "memory");
}
__device__ __forceinline__ void swap16(v4u& a, v4u& b, unsigned mask) {
    unsigned x0 = a[0], x1 = a[1], x2 = a[2], x3 = a[3], y0 = b[0], y1 = b[1], y2 = b[2], y3 = b[3];
    asm volatile("s_mov_b32 exec_lo, %8\n\ts_mov_b32 exec_hi, %8\n\tv_swap_b32 %0, %4\n\tv_swap_b32 %1, %5\n\tv_swap_b32 %2, %6\n\tv_swap_b32 %3, %7\n\t"
                 "s_mov_b64 exec, -1"
                 : "+v"(x0), "+v"(x1), "+v"(x2), "+v"(x3), "+v"(y0), "+v"(y1), "+v"(y2), "+v"(y3) : "s"(mask));
    a[0] = x0; a[1] = x1; a[2] = x2; a[3] = x3; b[0] = y0; b[1] = y1; b[2] = y2; b[3] = y3;
}

template <bool IS_W>
__device__ __forceinline__ void send_panel(const Params& p, LAS unsigned char* L, unsigned char* myws, int q, int d, int lane, unsigned t0, unsigned per) {
    const bool self = d == 0;
    const unsigned char* src = IS_W ? p.w + (size_t)(q - 16) * 256 : p.x + (size_t)q * 256 * KS;
    const unsigned lo = IS_W ? lane * 4 : (lane >> 2) * 4 * KS + (lane & 3) * 16;
    const unsigned so = lane * 16;
    unsigned gw[4][16];
    v4u ow[2][4];
    v4u gx[2][16];
    auto loadw = [&](int i, unsigned (&g)[16]) {
#pragma unroll
        for (int k = 0; k < 16; ++k) ld4(g[k], lo, src + (size_t)(i * 16 + k) * NN);
    };
    auto xformw = [&](unsigned (&g)[16], v4u (&ob)[4]) {
        landed(g);
#pragma unroll
        for (int t = 0; t < 4; ++t) {
            const unsigned a0 = g[4 * t], a1 = g[4 * t + 1], a2 = g[4 * t + 2], a3 = g[4 * t + 3];
            const unsigned x0 = __builtin_amdgcn_perm(a1, a0, 0x05010400u), x1 = __builtin_amdgcn_perm(a1, a0, 0x07030602u);
            const unsigned y0 = __builtin_amdgcn_perm(a3, a2, 0x05010400u), y1 = __builtin_amdgcn_perm(a3, a2, 0x07030602u);
            ob[0][t] = __builtin_amdgcn_perm(y0, x0, 0x05040100u);
            ob[1][t] = __builtin_amdgcn_perm(y0, x0, 0x07060302u);
            ob[2][t] = __builtin_amdgcn_perm(y1, x1, 0x05040100u);
            ob[3][t] = __builtin_amdgcn_perm(y1, x1, 0x07060302u);
        }
        landed(ob[0], ob[1], ob[2], ob[3]);
    };
    auto loadx = [&](int grp, int half, v4u (&g)[16]) {
#pragma unroll
        for (int j = 2 * half; j < 2 * half + 2; ++j) {
            const unsigned char* s = src + (size_t)j * 64 * KS + grp * 64;
            ld16<0>(g[4 * j], lo, s); ld16<KS>(g[4 * j + 1], lo, s); ld16<2 * KS>(g[4 * j + 2], lo, s); ld16<3 * KS>(g[4 * j + 3], lo, s);
        }
    };
    const bool b0 = lane & 1, b1 = lane & 2;
    auto xformx = [&](int half, v4u (&g)[16]) {
#pragma unroll
        for (int j = 2 * half; j < 2 * half + 2; ++j) {
            v4u &r0 = g[4 * j], &r1 = g[4 * j + 1], &r2 = g[4 * j + 2], &r3 = g[4 * j + 3];
            landed(r0, r1, r2, r3);
#pragma unroll
            for (int e = 0; e < 4; ++e) {
                const unsigned a0 = r0[e], a1 = r1[e], a2 = r2[e], a3 = r3[e];
                const unsigned u0 = __builtin_amdgcn_mov_dpp(a0, 0xB1, 0xf, 0xf, true), u1 = __builtin_amdgcn_mov_dpp(a1, 0xB1, 0xf, 0xf, true);
                const unsigned u2 = __builtin_amdgcn_mov_dpp(a2, 0xB1, 0xf, 0xf, true), u3 = __builtin_amdgcn_mov_dpp(a3, 0xB1, 0xf, 0xf, true);
                const unsigned n0 = b0 ? u1 : a0, n1 = b0 ? a1 : u0, n2 = b0 ? u3 : a2, n3 = b0 ? a3 : u2;
                const unsigned w0 = __builtin_amdgcn_mov_dpp(n0, 0x4E, 0xf, 0xf, true), w1 = __builtin_amdgcn_mov_dpp(n1, 0x4E, 0xf, 0xf, true);
                const unsigned w2 = __builtin_amdgcn_mov_dpp(n2, 0x4E, 0xf, 0xf, true), w3 = __builtin_amdgcn_mov_dpp(n3, 0x4E, 0xf, 0xf, true);
                r0[e] = b1 ? w2 : n0; r2[e] = b1 ? n2 : w0; r1[e] = b1 ? w3 : n1; r3[e] = b1 ? n3 : w1;
            }
            landed(r0, r1, r2, r3);
        }
    };
    unsigned* ctr = (unsigned*)(myws + WS_CTR + (size_t)blockIdx.x * 64);
    unsigned prevv;
    asm volatile("global_load_dword %0, %1, %2 sc1" : "=v"(prevv) : "v"(0u), "s"(ctr) : "memory");
    if constexpr (IS_W) { loadw(0, gw[0]); loadw(1, gw[1]); loadw(2, gw[2]); } else { loadx(0, 0, gx[0]); loadx(0, 1, gx[0]); }
    asm volatile("s_waitcnt vmcnt(%1)" : "+v"(prevv) : "n"(IS_W ? 48 : 16) : "memory");
    const unsigned prev = rfl(prevv), call = prev + 1u, fbase = prev * FSTRIDE;
    lds_st(L, L_CALL, call);
    asm volatile("s_waitcnt lgkmcnt(0)" ::: "memory");
    __builtin_amdgcn_s_barrier();
    FENCE();
    const unsigned dst = (unsigned)(p.rank + NR - d) & 7u;
    unsigned char* dws = p.ws[dst];
    unsigned char* db = dws + WS_DATA + (size_t)(call & 1u) * PAR_BYTES + (size_t)p.rank * SRC_BYTES + (size_t)q * CH;
    unsigned* dflag = (unsigned*)(dws + WS_FLAG + q * 128 + p.rank * 4);
    const unsigned fmask = ((d & 1) ? 0x05555555u : 0x0AAAAAAAu) | 0x08000000u | ((IS_W || self) ? 0xA0000000u : 0xF0000000u);
    if constexpr (IS_W) { wait_vm<32>(); xformw(gw[0], ow[0]); } else { wait_vm<0>(); xformx(0, gx[0]); xformx(1, gx[0]); }
    int ops = IS_W ? 32 : 0, h1 = 0, h2 = 0, h3 = 0, h4 = 0, h5 = 0, h6 = 0;
    int l1 = IS_W ? 16 : 0, l2 = IS_W ? 32 : 0, l3 = 0;
    int ia = 0, ib = 0;
    unsigned fa = 0, fb = 0;
    unsigned tt = 0, period = self ? 0u : per;
    bool wide = self;
    auto flag_try = [&](int ret) {
        if (ia != 0 && ret >= ia) { st4_sys_lane0(dflag, fa); ++ops; ia = ib; fa = fb; ib = 0; }
    };
    auto flag_put = [&](unsigned v) {
        if (ia == 0) { ia = h1; fa = v; } else { ib = h1; fb = v; }
    };
    auto slot = [&](unsigned char* a, v4u& v) {
        const int n0 = wide ? h4 + 0 : h2 + 0, n1 = wide ? h6 + 0 : h3 + 0, n2 = wide ? h6 + 0 : h4 + 0;
        const unsigned due = t0 + (tt >> 3);
        for (;;) {
            const int late = (int)((unsigned)rt() - due);
            const int ret = ops - vm_count();
            flag_try(ret);
            if (late >= 0 && ret >= (late >= LATE2 ? n2 : late >= LATE1 ? n1 : n0)) break;
            __builtin_amdgcn_s_sleep(1);
        }
        st16_sys(a, so, v);
        h6 = h5; h5 = h4; h4 = h3; h3 = h2; h2 = h1; h1 = ++ops;
        tt += period;
    };
    auto piece = [&](int i, v4u& s0, v4u& s1, v4u& s2, v4u& s3, auto&& loads, auto&& xform) {
        if (i == NBULK + 1 && !self) {
            tt += (IS_W ? 0u : 6u * per) - (unsigned)((q & 15) + (q >> 4)) * (per / 32);
            period = per / 2; wide = true;
        }
        const unsigned fv = fbase + ((IS_W || self || i < NBULK) ? i + 1 : 2 * i - (NBULK - 2));
        slot(db, s0);
        slot(db + 1024, s1);
        if (XMID && !IS_W && !self && i > NBULK) flag_put(fv - 1u);
        loads();
        slot(db + 2048, s2);
        xform();
        slot(db + 3072, s3);
        if ((fmask >> i) & 1u) flag_put(fv);
        db += ROW_BYTES;
    };
    auto arrived = [&](int l) { while (ops - vm_count() < l) __builtin_amdgcn_s_sleep(1); };
    STAMP(12);
    if constexpr (IS_W) {
        auto wpiece = [&](int i, v4u (&oc)[4], v4u (&on)[4], unsigned (&gl)[16], unsigned (&gt)[16]) {
            piece(i, oc[0], oc[1], oc[2], oc[3],
                  [&]() { if (i + 3 < NCH) { loadw(i + 3, gl); ops += 16; } l3 = ops; },
                  [&]() { arrived(l1); xformw(gt, on); l1 = l2; l2 = l3; });
        };
#pragma unroll 1
        for (int i = 0; i < NCH; i += 4) {
            wpiece(i, ow[0], ow[1], gw[3], gw[1]);
            wpiece(i + 1, ow[1], ow[0], gw[0], gw[2]);
            wpiece(i + 2, ow[0], ow[1], gw[1], gw[3]);
            wpiece(i + 3, ow[1], ow[0], gw[2], gw[0]);
        }
    } else {
        auto group = [&](int g, v4u (&c)[16], v4u (&n)[16]) {
            const bool more = g + 1 < NCH / 4;
            piece(4 * g, c[0], c[4], c[8], c[12], [&]() { if (more) { loadx(g + 1, 0, n); ops += 8; l1 = ops; } }, [&]() {});
            piece(4 * g + 1, c[1], c[5], c[9], c[13], [&]() { if (more) { loadx(g + 1, 1, n); ops += 8; l2 = ops; } }, [&]() {});
            piece(4 * g + 2, c[2], c[6], c[10], c[14], [&]() {}, [&]() { arrived(l1); xformx(0, n); });
            piece(4 * g + 3, c[3], c[7], c[11], c[15], [&]() {}, [&]() { arrived(l2); xformx(1, n); });
        };
#pragma unroll 1
        for (int g = 0; g < NCH / 4; g += 2) {
            group(g, gx[0], gx[1]);
            const unsigned last = rfl(g == NCH / 4 - 2 && !self ? ~0u : 0u);
#pragma unroll
            for (int a = 0; a < 4; ++a)
#pragma unroll
                for (int c = a + 1; c < 4; ++c) swap16(gx[1][4 * a + c], gx[1][4 * c + a], last);
            group(g + 1, gx[1], gx[0]);
        }
    }
    while (ia != 0) flag_try(ops - vm_count());
    if (lane == 0) __hip_atomic_store(ctr, call, __ATOMIC_RELAXED, __HIP_MEMORY_SCOPE_AGENT);
    STAMP(9);
}

__global__ void __launch_bounds__(NTHREADS) fused_gemm_ag(Params p) {
    __shared__ __attribute__((aligned(1024))) unsigned char lds_raw[LDS_BYTES];
    LAS unsigned char* L = (LAS unsigned char*)lds_raw;
    const int tid = threadIdx.x, lane = tid & 63;
    const int wave = rfl(tid >> 6);
    const int b = blockIdx.x, xcd = b & 7, idx = b >> 3;
    const int ti = (xcd >> 1) * 4 + (idx >> 3), tj = (xcd & 1) * 8 + (idx & 7);
    unsigned char* myws = p.ws[p.rank & 7];
    const unsigned tstart = (unsigned)rt();

    if (wave < 8) {
        if (wave == 0) STAMP(0);
        lds_st(L, L_CTL + wave * 256 + lane * 4, 0u);
        float scale = p.sx[0] * p.sw[0];
        asm volatile("s_waitcnt lgkmcnt(0)" ::: "memory");
        __builtin_amdgcn_s_barrier();
        FENCE();
        scale = __builtin_bit_cast(float, rfl(__builtin_bit_cast(unsigned, scale)));
        const int wr = wave >> 1, wc = wave & 1, i = lane & 31, h = lane >> 5;
        unsigned char* otile = (unsigned char*)(p.out + (size_t)(ti * 256 + wr * 64) * NN + tj * 256 + wc * 128);
        asm volatile("" : "+s"(scale), "+s"(otile));
        const int abase = h * 8192 + (64 * wr + i) * 16;
        const int bbase = 16384 + h * 8192 + (32 * wc + i) * 16;
        v16f acc[2][4];
#pragma unroll
        for (int m = 0; m < 2; ++m)
#pragma unroll
            for (int c = 0; c < 4; ++c)
#pragma unroll
                for (int e = 0; e < 16; ++e) acc[m][c][e] = 0.f;
        unsigned spin = 0;
        for (int s = 0; s < NSB; ++s) {
            for (;;) {
                v2u f = *(volatile LAS v2u*)(L + L_FULL);
                unsigned mn = rfl(f.x < f.y ? f.x : f.y);
                if (mn >= (unsigned)(4 * s + 4)) break;
                if (expired(spin, tstart)) { lds_st(L, L_BAD, 1u); break; }
                __builtin_amdgcn_s_sleep(1);
            }
            FENCE();
            LAS unsigned char* sl = L + (s & (NSLOT - 1)) * SLOT;
            v8i a[2], bf[4];
#pragma unroll
            for (int m = 0; m < 2; ++m) {
                v4i lo = *(LAS v4i*)(sl + abase + m * 512), hi = *(LAS v4i*)(sl + abase + m * 512 + 4096);
                a[m] = __builtin_shufflevector(lo, hi, 0, 1, 2, 3, 4, 5, 6, 7);
            }
#pragma unroll
            for (int c = 0; c < 4; ++c) {
                v4i lo = *(LAS v4i*)(sl + bbase + c * 1024), hi = *(LAS v4i*)(sl + bbase + c * 1024 + 4096);
                bf[c] = __builtin_shufflevector(lo, hi, 0, 1, 2, 3, 4, 5, 6, 7);
                if (c == 3) {
                    asm volatile("s_waitcnt lgkmcnt(0)" : "+v"(bf[3]) :: "memory");
                    lds_st(L, L_PROG + wave * 4, (unsigned)(s + 1 < NSB ? s + 1 : NSB + 7 * wr));
                    FENCE();
                }
#pragma unroll
                for (int m = 0; m < 2; ++m)
                    acc[m][c] = __builtin_amdgcn_mfma_scale_f32_32x32x64_f8f6f4(a[m], bf[c], acc[m][c], 0, 0, 0, 0, 0, 0);
            }
        }
        if (wave == 0) STAMP(1);
        unsigned va;
        {
            const unsigned lb = (unsigned)(uintptr_t)L;
            unsigned vb = lb + (32 * wc + i) * 16, t0, t1;
            unsigned m = 7 * wr, blk = (7 * wr + NXB - 1) % NXB, cnt, tmp, rel;
            va = lb + L_XR + blk * CH + h * 2048 + i * 16;
            u64 now;
#define POLL_ADDR \
            "v_bfe_u32 %[t0], %[va], 11, 1\n\ts_lshl3_add_u32 %[tmp], %[m], %[fin]\n\t" \
            "v_lshl_add_u32 %[t0], %[t0], 2, %[tmp]\n\t"
#define MOVE \
            "s_add_u32 %[blk], %[blk], 1\n\ts_cmp_eq_u32 %[blk], %[nxb]\n\ts_cselect_b32 %[blk], 0, %[blk]\n\t" \
            "s_cselect_b32 %[tmp], %[back], %[fwd]\n\t" \
            "v_and_b32 %[t0], 0xf000, %[t0]\n\tv_and_b32 %[vb], 0x1fff, %[vb]\n\tv_add_u32 %[va], %[tmp], %[va]\n\t" \
            "v_lshl_add_u32 %[vb], %[t0], 1, %[vb]\n\t"
#define RD(r0, r1, adr, off, hi) \
            "ds_read_b128 v[" #r0 "], %[" #adr "] offset:" #off "\n\tds_read_b128 v[" #r1 "], %[" #adr "] offset:" #off "+" #hi "\n\t"
#define RD_A0 RD(136:139, 140:143, va, 0, 1024)
#define RD_A1 RD(144:147, 148:151, va, 512, 1024)
#define RD_B0(off) RD(152:155, 156:159, vb, off, 4096)
#define RD_B1(off) RD(160:163, 164:167, vb, off, 4096)
#define MM_(c, a, b) "v_mfma_f32_32x32x64_f8f6f4 %[" #c "], v[" #a "], v[" #b "], %[" #c "]\n\t"
            asm volatile(
                "s_mov_b32 %[cnt], 7\n"
                "1:\n\t"
                POLL_ADDR
                "ds_read_b32 %[t0], %[t0]\n\ts_waitcnt lgkmcnt(0)\n\t"
                "v_cmp_le_u32 vcc, 0x01010000, %[t0]\n\ts_cmp_eq_u64 vcc, -1\n\ts_cbranch_scc1 3f\n\t"
                "s_memrealtime %[now]\n\ts_waitcnt lgkmcnt(0)\n\t"
                "s_mov_b64 vcc, %[now]\n\ts_sub_u32 %[tmp], vcc_lo, %[ts]\n\t"
                "s_cmp_gt_u32 %[tmp], %[dl]\n\ts_cbranch_scc0 1b\n\t"
                "v_mov_b32 %[t0], %[bad]\n\tv_mov_b32 %[t1], 1\n\tds_write_b32 %[t0], %[t1]\n\t"
                "v_mov_b32 %[t0], %[prog]\n\tv_mov_b32 %[t1], 1000\n\tds_write_b32 %[t0], %[t1]\n\t"
                "s_branch 9f\n"
                "3:\n\t"
                MOVE
                RD_B0(0) RD_A0 RD_A1
                "\n4:\n\t"
                RD_B1(1024)
                "s_add_u32 %[m], %[m], 1\n\ts_add_u32 %[rel], %[m], %[nsb]\n\ts_sub_u32 %[cnt], %[cnt], 1\n\t"
                "s_cmp_eq_u32 %[cnt], 0\n\ts_cselect_b32 %[rel], 1000, %[rel]\n\t"
                "v_mov_b32 %[t0], %[prog]\n\tv_mov_b32 %[t1], %[rel]\n\t"
                "s_waitcnt lgkmcnt(4)\n\t"
                MM_(c00, 136:143, 152:159)
                "s_waitcnt lgkmcnt(2)\n\t"
                MM_(c10, 144:151, 152:159)
                "ds_write_b32 %[t0], %[t1]\n\t"
                RD_B0(2048)
                POLL_ADDR
                "s_waitcnt lgkmcnt(3)\n\t"
                MM_(c01, 136:143, 160:167)
                "ds_read_b32 %[t0], %[t0]\n\t"
                MM_(c11, 144:151, 160:167)
                RD_B1(3072)
                "s_waitcnt lgkmcnt(3)\n\t"
                MM_(c02, 136:143, 152:159)
                "s_waitcnt lgkmcnt(2)\n\t"
                "v_cmp_le_u32 vcc, 0x01010000, %[t0]\n\t"
                MM_(c12, 144:151, 152:159)
                "s_cmp_eq_u32 %[cnt], 0\n\ts_cbranch_scc1 6f\n\t"
                "s_cmp_eq_u64 vcc, -1\n\ts_cbranch_scc0 6f\n\t"
                MOVE
                RD_B0(0)
                "s_waitcnt lgkmcnt(2)\n\t"
                MM_(c03, 136:143, 160:167)
                RD_A0
                MM_(c13, 144:151, 160:167)
                RD_A1
                "s_branch 4b\n"
                "6:\n\t"
                "s_waitcnt lgkmcnt(0)\n\t"
                MM_(c03, 136:143, 160:167)
                MM_(c13, 144:151, 160:167)
                "s_cmp_eq_u32 %[cnt], 0\n\ts_cbranch_scc0 1b\n"
                "9:\n\t"
                "s_waitcnt lgkmcnt(0)\n\ts_nop 15\n\ts_nop 15"
                : [c00] "+v"(acc[0][0]), [c01] "+v"(acc[0][1]), [c02] "+v"(acc[0][2]), [c03] "+v"(acc[0][3]),
                  [c10] "+v"(acc[1][0]), [c11] "+v"(acc[1][1]), [c12] "+v"(acc[1][2]), [c13] "+v"(acc[1][3]),
                  [va] "+v"(va), [vb] "+v"(vb), [t0] "=&v"(t0), [t1] "=&v"(t1),
                  [m] "+s"(m), [blk] "+s"(blk), [cnt] "=&s"(cnt), [tmp] "=&s"(tmp), [rel] "=&s"(rel), [now] "=&s"(now)
                : [fin] "s"(lb + L_FIN), [prog] "s"(lb + L_PROG + wave * 4), [bad] "s"(lb + L_BAD), [ts] "s"(tstart),
                  [dl] "s"(DEADLINE2), [nsb] "n"(NSB), [nxb] "n"(NXB), [fwd] "s"(CH), [back] "n"(-(NXB - 1) * CH)
                : "memory", "vcc", "scc",
                  "v136", "v137", "v138", "v139", "v140", "v141", "v142", "v143", "v144", "v145", "v146", "v147",
                  "v148", "v149", "v150", "v151", "v152", "v153", "v154", "v155", "v156", "v157", "v158", "v159",
                  "v160", "v161", "v162", "v163", "v164", "v165", "v166", "v167");
        }
        if (wc == 0) STAMP(20 + wr);
        const bool bad = lds_ld(L, L_BAD) != 0u;
        const float mul = bad ? __builtin_nanf("") : scale;
        const unsigned oo = (va & 0x800u) * 32u + (va & 0x1f0u);
#pragma unroll
        for (int m = 0; m < 2; ++m)
#pragma unroll
            for (int e = 0; e < 16; ++e) {
                v4f v = {acc[m][0][e] * mul, acc[m][1][e] * mul, acc[m][2][e] * mul, acc[m][3][e] * mul};
                st16_sys(otile + (size_t)(m * 32 + (e & 3) + 8 * (e >> 2)) * NN * 4, oo, __builtin_bit_cast(v4u, v));
                if (OWIN < 32 && wr < 3) wait_vm<OWIN>();
            }
        return;
    }

    __builtin_amdgcn_s_setprio(3);
    if (wave == 10) {
        const int q = (idx & 4) * 4 + (xcd & 3) * 4 + (idx & 3);
        const int d = (p.dtab >> (4 * ((xcd & 4) + (idx >> 3)))) & 7;
        const int ord = (q & 15) * 2 + (q >> 4);
        const unsigned per = xcd < 4 ? PER8 : PERX8;
        const unsigned t0 = tstart + (unsigned)LEAD + ord * per / 256;
        if (q < 16) send_panel<false>(p, L, myws, q, d, lane, t0, per);
        else send_panel<true>(p, L, myws, q, d, lane, t0, per);
        return;
    }

    if (wave == 11) {
        asm volatile("buffer_inv sc0 sc1" ::: "memory");
        __builtin_amdgcn_s_barrier();
        FENCE();
        const unsigned call = rfl(lds_ld(L, L_CALL));
        const unsigned fbase = (call - 1u) * FSTRIDE;
        wait_vm<0>();
        const int d = lane & 7;
        const unsigned srcr = (unsigned)(p.rank + d) & 7u;
        const unsigned* addr = lane < 8 ? (const unsigned*)(myws + WS_FLAG + ti * 128 + srcr * 4)
                             : lane < 16 ? (const unsigned*)(myws + WS_FLAG + (16 + tj) * 128 + srcr * 4)
                             : (const unsigned*)(myws + WS_TMO);
        const unsigned row0 = ((call & 1u) * NR + srcr) * NCH;
        const int nmax = lane >= 8 ? 0 : d == 0 ? NCH : NBULK, amax = lane >= 8 || d == 0 ? 0 : 8;
        int seen = 0, qn = 0;
        int seen2 = 0, fq = 0, glim = 2;
        for (;;) {
            unsigned v = __hip_atomic_load(addr, __ATOMIC_RELAXED, __HIP_MEMORY_SCOPE_SYSTEM);
            const unsigned tmo = __builtin_amdgcn_readlane(v, 16);
            bool dead = tmo == call;
            if (!dead && (unsigned)rt() - tstart > DEADLINE) {
                if (lane == 0) __hip_atomic_store((unsigned*)(myws + WS_TMO), call, __ATOMIC_RELAXED, __HIP_MEMORY_SCOPE_AGENT);
                dead = true;
            }
            int nx = (int)(v - fbase), nw = dpp0<0x128>(nx);
            if (dead) { nx = nw = 64; lds_st(L, L_BAD, 1u); }
            int nn = min(min(nx, nw), nmax) - seen;
            nn = max(nn, 0);
            if (__ballot(nn > 0)) {
                int pre = nn;
                pre += dpp0<0x111>(pre); pre += dpp0<0x112>(pre); pre += dpp0<0x114>(pre);
                const int pos = qn + pre - nn;
                FENCE();
                for (int j = 0; j < nn; ++j) lds_st(L, L_QUEUE + (pos + j) * 4, row0 + seen + j + 1u);
                seen += nn;
                qn += __builtin_amdgcn_readlane(pre, 7);
            }
            int a = min(nx - NBULK, amax);
            if (nw < NCH) a = min(a, (nw - NBULK) >> 1);
            for (;;) {
                const int n2 = max(min(a, glim) - seen2, 0);
                if (!__ballot(n2 > 0)) break;
                int pre = n2;
                pre += dpp0<0x111>(pre); pre += dpp0<0x112>(pre); pre += dpp0<0x114>(pre);
                const int pos = fq + pre - n2;
                FENCE();
                for (int j = 0; j < n2; ++j) {
                    const unsigned pr = (seen2 + j) & 1;
                    *(volatile LAS unsigned short*)(L + L_FIN + (pos + j) * 4) =
                        (unsigned short)((2 * d - 2 + pr) << 12 | (row0 + NBULK + ((seen2 + j) >> 1)));
                }
                seen2 += n2;
                fq += __builtin_amdgcn_readlane(pre, 7);
                if (fq != glim * (NR - 1)) break;
                STAMP(31 + glim / 2);
                glim += 2;
            }
            if (qn >= NQ && fq >= NFIN) break;
            __builtin_amdgcn_s_sleep(POLLSLEEP);
        }
        STAMP(10);
        return;
    }

    {
        const int ld = wave - 8;
        __builtin_amdgcn_s_barrier();
        FENCE();
        const unsigned voff = lane * 16;
        const unsigned char* dbase = myws + WS_DATA + (size_t)(ld ? 16 + tj : ti) * CH;
        const unsigned lb = (unsigned)(uintptr_t)L;
        int iss = 0, done = 0;
        unsigned spin = 0;
        while (done < NQ) {
            const unsigned v = lds_ld(L, lane < 16 ? L_PROG + (lane & 7) * 4 : L_QUEUE + (iss + (lane & 15)) * 4);
            int lim = iss + __builtin_ctzll(~(__ballot(v != 0u) >> 16));
            lim = min(min(lim, iss + 16), min(NQ, done + 15));
            const int st = iss >> 2;
            int fr = st + NSLOT;
#pragma unroll
            for (int j = NSLOT - 1; j >= 0; --j) if (__ballot(lane < 8 && (int)v < st + j - (NSLOT - 1))) fr = st + j;
            lim = min(lim, fr * 4);
            FENCE();
            const bool any = iss < lim;
            for (const int i0 = iss; iss < lim; ++iss) {
                const unsigned e = __builtin_amdgcn_readlane(v, 16 + iss - i0) - 1u;
                dma4k(dbase + (size_t)e * ROW_BYTES, lb + ld * 16384 + ((iss >> 2) & (NSLOT - 1)) * SLOT + (iss & 3) * CH, voff);
            }
            const int tg = iss - ((vm_count() + 3) >> 2);
            if (tg > done) {
                FENCE();
                done = tg;
                lds_st(L, L_FULL + ld * 4, (unsigned)done);
            } else if (!any) {
                if (expired(spin, tstart)) { lds_st(L, L_BAD, 1u); break; }
                __builtin_amdgcn_s_sleep(1);
            }
        }
        auto fin = [&](auto isw) {
            constexpr bool W = decltype(isw)::value;
            constexpr int NF = W ? NPAIR : NFIN;
            unsigned xa = lb + L_XR;
            iss = 0; done = 0;
            while (done < NF) {
                const unsigned v = lds_ld(L, lane < 16 ? L_PROG + (lane & 7) * 4 : L_FIN + (iss + (lane & 15)) * 4);
                int n = __builtin_ctz(~(unsigned)(__ballot((v & 0xfffu) != 0u) >> 16) | (1u << (W ? WBATCH : XBATCH)));
                int pm = (int)v;
                pm = min(pm, dpp0<0x121>(pm)); pm = min(pm, dpp0<0x122>(pm)); pm = min(pm, dpp0<0x124>(pm));
                pm = (int)rfl(pm) - NSB;
                int lim = W ? done + 7 : 2 * (pm + NXB);
                lim = lim < NF ? lim : NF;
                lim = pm < 0 ? 0 : lim - iss;
                n = n < lim ? n : lim;
                const unsigned off = (v & 0x1ffu) << 17 | (W ? (v & 0x1000u) << 6 : (v & 0x1000u) >> 1);
                FENCE();
                for (int j = 0; j < n; ++j) {
                    const unsigned vo = voff + __builtin_amdgcn_readlane(off, 16 + j);
                    if constexpr (W) {
                        const unsigned la = lb + (__builtin_amdgcn_readlane(v, 16 + j) & 0xf000u) * 2;
                        dma4k(dbase, la, vo);
                        dma4k(dbase + ROW_BYTES, la + CH, vo);
                    } else {
                        dma2k(dbase, xa, vo);
                        xa = xa == lb + L_CTL - 2048 ? lb + L_XR : xa + 2048;
                    }
                }
                if (n > 0) iss += n;
                const int vm = vm_count();
                const int tg = iss - (W ? (vm + 7) >> 3 : (vm + 1) >> 1);
                if (tg > done) {
                    FENCE();
                    const int last = tg == NF ? NFIN : tg;
                    if (done + lane < last) *(volatile LAS unsigned char*)(L + L_FIN + (done + lane) * 4 + 2 + W) = 1;
                    done = tg;
                } else if (n <= 0) {
                    if (expired(spin, tstart)) { lds_st(L, L_BAD, 1u); break; }
                    __builtin_amdgcn_s_sleep(1);
                }
            }
        };
        if (ld) fin(std::true_type{}); else fin(std::false_type{});
        STAMP(11 + 2 * ld);
    }
}

}

extern "C" void kernel_launch(int rank, int nranks, ncclComm_t comm, hipStream_t stream,
                              void* const* d_in, const int* in_sizes, int n_in,
                              void* d_out, int out_size, const int* in_dtypes, int out_dtype,
                              void* const* d_out_all, void* const* d_ws_all, size_t ws_bytes) {
    if (nranks != NR || n_in != 4 || ws_bytes < WS_NEED || in_sizes[0] != MM * KS || in_sizes[1] != KS * NN ||
        out_size != MM * NN || (unsigned)rank >= (unsigned)NR) {
        fprintf(stderr, "kernel_launch: unexpected problem geometry\n");
        return;
    }
    Params p;
    p.x = (const unsigned char*)d_in[0]; p.w = (const unsigned char*)d_in[1];
    p.sx = (const float*)d_in[2]; p.sw = (const float*)d_in[3];
    p.out = (float*)d_out; p.rank = rank;
    for (int r = 0; r < NR; ++r) p.ws[r] = (unsigned char*)d_ws_all[r];
    p.dtab = 0;
    for (int r = 0, lo = 0, hi = 4; r < NR; ++r) {
        const int dst = (rank + NR - 1 - r) & 7;
        const bool same = ((0xB2 >> dst) & 1) == ((0xB2 >> rank) & 1);
        p.dtab |= (unsigned)((rank - dst) & 7) << (4 * (same ? lo++ : hi++));
    }
    hipLaunchKernelGGL(fused_gemm_ag, dim3(NWG), dim3(NTHREADS), 0, stream, p);
    HIP_CHECK(hipGetLastError());
}
```

```cpp
#include <hip/hip_runtime.h>
#include <rccl/rccl.h>
#include <cstdio>
#include <cstdint>
#include <type_traits>

#define HIP_CHECK(expr) do { hipError_t _e = (expr); if (_e != hipSuccess) { \
    fprintf(stderr, "HIP error %s at %s:%d\n", hipGetErrorString(_e), __FILE__, __LINE__); \
    return; } } while (0)

namespace {

#define LAS __attribute__((address_space(3)))
typedef int v4i __attribute__((ext_vector_type(4)));
typedef int v8i __attribute__((ext_vector_type(8)));
typedef unsigned v2u __attribute__((ext_vector_type(2)));
typedef unsigned v4u __attribute__((ext_vector_type(4)));
typedef float v4f __attribute__((ext_vector_type(4)));
typedef float v16f __attribute__((ext_vector_type(16)));
typedef unsigned long long u64;

constexpr int NR = 8;
constexpr int MM = 4096, NN = 4096, KS = 512;
constexpr int NCH = 32;
constexpr int NBULK = 28;
constexpr int NPANEL = 32;
constexpr int CH = 4096;
constexpr int NWG = 256, NTHREADS = 768;
constexpr int NQ = (NR - 1) * NBULK + NCH;
constexpr int NSB = NQ / 4;
constexpr int NPAIR = 2 * (NR - 1);
constexpr int NFIN = 4 * NPAIR;
constexpr unsigned FSTRIDE = 64;
constexpr size_t WS_CTR = 0;
constexpr size_t WS_FLAG = 16384;
constexpr size_t WS_TMO = 16384 + 4096;
constexpr size_t WS_DBG = 32768;
constexpr size_t WS_DATA = 1u << 20;
constexpr size_t ROW_BYTES = (size_t)NPANEL * CH;
constexpr size_t SRC_BYTES = (size_t)NCH * ROW_BYTES;
constexpr size_t PAR_BYTES = (size_t)NR * SRC_BYTES;
constexpr size_t WS_NEED = WS_DATA + 2 * PAR_BYTES;
constexpr unsigned DEADLINE = 40000000u;
constexpr unsigned DEADLINE2 = 60000000u;
constexpr int NSLOT = 4, SLOT = 32768;
constexpr int L_XR = NPAIR * 2 * CH;
constexpr int NXB = 11;
constexpr int L_CTL = L_XR + NXB * CH;
constexpr int L_FULL = L_CTL;
constexpr int L_PROG = L_CTL + 64;
constexpr int L_BAD = L_CTL + 128;
constexpr int L_QUEUE = L_CTL + 256;
constexpr int L_FIN = L_CTL + 1536;
constexpr int L_CALL = L_CTL + 2048;
constexpr int LDS_BYTES = L_CALL + 64 + 256;
static_assert(L_QUEUE + (NQ + 32) * 4 <= L_FIN && L_FIN + (NFIN + 32) * 4 <= L_CALL && L_CTL >= NSLOT * SLOT, "LDS map");

#define PER8 448
#define PERX8 456
#define LEAD 200
#define POLLSLEEP 2
#define LATE1 100000
#define LATE2 100000
#define XBATCH 8
#define WBATCH 4
#define OWIN 63
#define XMID 1

struct Params {
    const unsigned char* x; const unsigned char* w; const float* sx; const float* sw;
    float* out; unsigned char* ws[NR]; int rank;
    unsigned dtab;
};

__device__ __forceinline__ u64 rt() { return __builtin_amdgcn_s_memrealtime(); }
__device__ __forceinline__ unsigned rfl(unsigned v) { return __builtin_amdgcn_readfirstlane(v); }
__device__ __forceinline__ unsigned lds_ld(LAS unsigned char* L, int off) { return *(volatile LAS unsigned*)(L + off); }
__device__ __forceinline__ void lds_st(LAS unsigned char* L, int off, unsigned v) { *(volatile LAS unsigned*)(L + off) = v; }
#define FENCE() asm volatile("" ::: "memory")
template <int N> __device__ __forceinline__ void wait_vm() { asm volatile("s_waitcnt vmcnt(%0)" :: "n"(N) : "memory"); }
__device__ __forceinline__ int vm_count() {
    unsigned r;
    asm volatile("s_getreg_b32 %0, hwreg(HW_REG_IB_STS)" : "=s"(r) :: "memory");
    return (int)((r & 15u) | ((r >> 18) & 48u));
}
__device__ __forceinline__ bool expired(unsigned& spin, unsigned t0) {
    return (++spin & 4095u) == 0u && (unsigned)rt() - t0 > DEADLINE2;
}
template <int CTRL> __device__ __forceinline__ int dpp0(int v) { return __builtin_amdgcn_update_dpp(0, v, CTRL, 0xf, 0xf, true); }

#define STAMPL(i) do {} while (0)
#define STAMP(i) do {} while (0)

__device__ __forceinline__ void dma4k(const unsigned char* g, unsigned ldsaddr, unsigned voff) {
    asm volatile("s_mov_b32 m0, %2\n\ts_nop 0\n\t"
                 "global_load_lds_dwordx4 %0, %1 offset:0\n\tglobal_load_lds_dwordx4 %0, %1 offset:1024\n\t"
                 "global_load_lds_dwordx4 %0, %1 offset:2048\n\tglobal_load_lds_dwordx4 %0, %1 offset:3072"
                 :: "v"(voff), "s"(g), "s"(ldsaddr) : "memory");
}
__device__ __forceinline__ void dma2k(const unsigned char* g, unsigned ldsaddr, unsigned voff) {
    asm volatile("s_mov_b32 m0, %2\n\ts_nop 0\n\t"
                 "global_load_lds_dwordx4 %0, %1 offset:0\n\tglobal_load_lds_dwordx4 %0, %1 offset:1024"
                 :: "v"(voff), "s"(g), "s"(ldsaddr) : "memory");
}
__device__ __forceinline__ void st16_sys(unsigned char* sbase, unsigned voff, v4u v) {
    asm volatile("global_store_dwordx4 %0, %1, %2 sc0 sc1\n\ts_nop 1" :: "v"(voff), "v"(v), "s"(sbase) : "memory");
}
__device__ __forceinline__ void st4_sys_lane0(unsigned* p, unsigned v) {
    u64 ex;
    asm volatile("s_mov_b64 %0, exec\n\ts_mov_b64 exec, 1\n\tglobal_store_dword %1, %2, %3 sc0 sc1\n\ts_mov_b64 exec, %0"
                 : "=&s"(ex) : "v"(0u), "v"(v), "s"(p) : "memory");
}
__device__ __forceinline__ void ld4(unsigned& d, unsigned voff, const unsigned char* sbase) {
    asm volatile("global_load_dword %0, %1, %2" : "=v"(d) : "v"(voff), "s"(sbase) : "memory");
}
template <int OFF> __device__ __forceinline__ void ld16(v4u& d, unsigned voff, const unsigned char* sbase) {
    asm volatile("global_load_dwordx4 %0, %1, %2 offset:%3" : "=v"(d) : "v"(voff), "s"(sbase), "n"(OFF) : "memory");
}
__device__ __forceinline__ void landed(unsigned (&g)[16]) {
    asm volatile("" : "+v"(g[0]), "+v"(g[1]), "+v"(g[2]), "+v"(g[3]), "+v"(g[4]), "+v"(g[5]), "+v"(g[6]), "+v"(g[7]),
                      "+v"(g[8]), "+v"(g[9]), "+v"(g[10]), "+v"(g[11]), "+v"(g[12]), "+v"(g[13]), "+v"(g[14]), "+v"(g[15]) :: "memory");
}
__device__ __forceinline__ void landed(v4u& a, v4u& b, v4u& c, v4u& d) {
    asm volatile("" : "+v"(a), "+v"(b), "+v"(c), "+v"(d) :: "memory");
}
__device__ __forceinline__ void swap16(v4u& a, v4u& b, unsigned mask) {
    unsigned x0 = a[0], x1 = a[1], x2 = a[2], x3 = a[3], y0 = b[0], y1 = b[1], y2 = b[2], y3 = b[3];
    asm volatile("s_mov_b32 exec_lo, %8\n\ts_mov_b32 exec_hi, %8\n\tv_swap_b32 %0, %4\n\tv_swap_b32 %1, %5\n\tv_swap_b32 %2, %6\n\tv_swap_b32 %3, %7\n\t"
                 "s_mov_b64 exec, -1"
                 : "+v"(x0), "+v"(x1), "+v"(x2), "+v"(x3), "+v"(y0), "+v"(y1), "+v"(y2), "+v"(y3) : "s"(mask));
    a[0] = x0; a[1] = x1; a[2] = x2; a[3] = x3; b[0] = y0; b[1] = y1; b[2] = y2; b[3] = y3;
}

template <bool IS_W>
__device__ __forceinline__ void send_panel(const Params& p, LAS unsigned char* L, unsigned char* myws, int q, int d, int lane, unsigned t0, unsigned per) {
    const bool self = d == 0;
    const unsigned char* src = IS_W ? p.w + (size_t)(q - 16) * 256 : p.x + (size_t)q * 256 * KS;
    const unsigned lo = IS_W ? lane * 4 : (lane >> 2) * 4 * KS + (lane & 3) * 16;
    const unsigned so = lane * 16;
    unsigned gw[4][16];
    v4u ow[2][4];
    v4u gx[2][16];
    auto loadw = [&](int i, unsigned (&g)[16]) {
#pragma unroll
        for (int k = 0; k < 16; ++k) ld4(g[k], lo, src + (size_t)(i * 16 + k) * NN);
    };
    auto xformw = [&](unsigned (&g)[16], v4u (&ob)[4]) {
        landed(g);
#pragma unroll
        for (int t = 0; t < 4; ++t) {
            const unsigned a0 = g[4 * t], a1 = g[4 * t + 1], a2 = g[4 * t + 2], a3 = g[4 * t + 3];
            const unsigned x0 = __builtin_amdgcn_perm(a1, a0, 0x05010400u), x1 = __builtin_amdgcn_perm(a1, a0, 0x07030602u);
            const unsigned y0 = __builtin_amdgcn_perm(a3, a2, 0x05010400u), y1 = __builtin_amdgcn_perm(a3, a2, 0x07030602u);
            ob[0][t] = __builtin_amdgcn_perm(y0, x0, 0x05040100u);
            ob[1][t] = __builtin_amdgcn_perm(y0, x0, 0x07060302u);
            ob[2][t] = __builtin_amdgcn_perm(y1, x1, 0x05040100u);
            ob[3][t] = __builtin_amdgcn_perm(y1, x1, 0x07060302u);
        }
        landed(ob[0], ob[1], ob[2], ob[3]);
    };
    auto loadx = [&](int grp, int half, v4u (&g)[16]) {
#pragma unroll
        for (int j = 2 * half; j < 2 * half + 2; ++j) {
            const unsigned char* s = src + (size_t)j * 64 * KS + grp * 64;
            ld16<0>(g[4 * j], lo, s); ld16<KS>(g[4 * j + 1], lo, s); ld16<2 * KS>(g[4 * j + 2], lo, s); ld16<3 * KS>(g[4 * j + 3], lo, s);
        }
    };
    const bool b0 = lane & 1, b1 = lane & 2;
    auto xformx = [&](int half, v4u (&g)[16]) {
#pragma unroll
        for (int j = 2 * half; j < 2 * half + 2; ++j) {
            v4u &r0 = g[4 * j], &r1 = g[4 * j + 1], &r2 = g[4 * j + 2], &r3 = g[4 * j + 3];
            landed(r0, r1, r2, r3);
#pragma unroll
            for (int e = 0; e < 4; ++e) {
                const unsigned a0 = r0[e], a1 = r1[e], a2 = r2[e], a3 = r3[e];
                const unsigned u0 = __builtin_amdgcn_mov_dpp(a0, 0xB1, 0xf, 0xf, true), u1 = __builtin_amdgcn_mov_dpp(a1, 0xB1, 0xf, 0xf, true);
                const unsigned u2 = __builtin_amdgcn_mov_dpp(a2, 0xB1, 0xf, 0xf, true), u3 = __builtin_amdgcn_mov_dpp(a3, 0xB1, 0xf, 0xf, true);
                const unsigned n0 = b0 ? u1 : a0, n1 = b0 ? a1 : u0, n2 = b0 ? u3 : a2, n3 = b0 ? a3 : u2;
                const unsigned w0 = __builtin_amdgcn_mov_dpp(n0, 0x4E, 0xf, 0xf, true), w1 = __builtin_amdgcn_mov_dpp(n1, 0x4E, 0xf, 0xf, true);
                const unsigned w2 = __builtin_amdgcn_mov_dpp(n2, 0x4E, 0xf, 0xf, true), w3 = __builtin_amdgcn_mov_dpp(n3, 0x4E, 0xf, 0xf, true);
                r0[e] = b1 ? w2 : n0; r2[e] = b1 ? n2 : w0; r1[e] = b1 ? w3 : n1; r3[e] = b1 ? n3 : w1;
            }
            landed(r0, r1, r2, r3);
        }
    };
    unsigned* ctr = (unsigned*)(myws + WS_CTR + (size_t)blockIdx.x * 64);
    unsigned prevv;
    asm volatile("global_load_dword %0, %1, %2 sc1" : "=v"(prevv) : "v"(0u), "s"(ctr) : "memory");
    if constexpr (IS_W) { loadw(0, gw[0]); loadw(1, gw[1]); loadw(2, gw[2]); } else { loadx(0, 0, gx[0]); loadx(0, 1, gx[0]); }
    asm volatile("s_waitcnt vmcnt(%1)" : "+v"(prevv) : "n"(IS_W ? 48 : 16) : "memory");
    const unsigned prev = rfl(prevv), call = prev + 1u, fbase = prev * FSTRIDE;
    lds_st(L, L_CALL, call);
    asm volatile("s_waitcnt lgkmcnt(0)" ::: "memory");
    __builtin_amdgcn_s_barrier();
    FENCE();
    const unsigned dst = (unsigned)(p.rank + NR - d) & 7u;
    unsigned char* dws = p.ws[dst];
    unsigned char* db = dws + WS_DATA + (size_t)(call & 1u) * PAR_BYTES + (size_t)p.rank * SRC_BYTES + (size_t)q * CH;
    unsigned* dflag = (unsigned*)(dws + WS_FLAG + q * 128 + p.rank * 4);
    const unsigned fmask = ((d & 1) ? 0x05555555u : 0x0AAAAAAAu) | 0x08000000u | ((IS_W || self) ? 0xA0000000u : 0xF0000000u);
    if constexpr (IS_W) { wait_vm<32>(); xformw(gw[0], ow[0]); } else { wait_vm<0>(); xformx(0, gx[0]); xformx(1, gx[0]); }
    int ops = IS_W ? 32 : 0, h1 = 0, h2 = 0, h3 = 0, h4 = 0, h5 = 0, h6 = 0;
    int l1 = IS_W ? 16 : 0, l2 = IS_W ? 32 : 0, l3 = 0;
    int ia = 0, ib = 0;
    unsigned fa = 0, fb = 0;
    unsigned tt = 0, period = self ? 0u : per;
    bool wide = self;
    auto flag_try = [&](int ret) {
        if (ia != 0 && ret >= ia) { st4_sys_lane0(dflag, fa); ++ops; ia = ib; fa = fb; ib = 0; }
    };
    auto flag_put = [&](unsigned v) {
        if (ia == 0) { ia = h1; fa = v; } else { ib = h1; fb = v; }
    };
    auto slot = [&](unsigned char* a, v4u& v) {
        const int n0 = wide ? h4 + 0 : h2 + 0, n1 = wide ? h6 + 0 : h3 + 0, n2 = wide ? h6 + 0 : h4 + 0;
        const unsigned due = t0 + (tt >> 3);
        for (;;) {
            const int late = (int)((unsigned)rt() - due);
            const int ret = ops - vm_count();
            flag_try(ret);
            if (late >= 0 && ret >= (late >= LATE2 ? n2 : late >= LATE1 ? n1 : n0)) break;
            __builtin_amdgcn_s_sleep(1);
        }
        st16_sys(a, so, v);
        h6 = h5; h5 = h4; h4 = h3; h3 = h2; h2 = h1; h1 = ++ops;
        tt += period;
    };
    auto piece = [&](int i, v4u& s0, v4u& s1, v4u& s2, v4u& s3, auto&& loads, auto&& xform) {
        if (i == NBULK + 1 && !self) {
            tt += (IS_W ? 0u : 6u * per) - (unsigned)((q & 15) + (q >> 4)) * (per / 32);
            period = per / 2; wide = true;
        }
        const unsigned fv = fbase + ((IS_W || self || i < NBULK) ? i + 1 : 2 * i - (NBULK - 2));
        slot(db, s0);
        slot(db + 1024, s1);
        if (XMID && !IS_W && !self && i > NBULK) flag_put(fv - 1u);
        loads();
        slot(db + 2048, s2);
        xform();
        slot(db + 3072, s3);
        if ((fmask >> i) & 1u) flag_put(fv);
        db += ROW_BYTES;
    };
    auto arrived = [&](int l) { while (ops - vm_count() < l) __builtin_amdgcn_s_sleep(1); };
    STAMP(12);
    if constexpr (IS_W) {
        auto wpiece = [&](int i, v4u (&oc)[4], v4u (&on)[4], unsigned (&gl)[16], unsigned (&gt)[16]) {
            piece(i, oc[0], oc[1], oc[2], oc[3],
                  [&]() { if (i + 3 < NCH) { loadw(i + 3, gl); ops += 16; } l3 = ops; },
                  [&]() { arrived(l1); xformw(gt, on); l1 = l2; l2 = l3; });
        };
#pragma unroll 1
        for (int i = 0; i < NCH; i += 4) {
            wpiece(i, ow[0], ow[1], gw[3], gw[1]);
            wpiece(i + 1, ow[1], ow[0], gw[0], gw[2]);
            wpiece(i + 2, ow[0], ow[1], gw[1], gw[3]);
            wpiece(i + 3, ow[1], ow[0], gw[2], gw[0]);
        }
    } else {
        auto group = [&](int g, v4u (&c)[16], v4u (&n)[16]) {
            const bool more = g + 1 < NCH / 4;
            piece(4 * g, c[0], c[4], c[8], c[12], [&]() { if (more) { loadx(g + 1, 0, n); ops += 8; l1 = ops; } }, [&]() {});
            piece(4 * g + 1, c[1], c[5], c[9], c[13], [&]() { if (more) { loadx(g + 1, 1, n); ops += 8; l2 = ops; } }, [&]() {});
            piece(4 * g + 2, c[2], c[6], c[10], c[14], [&]() {}, [&]() { arrived(l1); xformx(0, n); });
            piece(4 * g + 3, c[3], c[7], c[11], c[15], [&]() {}, [&]() { arrived(l2); xformx(1, n); });
        };
#pragma unroll 1
        for (int g = 0; g < NCH / 4; g += 2) {
            group(g, gx[0], gx[1]);
            const unsigned last = rfl(g == NCH / 4 - 2 && !self ? ~0u : 0u);
#pragma unroll
            for (int a = 0; a < 4; ++a)
#pragma unroll
                for (int c = a + 1; c < 4; ++c) swap16(gx[1][4 * a + c], gx[1][4 * c + a], last);
            group(g + 1, gx[1], gx[0]);
        }
    }
    while (ia != 0) flag_try(ops - vm_count());
    if (lane == 0) __hip_atomic_store(ctr, call, __ATOMIC_RELAXED, __HIP_MEMORY_SCOPE_AGENT);
    STAMP(9);
}

__global__ void __launch_bounds__(NTHREADS) fused_gemm_ag(Params p) {
    __shared__ __attribute__((aligned(1024))) unsigned char lds_raw[LDS_BYTES];
    LAS unsigned char* L = (LAS unsigned char*)lds_raw;
    const int tid = threadIdx.x, lane = tid & 63;
    const int wave = rfl(tid >> 6);
    const int b = blockIdx.x, xcd = b & 7, idx = b >> 3;
    const int ti = (xcd >> 1) * 4 + (idx >> 3), tj = (xcd & 1) * 8 + (idx & 7);
    unsigned char* myws = p.ws[p.rank & 7];
    const unsigned tstart = (unsigned)rt();

    if (wave < 8) {
        if (wave == 0) STAMP(0);
        lds_st(L, L_CTL + wave * 256 + lane * 4, 0u);
        float scale = p.sx[0] * p.sw[0];
        asm volatile("s_waitcnt lgkmcnt(0)" ::: "memory");
        __builtin_amdgcn_s_barrier();
        FENCE();
        scale = __builtin_bit_cast(float, rfl(__builtin_bit_cast(unsigned, scale)));
        const int wr = wave >> 1, wc = wave & 1, i = lane & 31, h = lane >> 5;
        unsigned char* otile = (unsigned char*)(p.out + (size_t)(ti * 256 + wr * 64) * NN + tj * 256 + wc * 128);
        asm volatile("" : "+s"(scale), "+s"(otile));
        const int abase = h * 8192 + (64 * wr + i) * 16;
        const int bbase = 16384 + h * 8192 + (32 * wc + i) * 16;
        v16f acc[2][4];
#pragma unroll
        for (int m = 0; m < 2; ++m)
#pragma unroll
            for (int c = 0; c < 4; ++c)
#pragma unroll
                for (int e = 0; e < 16; ++e) acc[m][c][e] = 0.f;
        unsigned spin = 0;
        for (int s = 0; s < NSB; ++s) {
            for (;;) {
                v2u f = *(volatile LAS v2u*)(L + L_FULL);
                unsigned mn = rfl(f.x < f.y ? f.x : f.y);
                if (mn >= (unsigned)(4 * s + 4)) break;
                if (expired(spin, tstart)) { lds_st(L, L_BAD, 1u); break; }
                __builtin_amdgcn_s_sleep(1);
            }
            FENCE();
            LAS unsigned char* sl = L + (s & (NSLOT - 1)) * SLOT;
            v8i a[2], bf[4];
#pragma unroll
            for (int m = 0; m < 2; ++m) {
                v4i lo = *(LAS v4i*)(sl + abase + m * 512), hi = *(LAS v4i*)(sl + abase + m * 512 + 4096);
                a[m] = __builtin_shufflevector(lo, hi, 0, 1, 2, 3, 4, 5, 6, 7);
            }
#pragma unroll
            for (int c = 0; c < 4; ++c) {
                v4i lo = *(LAS v4i*)(sl + bbase + c * 1024), hi = *(LAS v4i*)(sl + bbase + c * 1024 + 4096);
                bf[c] = __builtin_shufflevector(lo, hi, 0, 1, 2, 3, 4, 5, 6, 7);
                if (c == 3) {
                    asm volatile("s_waitcnt lgkmcnt(0)" : "+v"(bf[3]) :: "memory");
                    lds_st(L, L_PROG + wave * 4, (unsigned)(s + 1 < NSB ? s + 1 : NSB + 7 * wr));
                    FENCE();
                }
#pragma unroll
                for (int m = 0; m < 2; ++m)
                    acc[m][c] = __builtin_amdgcn_mfma_scale_f32_32x32x64_f8f6f4(a[m], bf[c], acc[m][c], 0, 0, 0, 0, 0, 0);
            }
        }
        if (wave == 0) STAMP(1);
        unsigned va;
        {
            const unsigned lb = (unsigned)(uintptr_t)L;
            unsigned vb = lb + (32 * wc + i) * 16, t0, t1;
            unsigned m = 7 * wr, blk = (7 * wr + NXB - 1) % NXB, cnt, tmp, rel;
            va = lb + L_XR + blk * CH + h * 2048 + i * 16;
            u64 now;
#define POLL_ADDR \
            "v_bfe_u32 %[t0], %[va], 11, 1\n\ts_lshl3_add_u32 %[tmp], %[m], %[fin]\n\t" \
            "v_lshl_add_u32 %[t0], %[t0], 2, %[tmp]\n\t"
#define MOVE \
            "s_add_u32 %[blk], %[blk], 1\n\ts_cmp_eq_u32 %[blk], %[nxb]\n\ts_cselect_b32 %[blk], 0, %[blk]\n\t" \
            "s_cselect_b32 %[tmp], %[back], %[fwd]\n\t" \
            "v_and_b32 %[t0], 0xf000, %[t0]\n\tv_and_b32 %[vb], 0x1fff, %[vb]\n\tv_add_u32 %[va], %[tmp], %[va]\n\t" \
            "v_lshl_add_u32 %[vb], %[t0], 1, %[vb]\n\t"
#define RD(r0, r1, adr, off, hi) \
            "ds_read_b128 v[" #r0 "], %[" #adr "] offset:" #off "\n\tds_read_b128 v[" #r1 "], %[" #adr "] offset:" #off "+" #hi "\n\t"
#define RD_A0 RD(136:139, 140:143, va, 0, 1024)
#define RD_A1 RD(144:147, 148:151, va, 512, 1024)
#define RD_B0(off) RD(152:155, 156:159, vb, off, 4096)
#define RD_B1(off) RD(160:163, 164:167, vb, off, 4096)
#define MM_(c, a, b) "v_mfma_f32_32x32x64_f8f6f4 %[" #c "], v[" #a "], v[" #b "], %[" #c "]\n\t"
            asm volatile(
                "s_mov_b32 %[cnt], 7\n"
                "1:\n\t"
                POLL_ADDR
                "ds_read_b32 %[t0], %[t0]\n\ts_waitcnt lgkmcnt(0)\n\t"
                "v_cmp_le_u32 vcc, 0x01010000, %[t0]\n\ts_cmp_eq_u64 vcc, -1\n\ts_cbranch_scc1 3f\n\t"
                "s_memrealtime %[now]\n\ts_waitcnt lgkmcnt(0)\n\t"
                "s_mov_b64 vcc, %[now]\n\ts_sub_u32 %[tmp], vcc_lo, %[ts]\n\t"
                "s_cmp_gt_u32 %[tmp], %[dl]\n\ts_cbranch_scc0 1b\n\t"
                "v_mov_b32 %[t0], %[bad]\n\tv_mov_b32 %[t1], 1\n\tds_write_b32 %[t0], %[t1]\n\t"
                "v_mov_b32 %[t0], %[prog]\n\tv_mov_b32 %[t1], 1000\n\tds_write_b32 %[t0], %[t1]\n\t"
                "s_branch 9f\n"
                "3:\n\t"
                MOVE
                RD_B0(0) RD_A0 RD_A1
                "\n4:\n\t"
                RD_B1(1024)
                "s_add_u32 %[m], %[m], 1\n\ts_add_u32 %[rel], %[m], %[nsb]\n\ts_sub_u32 %[cnt], %[cnt], 1\n\t"
                "s_cmp_eq_u32 %[cnt], 0\n\ts_cselect_b32 %[rel], 1000, %[rel]\n\t"
                "v_mov_b32 %[t0], %[prog]\n\tv_mov_b32 %[t1], %[rel]\n\t"
                "s_waitcnt lgkmcnt(4)\n\t"
                MM_(c00, 136:143, 152:159)
                "s_waitcnt lgkmcnt(2)\n\t"
                MM_(c10, 144:151, 152:159)
                "ds_write_b32 %[t0], %[t1]\n\t"
                RD_B0(2048)
                POLL_ADDR
                "s_waitcnt lgkmcnt(3)\n\t"
                MM_(c01, 136:143, 160:167)
                "ds_read_b32 %[t0], %[t0]\n\t"
                MM_(c11, 144:151, 160:167)
                RD_B1(3072)
                "s_waitcnt lgkmcnt(3)\n\t"
                MM_(c02, 136:143, 152:159)
                "s_waitcnt lgkmcnt(2)\n\t"
                "v_cmp_le_u32 vcc, 0x01010000, %[t0]\n\t"
                MM_(c12, 144:151, 152:159)
                "s_cmp_eq_u32 %[cnt], 0\n\ts_cbranch_scc1 6f\n\t"
                "s_cmp_eq_u64 vcc, -1\n\ts_cbranch_scc0 6f\n\t"
                MOVE
                RD_B0(0)
                "s_waitcnt lgkmcnt(2)\n\t"
                MM_(c03, 136:143, 160:167)
                RD_A0
                MM_(c13, 144:151, 160:167)
                RD_A1
                "s_branch 4b\n"
                "6:\n\t"
                "s_waitcnt lgkmcnt(0)\n\t"
                MM_(c03, 136:143, 160:167)
                MM_(c13, 144:151, 160:167)
                "s_cmp_eq_u32 %[cnt], 0\n\ts_cbranch_scc0 1b\n"
                "9:\n\t"
                "s_waitcnt lgkmcnt(0)\n\ts_nop 15\n\ts_nop 15"
                : [c00] "+v"(acc[0][0]), [c01] "+v"(acc[0][1]), [c02] "+v"(acc[0][2]), [c03] "+v"(acc[0][3]),
                  [c10] "+v"(acc[1][0]), [c11] "+v"(acc[1][1]), [c12] "+v"(acc[1][2]), [c13] "+v"(acc[1][3]),
                  [va] "+v"(va), [vb] "+v"(vb), [t0] "=&v"(t0), [t1] "=&v"(t1),
                  [m] "+s"(m), [blk] "+s"(blk), [cnt] "=&s"(cnt), [tmp] "=&s"(tmp), [rel] "=&s"(rel), [now] "=&s"(now)
                : [fin] "s"(lb + L_FIN), [prog] "s"(lb + L_PROG + wave * 4), [bad] "s"(lb + L_BAD), [ts] "s"(tstart),
                  [dl] "s"(DEADLINE2), [nsb] "n"(NSB), [nxb] "n"(NXB), [fwd] "s"(CH), [back] "n"(-(NXB - 1) * CH)
                : "memory", "vcc", "scc",
                  "v136", "v137", "v138", "v139", "v140", "v141", "v142", "v143", "v144", "v145", "v146", "v147",
                  "v148", "v149", "v150", "v151", "v152", "v153", "v154", "v155", "v156", "v157", "v158", "v159",
                  "v160", "v161", "v162", "v163", "v164", "v165", "v166", "v167");
        }
        if (wc == 0) STAMP(20 + wr);
        const bool bad = lds_ld(L, L_BAD) != 0u;
        const float mul = bad ? __builtin_nanf("") : scale;
        const unsigned oo = (va & 0x800u) * 32u + (va & 0x1f0u);
#pragma unroll
        for (int m = 0; m < 2; ++m)
#pragma unroll
            for (int e = 0; e < 16; ++e) {
                v4f v = {acc[m][0][e] * mul, acc[m][1][e] * mul, acc[m][2][e] * mul, acc[m][3][e] * mul};
                st16_sys(otile + (size_t)(m * 32 + (e & 3) + 8 * (e >> 2)) * NN * 4, oo, __builtin_bit_cast(v4u, v));
                if (OWIN < 32 && wr < 3) wait_vm<OWIN>();
            }
        return;
    }

    __builtin_amdgcn_s_setprio(3);
    if (wave == 10) {
        const int q = (idx & 4) * 4 + (xcd & 3) * 4 + (idx & 3);
        const int d = (p.dtab >> (4 * ((xcd & 4) + (idx >> 3)))) & 7;
        const int ord = (q & 15) * 2 + (q >> 4);
        const unsigned per = xcd < 4 ? PER8 : PERX8;
        const unsigned t0 = tstart + (unsigned)LEAD + ord * per / 256;
        if (q < 16) send_panel<false>(p, L, myws, q, d, lane, t0, per);
        else send_panel<true>(p, L, myws, q, d, lane, t0, per);
        return;
    }

    if (wave == 11) {
        asm volatile("buffer_inv sc0 sc1" ::: "memory");
        __builtin_amdgcn_s_barrier();
        FENCE();
        const unsigned call = rfl(lds_ld(L, L_CALL));
        const unsigned fbase = (call - 1u) * FSTRIDE;
        wait_vm<0>();
        const int d = lane & 7;
        const unsigned srcr = (unsigned)(p.rank + d) & 7u;
        const unsigned* addr = lane < 8 ? (const unsigned*)(myws + WS_FLAG + ti * 128 + srcr * 4)
                             : lane < 16 ? (const unsigned*)(myws + WS_FLAG + (16 + tj) * 128 + srcr * 4)
                             : (const unsigned*)(myws + WS_TMO);
        const unsigned row0 = ((call & 1u) * NR + srcr) * NCH;
        const int nmax = lane >= 8 ? 0 : d == 0 ? NCH : NBULK, amax = lane >= 8 || d == 0 ? 0 : 8;
        int seen = 0, qn = 0;
        int seen2 = 0, fq = 0, glim = 2;
        for (;;) {
            unsigned v = __hip_atomic_load(addr, __ATOMIC_RELAXED, __HIP_MEMORY_SCOPE_SYSTEM);
            const unsigned tmo = __builtin_amdgcn_readlane(v, 16);
            bool dead = tmo == call;
            if (!dead && (unsigned)rt() - tstart > DEADLINE) {
                if (lane == 0) __hip_atomic_store((unsigned*)(myws + WS_TMO), call, __ATOMIC_RELAXED, __HIP_MEMORY_SCOPE_AGENT);
                dead = true;
            }
            int nx = (int)(v - fbase), nw = dpp0<0x128>(nx);
            if (dead) { nx = nw = 64; lds_st(L, L_BAD, 1u); }
            int nn = min(min(nx, nw), nmax) - seen;
            nn = max(nn, 0);
            if (__ballot(nn > 0)) {
                int pre = nn;
                pre += dpp0<0x111>(pre); pre += dpp0<0x112>(pre); pre += dpp0<0x114>(pre);
                const int pos = qn + pre - nn;
                FENCE();
                for (int j = 0; j < nn; ++j) lds_st(L, L_QUEUE + (pos + j) * 4, row0 + seen + j + 1u);
                seen += nn;
                qn += __builtin_amdgcn_readlane(pre, 7);
            }
            int a = min(nx - NBULK, amax);
            if (nw < NCH) a = min(a, (nw - NBULK) >> 1);
            for (;;) {
                const int n2 = max(min(a, glim) - seen2, 0);
                if (!__ballot(n2 > 0)) break;
                int pre = n2;
                pre += dpp0<0x111>(pre); pre += dpp0<0x112>(pre); pre += dpp0<0x114>(pre);
                const int pos = fq + pre - n2;
                FENCE();
                for (int j = 0; j < n2; ++j) {
                    const unsigned pr = (seen2 + j) & 1;
                    *(volatile LAS unsigned short*)(L + L_FIN + (pos + j) * 4) =
                        (unsigned short)((2 * d - 2 + pr) << 12 | (row0 + NBULK + ((seen2 + j) >> 1)));
                }
                seen2 += n2;
                fq += __builtin_amdgcn_readlane(pre, 7);
                if (fq != glim * (NR - 1)) break;
                STAMP(31 + glim / 2);
                glim += 2;
            }
            if (qn >= NQ && fq >= NFIN) break;
            __builtin_amdgcn_s_sleep(POLLSLEEP);
        }
        STAMP(10);
        return;
    }

    {
        const int ld = wave - 8;
        __builtin_amdgcn_s_barrier();
        FENCE();
        const unsigned voff = lane * 16;
        const unsigned char* dbase = myws + WS_DATA + (size_t)(ld ? 16 + tj : ti) * CH;
        const unsigned lb = (unsigned)(uintptr_t)L;
        int iss = 0, done = 0;
        unsigned spin = 0;
        while (done < NQ) {
            const unsigned v = lds_ld(L, lane < 16 ? L_PROG + (lane & 7) * 4 : L_QUEUE + (iss + (lane & 15)) * 4);
            int lim = iss + __builtin_ctzll(~(__ballot(v != 0u) >> 16));
            lim = min(min(lim, iss + 16), min(NQ, done + 15));
            const int st = iss >> 2;
            int fr = st + NSLOT;
#pragma unroll
            for (int j = NSLOT - 1; j >= 0; --j) if (__ballot(lane < 8 && (int)v < st + j - (NSLOT - 1))) fr = st + j;
            lim = min(lim, fr * 4);
            FENCE();
            const bool any = iss < lim;
            for (const int i0 = iss; iss < lim; ++iss) {
                const unsigned e = __builtin_amdgcn_readlane(v, 16 + iss - i0) - 1u;
                dma4k(dbase + (size_t)e * ROW_BYTES, lb + ld * 16384 + ((iss >> 2) & (NSLOT - 1)) * SLOT + (iss & 3) * CH, voff);
            }
            const int tg = iss - ((vm_count() + 3) >> 2);
            if (tg > done) {
                FENCE();
                done = tg;
                lds_st(L, L_FULL + ld * 4, (unsigned)done);
            } else if (!any) {
                if (expired(spin, tstart)) { lds_st(L, L_BAD, 1u); break; }
                __builtin_amdgcn_s_sleep(1);
            }
        }
        auto fin = [&](auto isw) {
            constexpr bool W = decltype(isw)::value;
            constexpr int NF = W ? NPAIR : NFIN;
            unsigned xa = lb + L_XR;
            iss = 0; done = 0;
            while (done < NF) {
                const unsigned v = lds_ld(L, lane < 16 ? L_PROG + (lane & 7) * 4 : L_FIN + (iss + (lane & 15)) * 4);
                int n = __builtin_ctz(~(unsigned)(__ballot((v & 0xfffu) != 0u) >> 16) | (1u << (W ? WBATCH : XBATCH)));
                int pm = (int)v;
                pm = min(pm, dpp0<0x121>(pm)); pm = min(pm, dpp0<0x122>(pm)); pm = min(pm, dpp0<0x124>(pm));
                pm = (int)rfl(pm) - NSB;
                int lim = W ? done + 7 : 2 * (pm + NXB);
                lim = lim < NF ? lim : NF;
                lim = pm < 0 ? 0 : lim - iss;
                n = n < lim ? n : lim;
                const unsigned off = (v & 0x1ffu) << 17 | (W ? (v & 0x1000u) << 6 : (v & 0x1000u) >> 1);
                FENCE();
                for (int j = 0; j < n; ++j) {
                    const unsigned vo = voff + __builtin_amdgcn_readlane(off, 16 + j);
                    if constexpr (W) {
                        const unsigned la = lb + (__builtin_amdgcn_readlane(v, 16 + j) & 0xf000u) * 2;
                        dma4k(dbase, la, vo);
                        dma4k(dbase + ROW_BYTES, la + CH, vo);
                    } else {
                        dma2k(dbase, xa, vo);
                        xa = xa == lb + L_CTL - 2048 ? lb + L_XR : xa + 2048;
                    }
                }
                if (n > 0) iss += n;
                const int vm = vm_count();
                const int tg = iss - (W ? (vm + 7) >> 3 : (vm + 1) >> 1);
                if (tg > done) {
                    FENCE();
                    const int last = tg == NF ? NFIN : tg;
                    if (done + lane < last) *(volatile LAS unsigned char*)(L + L_FIN + (done + lane) * 4 + 2 + W) = 1;
                    done = tg;
                } else if (n <= 0) {
                    if (expired(spin, tstart)) { lds_st(L, L_BAD, 1u); break; }
                    __builtin_amdgcn_s_sleep(1);
                }
            }
        };
        if (ld) fin(std::true_type{}); else fin(std::false_type{});
        STAMP(11 + 2 * ld);
    }
}

}

extern "C" void kernel_launch(int rank, int nranks, ncclComm_t comm, hipStream_t stream,
                              void* const* d_in, const int* in_sizes, int n_in,
                              void* d_out, int out_size, const int* in_dtypes, int out_dtype,
                              void* const* d_out_all, void* const* d_ws_all, size_t ws_bytes) {
    if (nranks != NR || n_in != 4 || ws_bytes < WS_NEED || in_sizes[0] != MM * KS || in_sizes[1] != KS * NN ||
        out_size != MM * NN || (unsigned)rank >= (unsigned)NR) {
        fprintf(stderr, "kernel_launch: unexpected problem geometry\n");
        return;
    }
    Params p;
    p.x = (const unsigned char*)d_in[0]; p.w = (const unsigned char*)d_in[1];
    p.sx = (const float*)d_in[2]; p.sw = (const float*)d_in[3];
    p.out = (float*)d_out; p.rank = rank;
    for (int r = 0; r < NR; ++r) p.ws[r] = (unsigned char*)d_ws_all[r];
    {
        int dev = -1, cnt = -1; char bus[64] = {0}; char line[1024]; int n = 0;
        (void)hipGetDevice(&dev); (void)hipGetDeviceCount(&cnt); (void)hipDeviceGetPCIBusId(bus, 64, dev);
        n += snprintf(line + n, sizeof(line) - n, "DIAG rank %d dev %d cnt %d bus %s peers:", rank, dev, cnt, bus);
        for (int r = 0; r < NR; ++r) {
            hipPointerAttribute_t at; at.device = -7;
            hipError_t e = hipPointerGetAttributes(&at, d_ws_all[r]);
            n += snprintf(line + n, sizeof(line) - n, " %d(e%d,%p)", at.device, (int)e, d_ws_all[r]);
        }
        (void)hipGetLastError();
        for (int dd = 0; dd < cnt && dd < 8; ++dd) { (void)hipDeviceGetPCIBusId(bus, 64, dd); n += snprintf(line + n, sizeof(line) - n, " %s", bus + 5); }
        fprintf(stderr, "%s\n", line);
    }
    p.dtab = 0;
    for (int r = 0, lo = 0, hi = 4; r < NR; ++r) {
        const int dst = (rank + NR - 1 - r) & 7;
        const bool same = ((0xB2 >> dst) & 1) == ((0xB2 >> rank) & 1);
        p.dtab |= (unsigned)((rank - dst) & 7) << (4 * (same ? lo++ : hi++));
    }
    hipLaunchKernelGGL(fused_gemm_ag, dim3(NWG), dim3(NTHREADS), 0, stream, p);
    HIP_CHECK(hipGetLastError());
}
```

```cpp
#include <hip/hip_runtime.h>
#include <rccl/rccl.h>
#include <cstdio>
#include <cstdint>
#include <type_traits>

#define HIP_CHECK(expr) do { hipError_t _e = (expr); if (_e != hipSuccess) { \
    fprintf(stderr, "HIP error %s at %s:%d\n", hipGetErrorString(_e), __FILE__, __LINE__); \
    return; } } while (0)

namespace {

#define LAS __attribute__((address_space(3)))
typedef int v4i __attribute__((ext_vector_type(4)));
typedef int v8i __attribute__((ext_vector_type(8)));
typedef unsigned v2u __attribute__((ext_vector_type(2)));
typedef unsigned v4u __attribute__((ext_vector_type(4)));
typedef float v4f __attribute__((ext_vector_type(4)));
typedef float v16f __attribute__((ext_vector_type(16)));
typedef unsigned long long u64;

constexpr int NR = 8;
constexpr int MM = 4096, NN = 4096, KS = 512;
constexpr int NCH = 32;
constexpr int NBULK = 28;
constexpr int NPANEL = 32;
constexpr int CH = 4096;
constexpr int NWG = 256, NTHREADS = 768;
constexpr int NQ = (NR - 1) * NBULK + NCH;
constexpr int NSB = NQ / 4;
constexpr int NPAIR = 2 * (NR - 1);
constexpr int NFIN = 4 * NPAIR;
constexpr unsigned FSTRIDE = 64;
constexpr size_t WS_CTR = 0;
constexpr size_t WS_FLAG = 16384;
constexpr size_t WS_TMO = 16384 + 4096;
constexpr size_t WS_DBG = 32768;
constexpr size_t WS_DATA = 1u << 20;
constexpr size_t ROW_BYTES = (size_t)NPANEL * CH;
constexpr size_t SRC_BYTES = (size_t)NCH * ROW_BYTES;
constexpr size_t PAR_BYTES = (size_t)NR * SRC_BYTES;
constexpr size_t WS_NEED = WS_DATA + 2 * PAR_BYTES;
constexpr unsigned DEADLINE = 40000000u;
constexpr unsigned DEADLINE2 = 60000000u;
constexpr int NSLOT = 4, SLOT = 32768;
constexpr int L_XR = NPAIR * 2 * CH;
constexpr int NXB = 11;
constexpr int L_CTL = L_XR + NXB * CH;
constexpr int L_FULL = L_CTL;
constexpr int L_PROG = L_CTL + 64;
constexpr int L_BAD = L_CTL + 128;
constexpr int L_QUEUE = L_CTL + 256;
constexpr int L_FIN = L_CTL + 1536;
constexpr int L_CALL = L_CTL + 2048;
constexpr int LDS_BYTES = L_CALL + 64 + 256;
static_assert(L_QUEUE + (NQ + 32) * 4 <= L_FIN && L_FIN + (NFIN + 32) * 4 <= L_CALL && L_CTL >= NSLOT * SLOT, "LDS map");

#define PER8 448
#define PERX8 456
#define LEAD 200
#define POLLSLEEP 2
#define LATE1 100000
#define LATE2 100000
#define XBATCH 8
#define WBATCH 4
#define OWIN 63
#define XMID 1

struct Params {
    const unsigned char* x; const unsigned char* w; const float* sx; const float* sw;
    float* out; unsigned char* ws[NR]; int rank;
    unsigned dtab;
};

__device__ __forceinline__ u64 rt() { return __builtin_amdgcn_s_memrealtime(); }
__device__ __forceinline__ unsigned rfl(unsigned v) { return __builtin_amdgcn_readfirstlane(v); }
__device__ __forceinline__ unsigned lds_ld(LAS unsigned char* L, int off) { return *(volatile LAS unsigned*)(L + off); }
__device__ __forceinline__ void lds_st(LAS unsigned char* L, int off, unsigned v) { *(volatile LAS unsigned*)(L + off) = v; }
#define FENCE() asm volatile("" ::: "memory")
template <int N> __device__ __forceinline__ void wait_vm() { asm volatile("s_waitcnt vmcnt(%0)" :: "n"(N) : "memory"); }
__device__ __forceinline__ int vm_count() {
    unsigned r;
    asm volatile("s_getreg_b32 %0, hwreg(HW_REG_IB_STS)" : "=s"(r) :: "memory");
    return (int)((r & 15u) | ((r >> 18) & 48u));
}
__device__ __forceinline__ bool expired(unsigned& spin, unsigned t0) {
    return (++spin & 4095u) == 0u && (unsigned)rt() - t0 > DEADLINE2;
}
template <int CTRL> __device__ __forceinline__ int dpp0(int v) { return __builtin_amdgcn_update_dpp(0, v, CTRL, 0xf, 0xf, true); }

#define STAMPL(i) do {} while (0)
#define STAMP(i) do {} while (0)

__device__ __forceinline__ void dma4k(const unsigned char* g, unsigned ldsaddr, unsigned voff) {
    asm volatile("s_mov_b32 m0, %2\n\ts_nop 0\n\t"
                 "global_load_lds_dwordx4 %0, %1 offset:0\n\tglobal_load_lds_dwordx4 %0, %1 offset:1024\n\t"
                 "global_load_lds_dwordx4 %0, %1 offset:2048\n\tglobal_load_lds_dwordx4 %0, %1 offset:3072"
                 :: "v"(voff), "s"(g), "s"(ldsaddr) : "memory");
}
__device__ __forceinline__ void dma2k(const unsigned char* g, unsigned ldsaddr, unsigned voff) {
    asm volatile("s_mov_b32 m0, %2\n\ts_nop 0\n\t"
                 "global_load_lds_dwordx4 %0, %1 offset:0\n\tglobal_load_lds_dwordx4 %0, %1 offset:1024"
                 :: "v"(voff), "s"(g), "s"(ldsaddr) : "memory");
}
__device__ __forceinline__ void st16_sys(unsigned char* sbase, unsigned voff, v4u v) {
    asm volatile("global_store_dwordx4 %0, %1, %2 sc0 sc1\n\ts_nop 1" :: "v"(voff), "v"(v), "s"(sbase) : "memory");
}
__device__ __forceinline__ void st4_sys_lane0(unsigned* p, unsigned v) {
    u64 ex;
    asm volatile("s_mov_b64 %0, exec\n\ts_mov_b64 exec, 1\n\tglobal_store_dword %1, %2, %3 sc0 sc1\n\ts_mov_b64 exec, %0"
                 : "=&s"(ex) : "v"(0u), "v"(v), "s"(p) : "memory");
}
__device__ __forceinline__ void ld4(unsigned& d, unsigned voff, const unsigned char* sbase) {
    asm volatile("global_load_dword %0, %1, %2" : "=v"(d) : "v"(voff), "s"(sbase) : "memory");
}
template <int OFF> __device__ __forceinline__ void ld16(v4u& d, unsigned voff, const unsigned char* sbase) {
    asm volatile("global_load_dwordx4 %0, %1, %2 offset:%3" : "=v"(d) : "v"(voff), "s"(sbase), "n"(OFF) : "memory");
}
__device__ __forceinline__ void landed(unsigned (&g)[16]) {
    asm volatile("" : "+v"(g[0]), "+v"(g[1]), "+v"(g[2]), "+v"(g[3]), "+v"(g[4]), "+v"(g[5]), "+v"(g[6]), "+v"(g[7]),
                      "+v"(g[8]), "+v"(g[9]), "+v"(g[10]), "+v"(g[11]), "+v"(g[12]), "+v"(g[13]), "+v"(g[14]), "+v"(g[15]) :: "memory");
}
__device__ __forceinline__ void landed(v4u& a, v4u& b, v4u& c, v4u& d) {
    asm volatile("" : "+v"(a), "+v"(b), "+v"(c), "+v"(d) :: "memory");
}
__device__ __forceinline__ void swap16(v4u& a, v4u& b, unsigned mask) {
    unsigned x0 = a[0], x1 = a[1], x2 = a[2], x3 = a[3], y0 = b[0], y1 = b[1], y2 = b[2], y3 = b[3];
    asm volatile("s_mov_b32 exec_lo, %8\n\ts_mov_b32 exec_hi, %8\n\tv_swap_b32 %0, %4\n\tv_swap_b32 %1, %5\n\tv_swap_b32 %2, %6\n\tv_swap_b32 %3, %7\n\t"
                 "s_mov_b64 exec, -1"
                 : "+v"(x0), "+v"(x1), "+v"(x2), "+v"(x3), "+v"(y0), "+v"(y1), "+v"(y2), "+v"(y3) : "s"(mask));
    a[0] = x0; a[1] = x1; a[2] = x2; a[3] = x3; b[0] = y0; b[1] = y1; b[2] = y2; b[3] = y3;
}

template <bool IS_W>
__device__ __forceinline__ void send_panel(const Params& p, LAS unsigned char* L, unsigned char* myws, int q, int d, int lane, unsigned t0, unsigned per) {
    const bool self = d == 0;
    const unsigned char* src = IS_W ? p.w + (size_t)(q - 16) * 256 : p.x + (size_t)q * 256 * KS;
    const unsigned lo = IS_W ? lane * 4 : (lane >> 2) * 4 * KS + (lane & 3) * 16;
    const unsigned so = lane * 16;
    unsigned gw[4][16];
    v4u ow[2][4];
    v4u gx[2][16];
    auto loadw = [&](int i, unsigned (&g)[16]) {
#pragma unroll
        for (int k = 0; k < 16; ++k) ld4(g[k], lo, src + (size_t)(i * 16 + k) * NN);
    };
    auto xformw = [&](unsigned (&g)[16], v4u (&ob)[4]) {
        landed(g);
#pragma unroll
        for (int t = 0; t < 4; ++t) {
            const unsigned a0 = g[4 * t], a1 = g[4 * t + 1], a2 = g[4 * t + 2], a3 = g[4 * t + 3];
            const unsigned x0 = __builtin_amdgcn_perm(a1, a0, 0x05010400u), x1 = __builtin_amdgcn_perm(a1, a0, 0x07030602u);
            const unsigned y0 = __builtin_amdgcn_perm(a3, a2, 0x05010400u), y1 = __builtin_amdgcn_perm(a3, a2, 0x07030602u);
            ob[0][t] = __builtin_amdgcn_perm(y0, x0, 0x05040100u);
            ob[1][t] = __builtin_amdgcn_perm(y0, x0, 0x07060302u);
            ob[2][t] = __builtin_amdgcn_perm(y1, x1, 0x05040100u);
            ob[3][t] = __builtin_amdgcn_perm(y1, x1, 0x07060302u);
        }
        landed(ob[0], ob[1], ob[2], ob[3]);
    };
    auto loadx = [&](int grp, int half, v4u (&g)[16]) {
#pragma unroll
        for (int j = 2 * half; j < 2 * half + 2; ++j) {
            const unsigned char* s = src + (size_t)j * 64 * KS + grp * 64;
            ld16<0>(g[4 * j], lo, s); ld16<KS>(g[4 * j + 1], lo, s); ld16<2 * KS>(g[4 * j + 2], lo, s); ld16<3 * KS>(g[4 * j + 3], lo, s);
        }
    };
    const bool b0 = lane & 1, b1 = lane & 2;
    auto xformx = [&](int half, v4u (&g)[16]) {
#pragma unroll
        for (int j = 2 * half; j < 2 * half + 2; ++j) {
            v4u &r0 = g[4 * j], &r1 = g[4 * j + 1], &r2 = g[4 * j + 2], &r3 = g[4 * j + 3];
            landed(r0, r1, r2, r3);
#pragma unroll
            for (int e = 0; e < 4; ++e) {
                const unsigned a0 = r0[e], a1 = r1[e], a2 = r2[e], a3 = r3[e];
                const unsigned u0 = __builtin_amdgcn_mov_dpp(a0, 0xB1, 0xf, 0xf, true), u1 = __builtin_amdgcn_mov_dpp(a1, 0xB1, 0xf, 0xf, true);
                const unsigned u2 = __builtin_amdgcn_mov_dpp(a2, 0xB1, 0xf, 0xf, true), u3 = __builtin_amdgcn_mov_dpp(a3, 0xB1, 0xf, 0xf, true);
                const unsigned n0 = b0 ? u1 : a0, n1 = b0 ? a1 : u0, n2 = b0 ? u3 : a2, n3 = b0 ? a3 : u2;
                const unsigned w0 = __builtin_amdgcn_mov_dpp(n0, 0x4E, 0xf, 0xf, true), w1 = __builtin_amdgcn_mov_dpp(n1, 0x4E, 0xf, 0xf, true);
                const unsigned w2 = __builtin_amdgcn_mov_dpp(n2, 0x4E, 0xf, 0xf, true), w3 = __builtin_amdgcn_mov_dpp(n3, 0x4E, 0xf, 0xf, true);
                r0[e] = b1 ? w2 : n0; r2[e] = b1 ? n2 : w0; r1[e] = b1 ? w3 : n1; r3[e] = b1 ? n3 : w1;
            }
            landed(r0, r1, r2, r3);
        }
    };
    unsigned* ctr = (unsigned*)(myws + WS_CTR + (size_t)blockIdx.x * 64);
    unsigned prevv;
    asm volatile("global_load_dword %0, %1, %2 sc1" : "=v"(prevv) : "v"(0u), "s"(ctr) : "memory");
    if constexpr (IS_W) { loadw(0, gw[0]); loadw(1, gw[1]); loadw(2, gw[2]); } else { loadx(0, 0, gx[0]); loadx(0, 1, gx[0]); }
    asm volatile("s_waitcnt vmcnt(%1)" : "+v"(prevv) : "n"(IS_W ? 48 : 16) : "memory");
    const unsigned prev = rfl(prevv), call = prev + 1u, fbase = prev * FSTRIDE;
    lds_st(L, L_CALL, call);
    asm volatile("s_waitcnt lgkmcnt(0)" ::: "memory");
    __builtin_amdgcn_s_barrier();
    FENCE();
    const unsigned dst = (unsigned)(p.rank + NR - d) & 7u;
    unsigned char* dws = p.ws[dst];
    unsigned char* db = dws + WS_DATA + (size_t)(call & 1u) * PAR_BYTES + (size_t)p.rank * SRC_BYTES + (size_t)q * CH;
    unsigned* dflag = (unsigned*)(dws + WS_FLAG + q * 128 + p.rank * 4);
    const unsigned fmask = ((d & 1) ? 0x05555555u : 0x0AAAAAAAu) | 0x08000000u | ((IS_W || self) ? 0xA0000000u : 0xF0000000u);
    if constexpr (IS_W) { wait_vm<32>(); xformw(gw[0], ow[0]); } else { wait_vm<0>(); xformx(0, gx[0]); xformx(1, gx[0]); }
    int ops = IS_W ? 32 : 0, h1 = 0, h2 = 0, h3 = 0, h4 = 0, h5 = 0, h6 = 0;
    int l1 = IS_W ? 16 : 0, l2 = IS_W ? 32 : 0, l3 = 0;
    int ia = 0, ib = 0;
    unsigned fa = 0, fb = 0;
    unsigned tt = 0, period = self ? 0u : per;
    bool wide = self;
    auto flag_try = [&](int ret) {
        if (ia != 0 && ret >= ia) { st4_sys_lane0(dflag, fa); ++ops; ia = ib; fa = fb; ib = 0; }
    };
    auto flag_put = [&](unsigned v) {
        if (ia == 0) { ia = h1; fa = v; } else { ib = h1; fb = v; }
    };
    auto slot = [&](unsigned char* a, v4u& v) {
        const int n0 = wide ? h4 + 0 : h2 + 0, n1 = wide ? h6 + 0 : h3 + 0, n2 = wide ? h6 + 0 : h4 + 0;
        const unsigned due = t0 + (tt >> 3);
        for (;;) {
            const int late = (int)((unsigned)rt() - due);
            const int ret = ops - vm_count();
            flag_try(ret);
            if (late >= 0 && ret >= (late >= LATE2 ? n2 : late >= LATE1 ? n1 : n0)) break;
            __builtin_amdgcn_s_sleep(1);
        }
        st16_sys(a, so, v);
        h6 = h5; h5 = h4; h4 = h3; h3 = h2; h2 = h1; h1 = ++ops;
        tt += period;
    };
    auto piece = [&](int i, v4u& s0, v4u& s1, v4u& s2, v4u& s3, auto&& loads, auto&& xform) {
        if (i == NBULK + 1 && !self) {
            tt += (IS_W ? 0u : 6u * per) - (unsigned)((q & 15) + (q >> 4)) * (per / 32);
            period = per / 2; wide = true;
        }
        const unsigned fv = fbase + ((IS_W || self || i < NBULK) ? i + 1 : 2 * i - (NBULK - 2));
        slot(db, s0);
        slot(db + 1024, s1);
        if (XMID && !IS_W && !self && i > NBULK) flag_put(fv - 1u);
        loads();
        slot(db + 2048, s2);
        xform();
        slot(db + 3072, s3);
        if ((fmask >> i) & 1u) flag_put(fv);
        db += ROW_BYTES;
    };
    auto arrived = [&](int l) { while (ops - vm_count() < l) __builtin_amdgcn_s_sleep(1); };
    STAMP(12);
    if constexpr (IS_W) {
        auto wpiece = [&](int i, v4u (&oc)[4], v4u (&on)[4], unsigned (&gl)[16], unsigned (&gt)[16]) {
            piece(i, oc[0], oc[1], oc[2], oc[3],
                  [&]() { if (i + 3 < NCH) { loadw(i + 3, gl); ops += 16; } l3 = ops; },
                  [&]() { arrived(l1); xformw(gt, on); l1 = l2; l2 = l3; });
        };
#pragma unroll 1
        for (int i = 0; i < NCH; i += 4) {
            wpiece(i, ow[0], ow[1], gw[3], gw[1]);
            wpiece(i + 1, ow[1], ow[0], gw[0], gw[2]);
            wpiece(i + 2, ow[0], ow[1], gw[1], gw[3]);
            wpiece(i + 3, ow[1], ow[0], gw[2], gw[0]);
        }
    } else {
        auto group = [&](int g, v4u (&c)[16], v4u (&n)[16]) {
            const bool more = g + 1 < NCH / 4;
            piece(4 * g, c[0], c[4], c[8], c[12], [&]() { if (more) { loadx(g + 1, 0, n); ops += 8; l1 = ops; } }, [&]() {});
            piece(4 * g + 1, c[1], c[5], c[9], c[13], [&]() { if (more) { loadx(g + 1, 1, n); ops += 8; l2 = ops; } }, [&]() {});
            piece(4 * g + 2, c[2], c[6], c[10], c[14], [&]() {}, [&]() { arrived(l1); xformx(0, n); });
            piece(4 * g + 3, c[3], c[7], c[11], c[15], [&]() {}, [&]() { arrived(l2); xformx(1, n); });
        };
#pragma unroll 1
        for (int g = 0; g < NCH / 4; g += 2) {
            group(g, gx[0], gx[1]);
            const unsigned last = rfl(g == NCH / 4 - 2 && !self ? ~0u : 0u);
#pragma unroll
            for (int a = 0; a < 4; ++a)
#pragma unroll
                for (int c = a + 1; c < 4; ++c) swap16(gx[1][4 * a + c], gx[1][4 * c + a], last);
            group(g + 1, gx[1], gx[0]);
        }
    }
    while (ia != 0) flag_try(ops - vm_count());
    if (lane == 0) __hip_atomic_store(ctr, call, __ATOMIC_RELAXED, __HIP_MEMORY_SCOPE_AGENT);
    STAMP(9);
}

__global__ void __launch_bounds__(NTHREADS) fused_gemm_ag(Params p) {
    __shared__ __attribute__((aligned(1024))) unsigned char lds_raw[LDS_BYTES];
    LAS unsigned char* L = (LAS unsigned char*)lds_raw;
    const int tid = threadIdx.x, lane = tid & 63;
    const int wave = rfl(tid >> 6);
    const int b = blockIdx.x, xcd = b & 7, idx = b >> 3;
    const int ti = (xcd >> 1) * 4 + (idx >> 3), tj = (xcd & 1) * 8 + (idx & 7);
    unsigned char* myws = p.ws[p.rank & 7];
    const unsigned tstart = (unsigned)rt();

    if (wave < 8) {
        if (wave == 0) STAMP(0);
        lds_st(L, L_CTL + wave * 256 + lane * 4, 0u);
        float scale = p.sx[0] * p.sw[0];
        asm volatile("s_waitcnt lgkmcnt(0)" ::: "memory");
        __builtin_amdgcn_s_barrier();
        FENCE();
        scale = __builtin_bit_cast(float, rfl(__builtin_bit_cast(unsigned, scale)));
        const int wr = wave >> 1, wc = wave & 1, i = lane & 31, h = lane >> 5;
        unsigned char* otile = (unsigned char*)(p.out + (size_t)(ti * 256 + wr * 64) * NN + tj * 256 + wc * 128);
        asm volatile("" : "+s"(scale), "+s"(otile));
        const int abase = h * 8192 + (64 * wr + i) * 16;
        const int bbase = 16384 + h * 8192 + (32 * wc + i) * 16;
        v16f acc[2][4];
#pragma unroll
        for (int m = 0; m < 2; ++m)
#pragma unroll
            for (int c = 0; c < 4; ++c)
#pragma unroll
                for (int e = 0; e < 16; ++e) acc[m][c][e] = 0.f;
        unsigned spin = 0;
        for (int s = 0; s < NSB; ++s) {
            for (;;) {
                v2u f = *(volatile LAS v2u*)(L + L_FULL);
                unsigned mn = rfl(f.x < f.y ? f.x : f.y);
                if (mn >= (unsigned)(4 * s + 4)) break;
                if (expired(spin, tstart)) { lds_st(L, L_BAD, 1u); break; }
                __builtin_amdgcn_s_sleep(1);
            }
            FENCE();
            LAS unsigned char* sl = L + (s & (NSLOT - 1)) * SLOT;
            v8i a[2], bf[4];
#pragma unroll
            for (int m = 0; m < 2; ++m) {
                v4i lo = *(LAS v4i*)(sl + abase + m * 512), hi = *(LAS v4i*)(sl + abase + m * 512 + 4096);
                a[m] = __builtin_shufflevector(lo, hi, 0, 1, 2, 3, 4, 5, 6, 7);
            }
#pragma unroll
            for (int c = 0; c < 4; ++c) {
                v4i lo = *(LAS v4i*)(sl + bbase + c * 1024), hi = *(LAS v4i*)(sl + bbase + c * 1024 + 4096);
                bf[c] = __builtin_shufflevector(lo, hi, 0, 1, 2, 3, 4, 5, 6, 7);
                if (c == 3) {
                    asm volatile("s_waitcnt lgkmcnt(0)" : "+v"(bf[3]) :: "memory");
                    lds_st(L, L_PROG + wave * 4, (unsigned)(s + 1 < NSB ? s + 1 : NSB + 7 * wr));
                    FENCE();
                }
#pragma unroll
                for (int m = 0; m < 2; ++m)
                    acc[m][c] = __builtin_amdgcn_mfma_scale_f32_32x32x64_f8f6f4(a[m], bf[c], acc[m][c], 0, 0, 0, 0, 0, 0);
            }
        }
        if (wave == 0) STAMP(1);
        unsigned va;
        {
            const unsigned lb = (unsigned)(uintptr_t)L;
            unsigned vb = lb + (32 * wc + i) * 16, t0, t1;
            unsigned m = 7 * wr, blk = (7 * wr + NXB - 1) % NXB, cnt, tmp, rel;
            va = lb + L_XR + blk * CH + h * 2048 + i * 16;
            u64 now;
#define POLL_ADDR \
            "v_bfe_u32 %[t0], %[va], 11, 1\n\ts_lshl3_add_u32 %[tmp], %[m], %[fin]\n\t" \
            "v_lshl_add_u32 %[t0], %[t0], 2, %[tmp]\n\t"
#define MOVE \
            "s_add_u32 %[blk], %[blk], 1\n\ts_cmp_eq_u32 %[blk], %[nxb]\n\ts_cselect_b32 %[blk], 0, %[blk]\n\t" \
            "s_cselect_b32 %[tmp], %[back], %[fwd]\n\t" \
            "v_and_b32 %[t0], 0xf000, %[t0]\n\tv_and_b32 %[vb], 0x1fff, %[vb]\n\tv_add_u32 %[va], %[tmp], %[va]\n\t" \
            "v_lshl_add_u32 %[vb], %[t0], 1, %[vb]\n\t"
#define RD(r0, r1, adr, off, hi) \
            "ds_read_b128 v[" #r0 "], %[" #adr "] offset:" #off "\n\tds_read_b128 v[" #r1 "], %[" #adr "] offset:" #off "+" #hi "\n\t"
#define RD_A0 RD(136:139, 140:143, va, 0, 1024)
#define RD_A1 RD(144:147, 148:151, va, 512, 1024)
#define RD_B0(off) RD(152:155, 156:159, vb, off, 4096)
#define RD_B1(off) RD(160:163, 164:167, vb, off, 4096)
#define MM_(c, a, b) "v_mfma_f32_32x32x64_f8f6f4 %[" #c "], v[" #a "], v[" #b "], %[" #c "]\n\t"
            asm volatile(
                "s_mov_b32 %[cnt], 7\n"
                "1:\n\t"
                POLL_ADDR
                "ds_read_b32 %[t0], %[t0]\n\ts_waitcnt lgkmcnt(0)\n\t"
                "v_cmp_le_u32 vcc, 0x01010000, %[t0]\n\ts_cmp_eq_u64 vcc, -1\n\ts_cbranch_scc1 3f\n\t"
                "s_memrealtime %[now]\n\ts_waitcnt lgkmcnt(0)\n\t"
                "s_mov_b64 vcc, %[now]\n\ts_sub_u32 %[tmp], vcc_lo, %[ts]\n\t"
                "s_cmp_gt_u32 %[tmp], %[dl]\n\ts_cbranch_scc0 1b\n\t"
                "v_mov_b32 %[t0], %[bad]\n\tv_mov_b32 %[t1], 1\n\tds_write_b32 %[t0], %[t1]\n\t"
                "v_mov_b32 %[t0], %[prog]\n\tv_mov_b32 %[t1], 1000\n\tds_write_b32 %[t0], %[t1]\n\t"
                "s_branch 9f\n"
                "3:\n\t"
                MOVE
                RD_B0(0) RD_A0 RD_A1
                "\n4:\n\t"
                RD_B1(1024)
                "s_add_u32 %[m], %[m], 1\n\ts_add_u32 %[rel], %[m], %[nsb]\n\ts_sub_u32 %[cnt], %[cnt], 1\n\t"
                "s_cmp_eq_u32 %[cnt], 0\n\ts_cselect_b32 %[rel], 1000, %[rel]\n\t"
                "v_mov_b32 %[t0], %[prog]\n\tv_mov_b32 %[t1], %[rel]\n\t"
                "s_waitcnt lgkmcnt(4)\n\t"
                MM_(c00, 136:143, 152:159)
                "s_waitcnt lgkmcnt(2)\n\t"
                MM_(c10, 144:151, 152:159)
                "ds_write_b32 %[t0], %[t1]\n\t"
                RD_B0(2048)
                POLL_ADDR
                "s_waitcnt lgkmcnt(3)\n\t"
                MM_(c01, 136:143, 160:167)
                "ds_read_b32 %[t0], %[t0]\n\t"
                MM_(c11, 144:151, 160:167)
                RD_B1(3072)
                "s_waitcnt lgkmcnt(3)\n\t"
                MM_(c02, 136:143, 152:159)
                "s_waitcnt lgkmcnt(2)\n\t"
                "v_cmp_le_u32 vcc, 0x01010000, %[t0]\n\t"
                MM_(c12, 144:151, 152:159)
                "s_cmp_eq_u32 %[cnt], 0\n\ts_cbranch_scc1 6f\n\t"
                "s_cmp_eq_u64 vcc, -1\n\ts_cbranch_scc0 6f\n\t"
                MOVE
                RD_B0(0)
                "s_waitcnt lgkmcnt(2)\n\t"
                MM_(c03, 136:143, 160:167)
                RD_A0
                MM_(c13, 144:151, 160:167)
                RD_A1
                "s_branch 4b\n"
                "6:\n\t"
                "s_waitcnt lgkmcnt(0)\n\t"
                MM_(c03, 136:143, 160:167)
                MM_(c13, 144:151, 160:167)
                "s_cmp_eq_u32 %[cnt], 0\n\ts_cbranch_scc0 1b\n"
                "9:\n\t"
                "s_waitcnt lgkmcnt(0)\n\ts_nop 15\n\ts_nop 15"
                : [c00] "+v"(acc[0][0]), [c01] "+v"(acc[0][1]), [c02] "+v"(acc[0][2]), [c03] "+v"(acc[0][3]),
                  [c10] "+v"(acc[1][0]), [c11] "+v"(acc[1][1]), [c12] "+v"(acc[1][2]), [c13] "+v"(acc[1][3]),
                  [va] "+v"(va), [vb] "+v"(vb), [t0] "=&v"(t0), [t1] "=&v"(t1),
                  [m] "+s"(m), [blk] "+s"(blk), [cnt] "=&s"(cnt), [tmp] "=&s"(tmp), [rel] "=&s"(rel), [now] "=&s"(now)
                : [fin] "s"(lb + L_FIN), [prog] "s"(lb + L_PROG + wave * 4), [bad] "s"(lb + L_BAD), [ts] "s"(tstart),
                  [dl] "s"(DEADLINE2), [nsb] "n"(NSB), [nxb] "n"(NXB), [fwd] "s"(CH), [back] "n"(-(NXB - 1) * CH)
                : "memory", "vcc", "scc",
                  "v136", "v137", "v138", "v139", "v140", "v141", "v142", "v143", "v144", "v145", "v146", "v147",
                  "v148", "v149", "v150", "v151", "v152", "v153", "v154", "v155", "v156", "v157", "v158", "v159",
                  "v160", "v161", "v162", "v163", "v164", "v165", "v166", "v167");
        }
        if (wc == 0) STAMP(20 + wr);
        const bool bad = lds_ld(L, L_BAD) != 0u;
        const float mul = bad ? __builtin_nanf("") : scale;
        const unsigned oo = (va & 0x800u) * 32u + (va & 0x1f0u);
#pragma unroll
        for (int m = 0; m < 2; ++m)
#pragma unroll
            for (int e = 0; e < 16; ++e) {
                v4f v = {acc[m][0][e] * mul, acc[m][1][e] * mul, acc[m][2][e] * mul, acc[m][3][e] * mul};
                st16_sys(otile + (size_t)(m * 32 + (e & 3) + 8 * (e >> 2)) * NN * 4, oo, __builtin_bit_cast(v4u, v));
                if (OWIN < 32 && wr < 3) wait_vm<OWIN>();
            }
        return;
    }

    __builtin_amdgcn_s_setprio(3);
    if (wave == 10) {
        unsigned xcc;
        asm volatile("s_getreg_b32 %0, hwreg(HW_REG_XCC_ID)" : "=s"(xcc));
        const int q = (idx & 4) * 4 + (xcc & 3) * 4 + (idx & 3);
        const int d = (p.dtab >> (4 * ((xcc & 4) + (idx >> 3)))) & 7;
        const int ord = (q & 15) * 2 + (q >> 4);
        const unsigned per = (xcc & 4) ? PERX8 : PER8;
        const unsigned t0 = tstart + (unsigned)LEAD + ord * per / 256;
        if (q < 16) send_panel<false>(p, L, myws, q, d, lane, t0, per);
        else send_panel<true>(p, L, myws, q, d, lane, t0, per);
        return;
    }

    if (wave == 11) {
        asm volatile("buffer_inv sc0 sc1" ::: "memory");
        __builtin_amdgcn_s_barrier();
        FENCE();
        const unsigned call = rfl(lds_ld(L, L_CALL));
        const unsigned fbase = (call - 1u) * FSTRIDE;
        wait_vm<0>();
        const int d = lane & 7;
        const unsigned srcr = (unsigned)(p.rank + d) & 7u;
        const unsigned* addr = lane < 8 ? (const unsigned*)(myws + WS_FLAG + ti * 128 + srcr * 4)
                             : lane < 16 ? (const unsigned*)(myws + WS_FLAG + (16 + tj) * 128 + srcr * 4)
                             : (const unsigned*)(myws + WS_TMO);
        const unsigned row0 = ((call & 1u) * NR + srcr) * NCH;
        const int nmax = lane >= 8 ? 0 : d == 0 ? NCH : NBULK, amax = lane >= 8 || d == 0 ? 0 : 8;
        int seen = 0, qn = 0;
        int seen2 = 0, fq = 0, glim = 2;
        for (;;) {
            unsigned v = __hip_atomic_load(addr, __ATOMIC_RELAXED, __HIP_MEMORY_SCOPE_SYSTEM);
            const unsigned tmo = __builtin_amdgcn_readlane(v, 16);
            bool dead = tmo == call;
            if (!dead && (unsigned)rt() - tstart > DEADLINE) {
                if (lane == 0) __hip_atomic_store((unsigned*)(myws + WS_TMO), call, __ATOMIC_RELAXED, __HIP_MEMORY_SCOPE_AGENT);
                dead = true;
            }
            int nx = (int)(v - fbase), nw = dpp0<0x128>(nx);
            if (dead) { nx = nw = 64; lds_st(L, L_BAD, 1u); }
            int nn = min(min(nx, nw), nmax) - seen;
            nn = max(nn, 0);
            if (__ballot(nn > 0)) {
                int pre = nn;
                pre += dpp0<0x111>(pre); pre += dpp0<0x112>(pre); pre += dpp0<0x114>(pre);
                const int pos = qn + pre - nn;
                FENCE();
                for (int j = 0; j < nn; ++j) lds_st(L, L_QUEUE + (pos + j) * 4, row0 + seen + j + 1u);
                seen += nn;
                qn += __builtin_amdgcn_readlane(pre, 7);
            }
            int a = min(nx - NBULK, amax);
            if (nw < NCH) a = min(a, (nw - NBULK) >> 1);
            for (;;) {
                const int n2 = max(min(a, glim) - seen2, 0);
                if (!__ballot(n2 > 0)) break;
                int pre = n2;
                pre += dpp0<0x111>(pre); pre += dpp0<0x112>(pre); pre += dpp0<0x114>(pre);
                const int pos = fq + pre - n2;
                FENCE();
                for (int j = 0; j < n2; ++j) {
                    const unsigned pr = (seen2 + j) & 1;
                    *(volatile LAS unsigned short*)(L + L_FIN + (pos + j) * 4) =
                        (unsigned short)((2 * d - 2 + pr) << 12 | (row0 + NBULK + ((seen2 + j) >> 1)));
                }
                seen2 += n2;
                fq += __builtin_amdgcn_readlane(pre, 7);
                if (fq != glim * (NR - 1)) break;
                STAMP(31 + glim / 2);
                glim += 2;
            }
            if (qn >= NQ && fq >= NFIN) break;
            __builtin_amdgcn_s_sleep(POLLSLEEP);
        }
        STAMP(10);
        return;
    }

    {
        const int ld = wave - 8;
        __builtin_amdgcn_s_barrier();
        FENCE();
        const unsigned voff = lane * 16;
        const unsigned char* dbase = myws + WS_DATA + (size_t)(ld ? 16 + tj : ti) * CH;
        const unsigned lb = (unsigned)(uintptr_t)L;
        int iss = 0, done = 0;
        unsigned spin = 0;
        while (done < NQ) {
            const unsigned v = lds_ld(L, lane < 16 ? L_PROG + (lane & 7) * 4 : L_QUEUE + (iss + (lane & 15)) * 4);
            int lim = iss + __builtin_ctzll(~(__ballot(v != 0u) >> 16));
            lim = min(min(lim, iss + 16), min(NQ, done + 15));
            const int st = iss >> 2;
            int fr = st + NSLOT;
#pragma unroll
            for (int j = NSLOT - 1; j >= 0; --j) if (__ballot(lane < 8 && (int)v < st + j - (NSLOT - 1))) fr = st + j;
            lim = min(lim, fr * 4);
            FENCE();
            const bool any = iss < lim;
            for (const int i0 = iss; iss < lim; ++iss) {
                const unsigned e = __builtin_amdgcn_readlane(v, 16 + iss - i0) - 1u;
                dma4k(dbase + (size_t)e * ROW_BYTES, lb + ld * 16384 + ((iss >> 2) & (NSLOT - 1)) * SLOT + (iss & 3) * CH, voff);
            }
            const int tg = iss - ((vm_count() + 3) >> 2);
            if (tg > done) {
                FENCE();
                done = tg;
                lds_st(L, L_FULL + ld * 4, (unsigned)done);
            } else if (!any) {
                if (expired(spin, tstart)) { lds_st(L, L_BAD, 1u); break; }
                __builtin_amdgcn_s_sleep(1);
            }
        }
        auto fin = [&](auto isw) {
            constexpr bool W = decltype(isw)::value;
            constexpr int NF = W ? NPAIR : NFIN;
            unsigned xa = lb + L_XR;
            iss = 0; done = 0;
            while (done < NF) {
                const unsigned v = lds_ld(L, lane < 16 ? L_PROG + (lane & 7) * 4 : L_FIN + (iss + (lane & 15)) * 4);
                int n = __builtin_ctz(~(unsigned)(__ballot((v & 0xfffu) != 0u) >> 16) | (1u << (W ? WBATCH : XBATCH)));
                int pm = (int)v;
                pm = min(pm, dpp0<0x121>(pm)); pm = min(pm, dpp0<0x122>(pm)); pm = min(pm, dpp0<0x124>(pm));
                pm = (int)rfl(pm) - NSB;
                int lim = W ? done + 7 : 2 * (pm + NXB);
                lim = lim < NF ? lim : NF;
                lim = pm < 0 ? 0 : lim - iss;
                n = n < lim ? n : lim;
                const unsigned off = (v & 0x1ffu) << 17 | (W ? (v & 0x1000u) << 6 : (v & 0x1000u) >> 1);
                FENCE();
                for (int j = 0; j < n; ++j) {
                    const unsigned vo = voff + __builtin_amdgcn_readlane(off, 16 + j);
                    if constexpr (W) {
                        const unsigned la = lb + (__builtin_amdgcn_readlane(v, 16 + j) & 0xf000u) * 2;
                        dma4k(dbase, la, vo);
                        dma4k(dbase + ROW_BYTES, la + CH, vo);
                    } else {
                        dma2k(dbase, xa, vo);
                        xa = xa == lb + L_CTL - 2048 ? lb + L_XR : xa + 2048;
                    }
                }
                if (n > 0) iss += n;
                const int vm = vm_count();
                const int tg = iss - (W ? (vm + 7) >> 3 : (vm + 1) >> 1);
                if (tg > done) {
                    FENCE();
                    const int last = tg == NF ? NFIN : tg;
                    if (done + lane < last) *(volatile LAS unsigned char*)(L + L_FIN + (done + lane) * 4 + 2 + W) = 1;
                    done = tg;
                } else if (n <= 0) {
                    if (expired(spin, tstart)) { lds_st(L, L_BAD, 1u); break; }
                    __builtin_amdgcn_s_sleep(1);
                }
            }
        };
        if (ld) fin(std::true_type{}); else fin(std::false_type{});
        STAMP(11 + 2 * ld);
    }
}

}

extern "C" void kernel_launch(int rank, int nranks, ncclComm_t comm, hipStream_t stream,
                              void* const* d_in, const int* in_sizes, int n_in,
                              void* d_out, int out_size, const int* in_dtypes, int out_dtype,
                              void* const* d_out_all, void* const* d_ws_all, size_t ws_bytes) {
    if (nranks != NR || n_in != 4 || ws_bytes < WS_NEED || in_sizes[0] != MM * KS || in_sizes[1] != KS * NN ||
        out_size != MM * NN || (unsigned)rank >= (unsigned)NR) {
        fprintf(stderr, "kernel_launch: unexpected problem geometry\n");
        return;
    }
    Params p;
    p.x = (const unsigned char*)d_in[0]; p.w = (const unsigned char*)d_in[1];
    p.sx = (const float*)d_in[2]; p.sw = (const float*)d_in[3];
    p.out = (float*)d_out; p.rank = rank;
    for (int r = 0; r < NR; ++r) p.ws[r] = (unsigned char*)d_ws_all[r];
    p.dtab = 0;
    for (int r = 0, lo = 0, hi = 4; r < NR; ++r) {
        const int dst = (rank + NR - 1 - r) & 7;
        const bool same = ((0xB2 >> dst) & 1) == ((0xB2 >> rank) & 1);
        p.dtab |= (unsigned)((rank - dst) & 7) << (4 * (same ? lo++ : hi++));
    }
    hipLaunchKernelGGL(fused_gemm_ag, dim3(NWG), dim3(NTHREADS), 0, stream, p);
    HIP_CHECK(hipGetLastError());
}
```

```cpp
#include <hip/hip_runtime.h>
#include <rccl/rccl.h>
#include <cstdio>
#include <cstdint>
#include <type_traits>

#define HIP_CHECK(expr) do { hipError_t _e = (expr); if (_e != hipSuccess) { \
    fprintf(stderr, "HIP error %s at %s:%d\n", hipGetErrorString(_e), __FILE__, __LINE__); \
    return; } } while (0)

namespace {

#define LAS __attribute__((address_space(3)))
typedef int v4i __attribute__((ext_vector_type(4)));
typedef int v8i __attribute__((ext_vector_type(8)));
typedef unsigned v2u __attribute__((ext_vector_type(2)));
typedef unsigned v4u __attribute__((ext_vector_type(4)));
typedef float v4f __attribute__((ext_vector_type(4)));
typedef float v16f __attribute__((ext_vector_type(16)));
typedef unsigned long long u64;

constexpr int NR = 8;
constexpr int MM = 4096, NN = 4096, KS = 512;
constexpr int NCH = 32;
constexpr int NBULK = 28;
constexpr int NPANEL = 32;
constexpr int CH = 4096;
constexpr int NWG = 256, NTHREADS = 768;
constexpr int NQ = (NR - 1) * NBULK + NCH;
constexpr int NSB = NQ / 4;
constexpr int NPAIR = 2 * (NR - 1);
constexpr int NFIN = 4 * NPAIR;
constexpr unsigned FSTRIDE = 64;
constexpr size_t WS_CTR = 0;
constexpr size_t WS_FLAG = 16384;
constexpr size_t WS_TMO = 16384 + 4096;
constexpr size_t WS_DBG = 32768;
constexpr size_t WS_DATA = 1u << 20;
constexpr size_t ROW_BYTES = (size_t)NPANEL * CH;
constexpr size_t SRC_BYTES = (size_t)NCH * ROW_BYTES;
constexpr size_t PAR_BYTES = (size_t)NR * SRC_BYTES;
constexpr size_t WS_NEED = WS_DATA + 2 * PAR_BYTES;
constexpr unsigned DEADLINE = 40000000u;
constexpr unsigned DEADLINE2 = 60000000u;
constexpr int NSLOT = 4, SLOT = 32768;
constexpr int L_XR = NPAIR * 2 * CH;
constexpr int NXB = 11;
constexpr int L_CTL = L_XR + NXB * CH;
constexpr int L_FULL = L_CTL;
constexpr int L_PROG = L_CTL + 64;
constexpr int L_BAD = L_CTL + 128;
constexpr int L_QUEUE = L_CTL + 256;
constexpr int L_FIN = L_CTL + 1536;
constexpr int L_CALL = L_CTL + 2048;
constexpr int LDS_BYTES = L_CALL + 64 + 256;
static_assert(L_QUEUE + (NQ + 32) * 4 <= L_FIN && L_FIN + (NFIN + 32) * 4 <= L_CALL && L_CTL >= NSLOT * SLOT, "LDS map");

#define PER8 448
#define PERX8 452
#define LEAD 50
#define POLLSLEEP 2
#define LATE1 100000
#define LATE2 100000
#define XBATCH 8
#define WBATCH 4
#define OWIN 63
#define XMID 1

struct Params {
    const unsigned char* x; const unsigned char* w; unsigned char* myws;
    int rank;
    unsigned dtab;
    const float* sx; const float* sw; float* out;
    unsigned char* dws[NR];
};

__device__ __forceinline__ u64 rt() { return __builtin_amdgcn_s_memrealtime(); }
__device__ __forceinline__ unsigned rfl(unsigned v) { return __builtin_amdgcn_readfirstlane(v); }
__device__ __forceinline__ unsigned lds_ld(LAS unsigned char* L, int off) { return *(volatile LAS unsigned*)(L + off); }
__device__ __forceinline__ void lds_st(LAS unsigned char* L, int off, unsigned v) { *(volatile LAS unsigned*)(L + off) = v; }
#define FENCE() asm volatile("" ::: "memory")
template <int N> __device__ __forceinline__ void wait_vm() { asm volatile("s_waitcnt vmcnt(%0)" :: "n"(N) : "memory"); }
__device__ __forceinline__ int vm_count() {
    unsigned r;
    asm volatile("s_getreg_b32 %0, hwreg(HW_REG_IB_STS)" : "=s"(r) :: "memory");
    return (int)((r & 15u) | ((r >> 18) & 48u));
}
__device__ __forceinline__ bool expired(unsigned& spin, unsigned t0) {
    return (++spin & 4095u) == 0u && (unsigned)rt() - t0 > DEADLINE2;
}
template <int CTRL> __device__ __forceinline__ int dpp0(int v) { return __builtin_amdgcn_update_dpp(0, v, CTRL, 0xf, 0xf, true); }

#define STAMPL(i) do {} while (0)
#define STAMP(i) do {} while (0)

__device__ __forceinline__ void dma4k(const unsigned char* g, unsigned ldsaddr, unsigned voff) {
    asm volatile("s_mov_b32 m0, %2\n\ts_nop 0\n\t"
                 "global_load_lds_dwordx4 %0, %1 offset:0\n\tglobal_load_lds_dwordx4 %0, %1 offset:1024\n\t"
                 "global_load_lds_dwordx4 %0, %1 offset:2048\n\tglobal_load_lds_dwordx4 %0, %1 offset:3072"
                 :: "v"(voff), "s"(g), "s"(ldsaddr) : "memory");
}
__device__ __forceinline__ void dma2k(const unsigned char* g, unsigned ldsaddr, unsigned voff) {
    asm volatile("s_mov_b32 m0, %2\n\ts_nop 0\n\t"
                 "global_load_lds_dwordx4 %0, %1 offset:0\n\tglobal_load_lds_dwordx4 %0, %1 offset:1024"
                 :: "v"(voff), "s"(g), "s"(ldsaddr) : "memory");
}
__device__ __forceinline__ void st16_sys(unsigned char* sbase, unsigned voff, v4u v) {
    asm volatile("global_store_dwordx4 %0, %1, %2 sc0 sc1\n\ts_nop 1" :: "v"(voff), "v"(v), "s"(sbase) : "memory");
}
__device__ __forceinline__ void st4_sys_lane0(unsigned* p, unsigned v) {
    u64 ex;
    asm volatile("s_mov_b64 %0, exec\n\ts_mov_b64 exec, 1\n\tglobal_store_dword %1, %2, %3 sc0 sc1\n\ts_mov_b64 exec, %0"
                 : "=&s"(ex) : "v"(0u), "v"(v), "s"(p) : "memory");
}
__device__ __forceinline__ void ld4(unsigned& d, unsigned voff, const unsigned char* sbase) {
    asm volatile("global_load_dword %0, %1, %2" : "=v"(d) : "v"(voff), "s"(sbase) : "memory");
}
template <int OFF> __device__ __forceinline__ void ld16(v4u& d, unsigned voff, const unsigned char* sbase) {
    asm volatile("global_load_dwordx4 %0, %1, %2 offset:%3" : "=v"(d) : "v"(voff), "s"(sbase), "n"(OFF) : "memory");
}
__device__ __forceinline__ void landed(unsigned (&g)[16]) {
    asm volatile("" : "+v"(g[0]), "+v"(g[1]), "+v"(g[2]), "+v"(g[3]), "+v"(g[4]), "+v"(g[5]), "+v"(g[6]), "+v"(g[7]),
                      "+v"(g[8]), "+v"(g[9]), "+v"(g[10]), "+v"(g[11]), "+v"(g[12]), "+v"(g[13]), "+v"(g[14]), "+v"(g[15]) :: "memory");
}
__device__ __forceinline__ void landed(v4u& a, v4u& b, v4u& c, v4u& d) {
    asm volatile("" : "+v"(a), "+v"(b), "+v"(c), "+v"(d) :: "memory");
}
__device__ __forceinline__ void swap16(v4u& a, v4u& b, unsigned mask) {
    unsigned x0 = a[0], x1 = a[1], x2 = a[2], x3 = a[3], y0 = b[0], y1 = b[1], y2 = b[2], y3 = b[3];
    asm volatile("s_mov_b32 exec_lo, %8\n\ts_mov_b32 exec_hi, %8\n\tv_swap_b32 %0, %4\n\tv_swap_b32 %1, %5\n\tv_swap_b32 %2, %6\n\tv_swap_b32 %3, %7\n\t"
                 "s_mov_b64 exec, -1"
                 : "+v"(x0), "+v"(x1), "+v"(x2), "+v"(x3), "+v"(y0), "+v"(y1), "+v"(y2), "+v"(y3) : "s"(mask));
    a[0] = x0; a[1] = x1; a[2] = x2; a[3] = x3; b[0] = y0; b[1] = y1; b[2] = y2; b[3] = y3;
}

template <bool IS_W>
__device__ __forceinline__ void send_panel(const unsigned char* src, LAS unsigned char* L, unsigned char* myws, unsigned char* dws,
                                           int rank, int q, int d, int lane, unsigned t0, unsigned per) {
    const bool self = d == 0;
    const unsigned lo = IS_W ? lane * 4 : (lane >> 2) * 4 * KS + (lane & 3) * 16;
    const unsigned so = lane * 16;
    unsigned gw[4][16];
    v4u ow[2][4];
    v4u gx[2][16];
    auto loadw = [&](int i, unsigned (&g)[16]) {
#pragma unroll
        for (int k = 0; k < 16; ++k) ld4(g[k], lo, src + (size_t)(i * 16 + k) * NN);
    };
    auto xformw = [&](unsigned (&g)[16], v4u (&ob)[4]) {
        landed(g);
#pragma unroll
        for (int t = 0; t < 4; ++t) {
            const unsigned a0 = g[4 * t], a1 = g[4 * t + 1], a2 = g[4 * t + 2], a3 = g[4 * t + 3];
            const unsigned x0 = __builtin_amdgcn_perm(a1, a0, 0x05010400u), x1 = __builtin_amdgcn_perm(a1, a0, 0x07030602u);
            const unsigned y0 = __builtin_amdgcn_perm(a3, a2, 0x05010400u), y1 = __builtin_amdgcn_perm(a3, a2, 0x07030602u);
            ob[0][t] = __builtin_amdgcn_perm(y0, x0, 0x05040100u);
            ob[1][t] = __builtin_amdgcn_perm(y0, x0, 0x07060302u);
            ob[2][t] = __builtin_amdgcn_perm(y1, x1, 0x05040100u);
            ob[3][t] = __builtin_amdgcn_perm(y1, x1, 0x07060302u);
        }
        landed(ob[0], ob[1], ob[2], ob[3]);
    };
    auto loadx = [&](int grp, int half, v4u (&g)[16]) {
#pragma unroll
        for (int j = 2 * half; j < 2 * half + 2; ++j) {
            const unsigned char* s = src + (size_t)j * 64 * KS + grp * 64;
            ld16<0>(g[4 * j], lo, s); ld16<KS>(g[4 * j + 1], lo, s); ld16<2 * KS>(g[4 * j + 2], lo, s); ld16<3 * KS>(g[4 * j + 3], lo, s);
        }
    };
    const bool b0 = lane & 1, b1 = lane & 2;
    auto xformx = [&](int half, v4u (&g)[16]) {
#pragma unroll
        for (int j = 2 * half; j < 2 * half + 2; ++j) {
            v4u &r0 = g[4 * j], &r1 = g[4 * j + 1], &r2 = g[4 * j + 2], &r3 = g[4 * j + 3];
            landed(r0, r1, r2, r3);
#pragma unroll
            for (int e = 0; e < 4; ++e) {
                const unsigned a0 = r0[e], a1 = r1[e], a2 = r2[e], a3 = r3[e];
                const unsigned u0 = __builtin_amdgcn_mov_dpp(a0, 0xB1, 0xf, 0xf, true), u1 = __builtin_amdgcn_mov_dpp(a1, 0xB1, 0xf, 0xf, true);
                const unsigned u2 = __builtin_amdgcn_mov_dpp(a2, 0xB1, 0xf, 0xf, true), u3 = __builtin_amdgcn_mov_dpp(a3, 0xB1, 0xf, 0xf, true);
                const unsigned n0 = b0 ? u1 : a0, n1 = b0 ? a1 : u0, n2 = b0 ? u3 : a2, n3 = b0 ? a3 : u2;
                const unsigned w0 = __builtin_amdgcn_mov_dpp(n0, 0x4E, 0xf, 0xf, true), w1 = __builtin_amdgcn_mov_dpp(n1, 0x4E, 0xf, 0xf, true);
                const unsigned w2 = __builtin_amdgcn_mov_dpp(n2, 0x4E, 0xf, 0xf, true), w3 = __builtin_amdgcn_mov_dpp(n3, 0x4E, 0xf, 0xf, true);
                r0[e] = b1 ? w2 : n0; r2[e] = b1 ? n2 : w0; r1[e] = b1 ? w3 : n1; r3[e] = b1 ? n3 : w1;
            }
            landed(r0, r1, r2, r3);
        }
    };
    unsigned* ctr = (unsigned*)(myws + WS_CTR + (size_t)blockIdx.x * 64);
    unsigned prevv;
    STAMPL(9);
    asm volatile("global_load_dword %0, %1, %2 sc1" : "=v"(prevv) : "v"(0u), "s"(ctr) : "memory");
    if constexpr (IS_W) { loadw(0, gw[0]); loadw(1, gw[1]); } else { loadx(0, 0, gx[0]); loadx(0, 1, gx[0]); }
    STAMPL(10);
    asm volatile("s_waitcnt vmcnt(%1)" : "+v"(prevv) : "n"(IS_W ? 32 : 16) : "memory");
    STAMPL(11);
    const unsigned prev = rfl(prevv), call = prev + 1u, fbase = prev * FSTRIDE;
    lds_st(L, L_CALL, call);
    asm volatile("s_waitcnt lgkmcnt(0)" ::: "memory");
    __builtin_amdgcn_s_barrier();
    FENCE();
    STAMPL(12);
    unsigned char* db = dws + WS_DATA + (size_t)(call & 1u) * PAR_BYTES + (size_t)rank * SRC_BYTES + (size_t)q * CH;
    unsigned* dflag = (unsigned*)(dws + WS_FLAG + q * 128 + rank * 4);
    const unsigned fmask = ((d & 1) ? 0x05555555u : 0x0AAAAAAAu) | 0x08000000u | ((IS_W || self) ? 0xA0000000u : 0xF0000000u);
    if constexpr (IS_W) { loadw(2, gw[2]); wait_vm<32>(); STAMPL(13); xformw(gw[0], ow[0]); }
    else { wait_vm<8>(); STAMPL(13); xformx(0, gx[0]); wait_vm<0>(); xformx(1, gx[0]); }
    int ops = IS_W ? 32 : 0, h1 = 0, h2 = 0, h3 = 0, h4 = 0, h5 = 0, h6 = 0;
    int l1 = IS_W ? 16 : 0, l2 = IS_W ? 32 : 0, l3 = 0;
    int ia = 0, ib = 0;
    unsigned fa = 0, fb = 0;
    unsigned tt = 0, period = self ? 0u : per;
    bool wide = self;
    auto flag_try = [&](int ret) {
        if (ia != 0 && ret >= ia) { st4_sys_lane0(dflag, fa); ++ops; ia = ib; fa = fb; ib = 0; }
    };
    auto flag_put = [&](unsigned v) {
        if (ia == 0) { ia = h1; fa = v; } else { ib = h1; fb = v; }
    };
    auto slot = [&](unsigned char* a, v4u& v) {
        const int n0 = wide ? h4 + 0 : h2 + 0, n1 = wide ? h6 + 0 : h3 + 0, n2 = wide ? h6 + 0 : h4 + 0;
        const unsigned due = t0 + (tt >> 3);
        for (;;) {
            const int late = (int)((unsigned)rt() - due);
            const int ret = ops - vm_count();
            flag_try(ret);
            if (late >= 0 && ret >= (late >= LATE2 ? n2 : late >= LATE1 ? n1 : n0)) break;
            __builtin_amdgcn_s_sleep(1);
        }
        st16_sys(a, so, v);
        h6 = h5; h5 = h4; h4 = h3; h3 = h2; h2 = h1; h1 = ++ops;
        tt += period;
    };
    auto piece = [&](int i, v4u& s0, v4u& s1, v4u& s2, v4u& s3, auto&& loads, auto&& xform) {
        if (i == NBULK + 1 && !self) {
            if (!IS_W) tt += 6u * per - per / 2;
            period = per / 2; wide = true;
        }
        const unsigned fv = fbase + ((IS_W || self || i < NBULK) ? i + 1 : 2 * i - (NBULK - 2));
        slot(db, s0);
        slot(db + 1024, s1);
        if (XMID && !IS_W && !self && i > NBULK) flag_put(fv - 1u);
        loads();
        slot(db + 2048, s2);
        xform();
        slot(db + 3072, s3);
        if ((fmask >> i) & 1u) flag_put(fv);
        db += ROW_BYTES;
    };
    auto arrived = [&](int l) { while (ops - vm_count() < l) __builtin_amdgcn_s_sleep(1); };
    STAMPL(14);
    if constexpr (IS_W) {
        auto wpiece = [&](int i, v4u (&oc)[4], v4u (&on)[4], unsigned (&gl)[16], unsigned (&gt)[16]) {
            piece(i, oc[0], oc[1], oc[2], oc[3],
                  [&]() { if (i + 3 < NCH) { loadw(i + 3, gl); ops += 16; } l3 = ops; },
                  [&]() { arrived(l1); xformw(gt, on); l1 = l2; l2 = l3; });
        };
#pragma unroll 1
        for (int i = 0; i < NCH; i += 4) {
            wpiece(i, ow[0], ow[1], gw[3], gw[1]);
            wpiece(i + 1, ow[1], ow[0], gw[0], gw[2]);
            wpiece(i + 2, ow[0], ow[1], gw[1], gw[3]);
            wpiece(i + 3, ow[1], ow[0], gw[2], gw[0]);
        }
    } else {
        auto group = [&](int g, v4u (&c)[16], v4u (&n)[16]) {
            const bool more = g + 1 < NCH / 4;
            piece(4 * g, c[0], c[4], c[8], c[12], [&]() { if (more) { loadx(g + 1, 0, n); ops += 8; l1 = ops; } }, [&]() {});
            piece(4 * g + 1, c[1], c[5], c[9], c[13], [&]() { if (more) { loadx(g + 1, 1, n); ops += 8; l2 = ops; } }, [&]() {});
            piece(4 * g + 2, c[2], c[6], c[10], c[14], [&]() {}, [&]() { arrived(l1); xformx(0, n); });
            piece(4 * g + 3, c[3], c[7], c[11], c[15], [&]() {}, [&]() { arrived(l2); xformx(1, n); });
        };
#pragma unroll 1
        for (int g = 0; g < NCH / 4; g += 2) {
            group(g, gx[0], gx[1]);
            const unsigned last = rfl(g == NCH / 4 - 2 && !self ? ~0u : 0u);
#pragma unroll
            for (int a = 0; a < 4; ++a)
#pragma unroll
                for (int c = a + 1; c < 4; ++c) swap16(gx[1][4 * a + c], gx[1][4 * c + a], last);
            group(g + 1, gx[1], gx[0]);
        }
    }
    while (ia != 0) flag_try(ops - vm_count());
    if (lane == 0) __hip_atomic_store(ctr, call, __ATOMIC_RELAXED, __HIP_MEMORY_SCOPE_AGENT);
    STAMP(9);
}

__global__ void __launch_bounds__(NTHREADS) fused_gemm_ag(Params p) {
    __shared__ __attribute__((aligned(1024))) unsigned char lds_raw[LDS_BYTES];
    LAS unsigned char* L = (LAS unsigned char*)lds_raw;
    const int tid = threadIdx.x, lane = tid & 63;
    const int wave = rfl(tid >> 6);
    const int b = blockIdx.x, xcd = b & 7, idx = b >> 3;
    const int ti = (xcd >> 1) * 4 + (idx >> 3), tj = (xcd & 1) * 8 + (idx & 7);
    unsigned xcc;
    asm volatile("s_getreg_b32 %0, hwreg(HW_REG_XCC_ID)" : "=s"(xcc));
    const int q = (idx & 4) * 4 + (xcc & 3) * 4 + (idx & 3);
    int sl = (xcc & 4) + (idx >> 3);
    if (((idx >> 2) & 1) == (xcc & 1)) sl = sl == 1 ? 4 + (idx & 3) : sl == 4 + (idx & 3) ? 1 : sl;
    const unsigned char *px = p.x, *pw = p.w;
    unsigned char *myws = p.myws, *dws = p.dws[sl & 7];
    int rank = p.rank;
    unsigned dtab = p.dtab;
    unsigned tstart = (unsigned)rt();
    asm volatile("" : "+s"(px), "+s"(pw), "+s"(myws), "+s"(dws), "+s"(rank), "+s"(dtab), "+s"(tstart));

    if (wave < 8) {
        if (wave == 0) STAMP(0);
        lds_st(L, L_CTL + wave * 256 + lane * 4, 0u);
        float scale = p.sx[0] * p.sw[0];
        asm volatile("s_waitcnt lgkmcnt(0)" ::: "memory");
        __builtin_amdgcn_s_barrier();
        FENCE();
        scale = __builtin_bit_cast(float, rfl(__builtin_bit_cast(unsigned, scale)));
        const int wr = wave >> 1, wc = wave & 1, i = lane & 31, h = lane >> 5;
        unsigned char* otile = (unsigned char*)(p.out + (size_t)(ti * 256 + wr * 64) * NN + tj * 256 + wc * 128);
        asm volatile("" : "+s"(scale), "+s"(otile));
        const int abase = h * 8192 + (64 * wr + i) * 16;
        const int bbase = 16384 + h * 8192 + (32 * wc + i) * 16;
        v16f acc[2][4];
#pragma unroll
        for (int m = 0; m < 2; ++m)
#pragma unroll
            for (int c = 0; c < 4; ++c)
#pragma unroll
                for (int e = 0; e < 16; ++e) acc[m][c][e] = 0.f;
        unsigned spin = 0;
        for (int s = 0; s < NSB; ++s) {
            for (;;) {
                v2u f = *(volatile LAS v2u*)(L + L_FULL);
                unsigned mn = rfl(f.x < f.y ? f.x : f.y);
                if (mn >= (unsigned)(4 * s + 4)) break;
                if (expired(spin, tstart)) { lds_st(L, L_BAD, 1u); break; }
                __builtin_amdgcn_s_sleep(1);
            }
            FENCE();
            LAS unsigned char* sl = L + (s & (NSLOT - 1)) * SLOT;
            v8i a[2], bf[4];
#pragma unroll
            for (int m = 0; m < 2; ++m) {
                v4i lo = *(LAS v4i*)(sl + abase + m * 512), hi = *(LAS v4i*)(sl + abase + m * 512 + 4096);
                a[m] = __builtin_shufflevector(lo, hi, 0, 1, 2, 3, 4, 5, 6, 7);
            }
#pragma unroll
            for (int c = 0; c < 4; ++c) {
                v4i lo = *(LAS v4i*)(sl + bbase + c * 1024), hi = *(LAS v4i*)(sl + bbase + c * 1024 + 4096);
                bf[c] = __builtin_shufflevector(lo, hi, 0, 1, 2, 3, 4, 5, 6, 7);
                if (c == 3) {
                    asm volatile("s_waitcnt lgkmcnt(0)" : "+v"(bf[3]) :: "memory");
                    lds_st(L, L_PROG + wave * 4, (unsigned)(s + 1 < NSB ? s + 1 : NSB + 7 * wr));
                    FENCE();
                }
#pragma unroll
                for (int m = 0; m < 2; ++m)
                    acc[m][c] = __builtin_amdgcn_mfma_scale_f32_32x32x64_f8f6f4(a[m], bf[c], acc[m][c], 0, 0, 0, 0, 0, 0);
            }
        }
        if (wave == 0) STAMP(1);
        unsigned va;
        {
            const unsigned lb = (unsigned)(uintptr_t)L;
            unsigned vb = lb + (32 * wc + i) * 16, t0, t1;
            unsigned m = 7 * wr, blk = (7 * wr + NXB - 1) % NXB, cnt, tmp, rel;
            va = lb + L_XR + blk * CH + h * 2048 + i * 16;
            u64 now;
#define POLL_ADDR \
            "v_bfe_u32 %[t0], %[va], 11, 1\n\ts_lshl3_add_u32 %[tmp], %[m], %[fin]\n\t" \
            "v_lshl_add_u32 %[t0], %[t0], 2, %[tmp]\n\t"
#define MOVE \
            "s_add_u32 %[blk], %[blk], 1\n\ts_cmp_eq_u32 %[blk], %[nxb]\n\ts_cselect_b32 %[blk], 0, %[blk]\n\t" \
            "s_cselect_b32 %[tmp], %[back], %[fwd]\n\t" \
            "v_and_b32 %[t0], 0xf000, %[t0]\n\tv_and_b32 %[vb], 0x1fff, %[vb]\n\tv_add_u32 %[va], %[tmp], %[va]\n\t" \
            "v_lshl_add_u32 %[vb], %[t0], 1, %[vb]\n\t"
#define RD(r0, r1, adr, off, hi) \
            "ds_read_b128 v[" #r0 "], %[" #adr "] offset:" #off "\n\tds_read_b128 v[" #r1 "], %[" #adr "] offset:" #off "+" #hi "\n\t"
#define RD_A0 RD(136:139, 140:143, va, 0, 1024)
#define RD_A1 RD(144:147, 148:151, va, 512, 1024)
#define RD_B0(off) RD(152:155, 156:159, vb, off, 4096)
#define RD_B1(off) RD(160:163, 164:167, vb, off, 4096)
#define MM_(c, a, b) "v_mfma_f32_32x32x64_f8f6f4 %[" #c "], v[" #a "], v[" #b "], %[" #c "]\n\t"
            asm volatile(
                "s_mov_b32 %[cnt], 7\n"
                "1:\n\t"
                POLL_ADDR
                "ds_read_b32 %[t0], %[t0]\n\ts_waitcnt lgkmcnt(0)\n\t"
                "v_cmp_le_u32 vcc, 0x01010000, %[t0]\n\ts_cmp_eq_u64 vcc, -1\n\ts_cbranch_scc1 3f\n\t"
                "s_memrealtime %[now]\n\ts_waitcnt lgkmcnt(0)\n\t"
                "s_mov_b64 vcc, %[now]\n\ts_sub_u32 %[tmp], vcc_lo, %[ts]\n\t"
                "s_cmp_gt_u32 %[tmp], %[dl]\n\ts_cbranch_scc0 1b\n\t"
                "v_mov_b32 %[t0], %[bad]\n\tv_mov_b32 %[t1], 1\n\tds_write_b32 %[t0], %[t1]\n\t"
                "v_mov_b32 %[t0], %[prog]\n\tv_mov_b32 %[t1], 1000\n\tds_write_b32 %[t0], %[t1]\n\t"
                "s_branch 9f\n"
                "3:\n\t"
                MOVE
                RD_B0(0) RD_A0 RD_A1
                "\n4:\n\t"
                RD_B1(1024)
                "s_add_u32 %[m], %[m], 1\n\ts_add_u32 %[rel], %[m], %[nsb]\n\ts_sub_u32 %[cnt], %[cnt], 1\n\t"
                "s_cmp_eq_u32 %[cnt], 0\n\ts_cselect_b32 %[rel], 1000, %[rel]\n\t"
                "v_mov_b32 %[t0], %[prog]\n\tv_mov_b32 %[t1], %[rel]\n\t"
                "s_waitcnt lgkmcnt(4)\n\t"
                MM_(c00, 136:143, 152:159)
                "s_waitcnt lgkmcnt(2)\n\t"
                MM_(c10, 144:151, 152:159)
                "ds_write_b32 %[t0], %[t1]\n\t"
                RD_B0(2048)
                POLL_ADDR
                "s_waitcnt lgkmcnt(3)\n\t"
                MM_(c01, 136:143, 160:167)
                "ds_read_b32 %[t0], %[t0]\n\t"
                MM_(c11, 144:151, 160:167)
                RD_B1(3072)
                "s_waitcnt lgkmcnt(3)\n\t"
                MM_(c02, 136:143, 152:159)
                "s_waitcnt lgkmcnt(2)\n\t"
                "v_cmp_le_u32 vcc, 0x01010000, %[t0]\n\t"
                MM_(c12, 144:151, 152:159)
                "s_cmp_eq_u32 %[cnt], 0\n\ts_cbranch_scc1 6f\n\t"
                "s_cmp_eq_u64 vcc, -1\n\ts_cbranch_scc0 6f\n\t"
                MOVE
                RD_B0(0)
                "s_waitcnt lgkmcnt(2)\n\t"
                MM_(c03, 136:143, 160:167)
                RD_A0
                MM_(c13, 144:151, 160:167)
                RD_A1
                "s_branch 4b\n"
                "6:\n\t"
                "s_waitcnt lgkmcnt(0)\n\t"
                MM_(c03, 136:143, 160:167)
                MM_(c13, 144:151, 160:167)
                "s_cmp_eq_u32 %[cnt], 0\n\ts_cbranch_scc0 1b\n"
                "9:\n\t"
                "s_waitcnt lgkmcnt(0)\n\ts_nop 15\n\ts_nop 15"
                : [c00] "+v"(acc[0][0]), [c01] "+v"(acc[0][1]), [c02] "+v"(acc[0][2]), [c03] "+v"(acc[0][3]),
                  [c10] "+v"(acc[1][0]), [c11] "+v"(acc[1][1]), [c12] "+v"(acc[1][2]), [c13] "+v"(acc[1][3]),
                  [va] "+v"(va), [vb] "+v"(vb), [t0] "=&v"(t0), [t1] "=&v"(t1),
                  [m] "+s"(m), [blk] "+s"(blk), [cnt] "=&s"(cnt), [tmp] "=&s"(tmp), [rel] "=&s"(rel), [now] "=&s"(now)
                : [fin] "s"(lb + L_FIN), [prog] "s"(lb + L_PROG + wave * 4), [bad] "s"(lb + L_BAD), [ts] "s"(tstart),
                  [dl] "s"(DEADLINE2), [nsb] "n"(NSB), [nxb] "n"(NXB), [fwd] "s"(CH), [back] "n"(-(NXB - 1) * CH)
                : "memory", "vcc", "scc",
                  "v136", "v137", "v138", "v139", "v140", "v141", "v142", "v143", "v144", "v145", "v146", "v147",
                  "v148", "v149", "v150", "v151", "v152", "v153", "v154", "v155", "v156", "v157", "v158", "v159",
                  "v160", "v161", "v162", "v163", "v164", "v165", "v166", "v167");
        }
        if (wc == 0) STAMP(20 + wr);
        const bool bad = lds_ld(L, L_BAD) != 0u;
        const float mul = bad ? __builtin_nanf("") : scale;
        const unsigned oo = (va & 0x800u) * 32u + (va & 0x1f0u);
#pragma unroll
        for (int m = 0; m < 2; ++m)
#pragma unroll
            for (int e = 0; e < 16; ++e) {
                v4f v = {acc[m][0][e] * mul, acc[m][1][e] * mul, acc[m][2][e] * mul, acc[m][3][e] * mul};
                st16_sys(otile + (size_t)(m * 32 + (e & 3) + 8 * (e >> 2)) * NN * 4, oo, __builtin_bit_cast(v4u, v));
                if (OWIN < 32 && wr < 3) wait_vm<OWIN>();
            }
        return;
    }

    __builtin_amdgcn_s_setprio(3);
    if (wave == 10) {
        const int d = (dtab >> (4 * sl)) & 7;
        const int ord = q ^ 16;
        const unsigned per = (sl & 4) ? PERX8 : PER8;
        const unsigned t0 = tstart + (unsigned)LEAD + ord * per / 256;
        if (q < 16) send_panel<false>(px + (size_t)q * 256 * KS, L, myws, dws, rank, q, d, lane, t0, per);
        else send_panel<true>(pw + (q - 16) * 256, L, myws, dws, rank, q, d, lane, t0, per);
        return;
    }

    if (wave == 11) {
        __builtin_amdgcn_s_barrier();
        FENCE();
        const unsigned call = rfl(lds_ld(L, L_CALL));
        const unsigned fbase = (call - 1u) * FSTRIDE;
        const int d = lane & 7;
        const unsigned srcr = (unsigned)(rank + d) & 7u;
        const unsigned* addr = lane < 8 ? (const unsigned*)(myws + WS_FLAG + ti * 128 + srcr * 4)
                             : lane < 16 ? (const unsigned*)(myws + WS_FLAG + (16 + tj) * 128 + srcr * 4)
                             : (const unsigned*)(myws + WS_TMO);
        const unsigned row0 = ((call & 1u) * NR + srcr) * NCH;
        const int nmax = lane >= 8 ? 0 : d == 0 ? NCH : NBULK, amax = lane >= 8 || d == 0 ? 0 : 8;
        int seen = 0, qn = 0;
        int seen2 = 0, fq = 0, glim = 2;
        for (;;) {
            unsigned v = __hip_atomic_load(addr, __ATOMIC_RELAXED, __HIP_MEMORY_SCOPE_SYSTEM);
            const unsigned tmo = __builtin_amdgcn_readlane(v, 16);
            bool dead = tmo == call;
            if (!dead && (unsigned)rt() - tstart > DEADLINE) {
                if (lane == 0) __hip_atomic_store((unsigned*)(myws + WS_TMO), call, __ATOMIC_RELAXED, __HIP_MEMORY_SCOPE_AGENT);
                dead = true;
            }
            int nx = (int)(v - fbase), nw = dpp0<0x128>(nx);
            if (dead) { nx = nw = 64; lds_st(L, L_BAD, 1u); }
            int nn = min(min(nx, nw), nmax) - seen;
            nn = max(nn, 0);
            if (__ballot(nn > 0)) {
                int pre = nn;
                pre += dpp0<0x111>(pre); pre += dpp0<0x112>(pre); pre += dpp0<0x114>(pre);
                const int pos = qn + pre - nn;
                FENCE();
                for (int j = 0; j < nn; ++j) lds_st(L, L_QUEUE + (pos + j) * 4, row0 + seen + j + 1u);
                seen += nn;
                qn += __builtin_amdgcn_readlane(pre, 7);
            }
            int a = min(nx - NBULK, amax);
            if (nw < NCH) a = min(a, (nw - NBULK) >> 1);
            for (;;) {
                const int n2 = max(min(a, glim) - seen2, 0);
                if (!__ballot(n2 > 0)) break;
                int pre = n2;
                pre += dpp0<0x111>(pre); pre += dpp0<0x112>(pre); pre += dpp0<0x114>(pre);
                const int pos = fq + pre - n2;
                FENCE();
                for (int j = 0; j < n2; ++j) {
                    const unsigned pr = (seen2 + j) & 1;
                    *(volatile LAS unsigned short*)(L + L_FIN + (pos + j) * 4) =
                        (unsigned short)((2 * d - 2 + pr) << 12 | (row0 + NBULK + ((seen2 + j) >> 1)));
                }
                seen2 += n2;
                fq += __builtin_amdgcn_readlane(pre, 7);
                if (fq != glim * (NR - 1)) break;
                STAMP(31 + glim / 2);
                glim += 2;
            }
            if (qn >= NQ && fq >= NFIN) break;
            __builtin_amdgcn_s_sleep(POLLSLEEP);
        }
        STAMP(10);
        return;
    }

    {
        const int ld = wave - 8;
        __builtin_amdgcn_s_barrier();
        FENCE();
        const unsigned voff = lane * 16;
        const unsigned char* dbase = myws + WS_DATA + (size_t)(ld ? 16 + tj : ti) * CH;
        const unsigned lb = (unsigned)(uintptr_t)L;
        int iss = 0, done = 0;
        unsigned spin = 0;
        while (done < NQ) {
            const unsigned v = lds_ld(L, lane < 16 ? L_PROG + (lane & 7) * 4 : L_QUEUE + (iss + (lane & 15)) * 4);
            int lim = iss + __builtin_ctzll(~(__ballot(v != 0u) >> 16));
            lim = min(min(lim, iss + 16), min(NQ, done + 15));
            const int st = iss >> 2;
            int fr = st + NSLOT;
#pragma unroll
            for (int j = NSLOT - 1; j >= 0; --j) if (__ballot(lane < 8 && (int)v < st + j - (NSLOT - 1))) fr = st + j;
            lim = min(lim, fr * 4);
            FENCE();
            const bool any = iss < lim;
            for (const int i0 = iss; iss < lim; ++iss) {
                const unsigned e = __builtin_amdgcn_readlane(v, 16 + iss - i0) - 1u;
                dma4k(dbase + (size_t)e * ROW_BYTES, lb + ld * 16384 + ((iss >> 2) & (NSLOT - 1)) * SLOT + (iss & 3) * CH, voff);
            }
            const int tg = iss - ((vm_count() + 3) >> 2);
            if (tg > done) {
                FENCE();
                done = tg;
                lds_st(L, L_FULL + ld * 4, (unsigned)done);
            } else if (!any) {
                if (expired(spin, tstart)) { lds_st(L, L_BAD, 1u); break; }
                __builtin_amdgcn_s_sleep(1);
            }
        }
        auto fin = [&](auto isw) {
            constexpr bool W = decltype(isw)::value;
            constexpr int NF = W ? NPAIR : NFIN;
            unsigned xa = lb + L_XR;
            iss = 0; done = 0;
            while (done < NF) {
                const unsigned v = lds_ld(L, lane < 16 ? L_PROG + (lane & 7) * 4 : L_FIN + (iss + (lane & 15)) * 4);
                int n = __builtin_ctz(~(unsigned)(__ballot((v & 0xfffu) != 0u) >> 16) | (1u << (W ? WBATCH : XBATCH)));
                int pm = (int)v;
                pm = min(pm, dpp0<0x121>(pm)); pm = min(pm, dpp0<0x122>(pm)); pm = min(pm, dpp0<0x124>(pm));
                pm = (int)rfl(pm) - NSB;
                int lim = W ? done + 7 : 2 * (pm + NXB);
                lim = lim < NF ? lim : NF;
                lim = pm < 0 ? 0 : lim - iss;
                n = n < lim ? n : lim;
                const unsigned off = (v & 0x1ffu) << 17 | (W ? (v & 0x1000u) << 6 : (v & 0x1000u) >> 1);
                FENCE();
                for (int j = 0; j < n; ++j) {
                    const unsigned vo = voff + __builtin_amdgcn_readlane(off, 16 + j);
                    if constexpr (W) {
                        const unsigned la = lb + (__builtin_amdgcn_readlane(v, 16 + j) & 0xf000u) * 2;
                        dma4k(dbase, la, vo);
                        dma4k(dbase + ROW_BYTES, la + CH, vo);
                    } else {
                        dma2k(dbase, xa, vo);
                        xa = xa == lb + L_CTL - 2048 ? lb + L_XR : xa + 2048;
                    }
                }
                if (n > 0) iss += n;
                const int vm = vm_count();
                const int tg = iss - (W ? (vm + 7) >> 3 : (vm + 1) >> 1);
                if (tg > done) {
                    FENCE();
                    const int last = tg == NF ? NFIN : tg;
                    if (done + lane < last) *(volatile LAS unsigned char*)(L + L_FIN + (done + lane) * 4 + 2 + W) = 1;
                    done = tg;
                } else if (n <= 0) {
                    if (expired(spin, tstart)) { lds_st(L, L_BAD, 1u); break; }
                    __builtin_amdgcn_s_sleep(1);
                }
            }
        };
        if (ld) fin(std::true_type{}); else fin(std::false_type{});
        STAMP(11 + 2 * ld);
    }
}

}

extern "C" void kernel_launch(int rank, int nranks, ncclComm_t comm, hipStream_t stream,
                              void* const* d_in, const int* in_sizes, int n_in,
                              void* d_out, int out_size, const int* in_dtypes, int out_dtype,
                              void* const* d_out_all, void* const* d_ws_all, size_t ws_bytes) {
    if (nranks != NR || n_in != 4 || ws_bytes < WS_NEED || in_sizes[0] != MM * KS || in_sizes[1] != KS * NN ||
        out_size != MM * NN || (unsigned)rank >= (unsigned)NR) {
        fprintf(stderr, "kernel_launch: unexpected problem geometry\n");
        return;
    }
    Params p;
    p.x = (const unsigned char*)d_in[0]; p.w = (const unsigned char*)d_in[1];
    p.sx = (const float*)d_in[2]; p.sw = (const float*)d_in[3];
    p.out = (float*)d_out; p.rank = rank;
    p.myws = (unsigned char*)d_ws_all[rank];
    const unsigned dtab[NR] = {0x14735602u, 0x73164502u, 0x13654207u, 0x74623501u, 0x16245307u, 0x75234601u, 0x15724306u, 0x75143206u};
    p.dtab = dtab[rank];
    for (int s = 0; s < NR; ++s) p.dws[s] = (unsigned char*)d_ws_all[(rank + NR - ((p.dtab >> (4 * s)) & 7)) & 7];
    hipLaunchKernelGGL(fused_gemm_ag, dim3(NWG), dim3(NTHREADS), 0, stream, p);
    HIP_CHECK(hipGetLastError());
}
```

```cpp
#include <hip/hip_runtime.h>
#include <rccl/rccl.h>
#include <cstdio>
#include <cstdint>
#include <type_traits>

#define HIP_CHECK(expr) do { hipError_t _e = (expr); if (_e != hipSuccess) { \
    fprintf(stderr, "HIP error %s at %s:%d\n", hipGetErrorString(_e), __FILE__, __LINE__); \
    return; } } while (0)

namespace {

#define LAS __attribute__((address_space(3)))
typedef int v4i __attribute__((ext_vector_type(4)));
typedef int v8i __attribute__((ext_vector_type(8)));
typedef unsigned v2u __attribute__((ext_vector_type(2)));
typedef unsigned v4u __attribute__((ext_vector_type(4)));
typedef float v4f __attribute__((ext_vector_type(4)));
typedef float v16f __attribute__((ext_vector_type(16)));
typedef unsigned long long u64;

constexpr int NR = 8;
constexpr int MM = 4096, NN = 4096, KS = 512;
constexpr int NCH = 32;
constexpr int NBULK = 28;
constexpr int NPANEL = 32;
constexpr int CH = 4096;
constexpr int NWG = 256, NTHREADS = 768;
constexpr int NQ = (NR - 1) * NBULK + NCH;
constexpr int NSB = NQ / 4;
constexpr int NPAIR = 2 * (NR - 1);
constexpr int NFIN = 4 * NPAIR;
constexpr unsigned FSTRIDE = 64;
constexpr size_t WS_CTR = 0;
#define FREP 1
#define FLINE 128
#define FSTR (FLINE * FREP)
constexpr size_t WS_FLAG = 524288;
constexpr size_t WS_TMO = 16384 + 4096;
constexpr size_t WS_DBG = 32768;
constexpr size_t WS_DATA = 1u << 20;
constexpr size_t ROW_BYTES = (size_t)NPANEL * CH;
constexpr size_t SRC_BYTES = (size_t)NCH * ROW_BYTES;
constexpr size_t PAR_BYTES = (size_t)NR * SRC_BYTES;
#define NPAR 2
constexpr size_t WS_NEED = WS_DATA + NPAR * PAR_BYTES;
constexpr unsigned DEADLINE = 40000000u;
constexpr unsigned DEADLINE2 = 60000000u;
constexpr int NSLOT = 4, SLOT = 32768;
constexpr int L_XR = NPAIR * 2 * CH;
constexpr int NXB = 11;
constexpr int L_CTL = L_XR + NXB * CH;
constexpr int L_FULL = L_CTL;
constexpr int L_PROG = L_CTL + 64;
constexpr int L_BAD = L_CTL + 128;
constexpr int L_QUEUE = L_CTL + 256;
constexpr int L_FIN = L_CTL + 1536;
constexpr int L_CALL = L_CTL + 2048;
constexpr int LDS_BYTES = L_CALL + 64 + 256;
static_assert(L_QUEUE + (NQ + 32) * 4 <= L_FIN && L_FIN + (NFIN + 32) * 4 <= L_CALL && L_CTL >= NSLOT * SLOT, "LDS map");

#define PER8 442
#define PERX8 444
#define LEAD 50
#define POLLSLEEP 2
#define POLLSLEEP2 POLLSLEEP
#define PFAST 14
#define PD 1
#define PGAP 16
#define REBASE 1
#define RB0 10
#define XDELAY 200
#define FARLATE 0
#define RBMAX 100000
#define NOSWAP 0
#define LATE1 100000
#define LATE2 100000
#define XBATCH 8
#define WBATCH 4
#define OWIN 63
#define XMID 1

struct Params {
    const unsigned char* x; const unsigned char* w; unsigned char* myws;
    int rank;
    unsigned dtab;
    const float* sx; const float* sw; float* out;
    unsigned char* dws[NR];
};

__device__ __forceinline__ u64 rt() { return __builtin_amdgcn_s_memrealtime(); }
__device__ __forceinline__ unsigned rfl(unsigned v) { return __builtin_amdgcn_readfirstlane(v); }
__device__ __forceinline__ unsigned lds_ld(LAS unsigned char* L, int off) { return *(volatile LAS unsigned*)(L + off); }
__device__ __forceinline__ void lds_st(LAS unsigned char* L, int off, unsigned v) { *(volatile LAS unsigned*)(L + off) = v; }
#define FENCE() asm volatile("" ::: "memory")
template <int N> __device__ __forceinline__ void wait_vm() { asm volatile("s_waitcnt vmcnt(%0)" :: "n"(N) : "memory"); }
__device__ __forceinline__ int vm_count() {
    unsigned r;
    asm volatile("s_getreg_b32 %0, hwreg(HW_REG_IB_STS)" : "=s"(r) :: "memory");
    return (int)((r & 15u) | ((r >> 18) & 48u));
}
__device__ __forceinline__ bool expired(unsigned& spin, unsigned t0) {
    return (++spin & 4095u) == 0u && (unsigned)rt() - t0 > DEADLINE2;
}
template <int CTRL> __device__ __forceinline__ int dpp0(int v) { return __builtin_amdgcn_update_dpp(0, v, CTRL, 0xf, 0xf, true); }

#define STAMPL(i) do {} while (0)
#define STAMP(i) do {} while (0)

__device__ __forceinline__ void dma4k(const unsigned char* g, unsigned ldsaddr, unsigned voff) {
    asm volatile("s_mov_b32 m0, %2\n\ts_nop 0\n\t"
                 "global_load_lds_dwordx4 %0, %1 offset:0\n\tglobal_load_lds_dwordx4 %0, %1 offset:1024\n\t"
                 "global_load_lds_dwordx4 %0, %1 offset:2048\n\tglobal_load_lds_dwordx4 %0, %1 offset:3072"
                 :: "v"(voff), "s"(g), "s"(ldsaddr) : "memory");
}
__device__ __forceinline__ void dma2k(const unsigned char* g, unsigned ldsaddr, unsigned voff) {
    asm volatile("s_mov_b32 m0, %2\n\ts_nop 0\n\t"
                 "global_load_lds_dwordx4 %0, %1 offset:0\n\tglobal_load_lds_dwordx4 %0, %1 offset:1024"
                 :: "v"(voff), "s"(g), "s"(ldsaddr) : "memory");
}
__device__ __forceinline__ void st16_sys(unsigned char* sbase, unsigned voff, v4u v) {
    asm volatile("global_store_dwordx4 %0, %1, %2 sc0 sc1\n\ts_nop 1" :: "v"(voff), "v"(v), "s"(sbase) : "memory");
}
__device__ __forceinline__ void st4_sys_lane0(unsigned* p, unsigned v, unsigned voff) {
#define FOP "global_atomic_swap %0, %1, %2 sc1"
    asm volatile("s_mov_b64 exec, %3\n\t" FOP "\n\ts_mov_b64 exec, -1"
                 :: "v"(voff), "v"(v), "s"(p), "n"((1 << FREP) - 1) : "memory");
}
__device__ __forceinline__ void ld4(unsigned& d, unsigned voff, const unsigned char* sbase) {
    asm volatile("global_load_dword %0, %1, %2" : "=v"(d) : "v"(voff), "s"(sbase) : "memory");
}
template <int OFF> __device__ __forceinline__ void ld16(v4u& d, unsigned voff, const unsigned char* sbase) {
    asm volatile("global_load_dwordx4 %0, %1, %2 offset:%3" : "=v"(d) : "v"(voff), "s"(sbase), "n"(OFF) : "memory");
}
__device__ __forceinline__ void landed(unsigned (&g)[16]) {
    asm volatile("" : "+v"(g[0]), "+v"(g[1]), "+v"(g[2]), "+v"(g[3]), "+v"(g[4]), "+v"(g[5]), "+v"(g[6]), "+v"(g[7]),
                      "+v"(g[8]), "+v"(g[9]), "+v"(g[10]), "+v"(g[11]), "+v"(g[12]), "+v"(g[13]), "+v"(g[14]), "+v"(g[15]) :: "memory");
}
__device__ __forceinline__ void landed(v4u& a, v4u& b, v4u& c, v4u& d) {
    asm volatile("" : "+v"(a), "+v"(b), "+v"(c), "+v"(d) :: "memory");
}
__device__ __forceinline__ void swap16(v4u& a, v4u& b, unsigned mask) {
    unsigned x0 = a[0], x1 = a[1], x2 = a[2], x3 = a[3], y0 = b[0], y1 = b[1], y2 = b[2], y3 = b[3];
    asm volatile("s_mov_b32 exec_lo, %8\n\ts_mov_b32 exec_hi, %8\n\tv_swap_b32 %0, %4\n\tv_swap_b32 %1, %5\n\tv_swap_b32 %2, %6\n\tv_swap_b32 %3, %7\n\t"
                 "s_mov_b64 exec, -1"
                 : "+v"(x0), "+v"(x1), "+v"(x2), "+v"(x3), "+v"(y0), "+v"(y1), "+v"(y2), "+v"(y3) : "s"(mask));
    a[0] = x0; a[1] = x1; a[2] = x2; a[3] = x3; b[0] = y0; b[1] = y1; b[2] = y2; b[3] = y3;
}

template <bool IS_W>
__device__ __forceinline__ void send_panel(const unsigned char* src, LAS unsigned char* L, unsigned char* myws, unsigned char* dws,
                                           int rank, int q, int d, int lane, unsigned t0, unsigned per, int late1) {
    const bool self = d == 0;
    const unsigned lo = IS_W ? lane * 4 : (lane >> 2) * 4 * KS + (lane & 3) * 16;
    const unsigned so = lane * 16, fo = lane * FLINE;
    unsigned gw[4][16];
    v4u ow[2][4];
    v4u gx[2][16];
    auto loadw = [&](int i, unsigned (&g)[16]) {
#pragma unroll
        for (int k = 0; k < 16; ++k) ld4(g[k], lo, src + (size_t)(i * 16 + k) * NN);
    };
    auto xformw = [&](unsigned (&g)[16], v4u (&ob)[4]) {
        landed(g);
#pragma unroll
        for (int t = 0; t < 4; ++t) {
            const unsigned a0 = g[4 * t], a1 = g[4 * t + 1], a2 = g[4 * t + 2], a3 = g[4 * t + 3];
            const unsigned x0 = __builtin_amdgcn_perm(a1, a0, 0x05010400u), x1 = __builtin_amdgcn_perm(a1, a0, 0x07030602u);
            const unsigned y0 = __builtin_amdgcn_perm(a3, a2, 0x05010400u), y1 = __builtin_amdgcn_perm(a3, a2, 0x07030602u);
            ob[0][t] = __builtin_amdgcn_perm(y0, x0, 0x05040100u);
            ob[1][t] = __builtin_amdgcn_perm(y0, x0, 0x07060302u);
            ob[2][t] = __builtin_amdgcn_perm(y1, x1, 0x05040100u);
            ob[3][t] = __builtin_amdgcn_perm(y1, x1, 0x07060302u);
        }
        landed(ob[0], ob[1], ob[2], ob[3]);
    };
    auto loadx = [&](int grp, int half, v4u (&g)[16]) {
#pragma unroll
        for (int j = 2 * half; j < 2 * half + 2; ++j) {
            const unsigned char* s = src + (size_t)j * 64 * KS + grp * 64;
            ld16<0>(g[4 * j], lo, s); ld16<KS>(g[4 * j + 1], lo, s); ld16<2 * KS>(g[4 * j + 2], lo, s); ld16<3 * KS>(g[4 * j + 3], lo, s);
        }
    };
    const bool b0 = lane & 1, b1 = lane & 2;
    auto xformx = [&](int half, v4u (&g)[16]) {
#pragma unroll
        for (int j = 2 * half; j < 2 * half + 2; ++j) {
            v4u &r0 = g[4 * j], &r1 = g[4 * j + 1], &r2 = g[4 * j + 2], &r3 = g[4 * j + 3];
            landed(r0, r1, r2, r3);
#pragma unroll
            for (int e = 0; e < 4; ++e) {
                const unsigned a0 = r0[e], a1 = r1[e], a2 = r2[e], a3 = r3[e];
                const unsigned u0 = __builtin_amdgcn_mov_dpp(a0, 0xB1, 0xf, 0xf, true), u1 = __builtin_amdgcn_mov_dpp(a1, 0xB1, 0xf, 0xf, true);
                const unsigned u2 = __builtin_amdgcn_mov_dpp(a2, 0xB1, 0xf, 0xf, true), u3 = __builtin_amdgcn_mov_dpp(a3, 0xB1, 0xf, 0xf, true);
                const unsigned n0 = b0 ? u1 : a0, n1 = b0 ? a1 : u0, n2 = b0 ? u3 : a2, n3 = b0 ? a3 : u2;
                const unsigned w0 = __builtin_amdgcn_mov_dpp(n0, 0x4E, 0xf, 0xf, true), w1 = __builtin_amdgcn_mov_dpp(n1, 0x4E, 0xf, 0xf, true);
                const unsigned w2 = __builtin_amdgcn_mov_dpp(n2, 0x4E, 0xf, 0xf, true), w3 = __builtin_amdgcn_mov_dpp(n3, 0x4E, 0xf, 0xf, true);
                r0[e] = b1 ? w2 : n0; r2[e] = b1 ? n2 : w0; r1[e] = b1 ? w3 : n1; r3[e] = b1 ? n3 : w1;
            }
            landed(r0, r1, r2, r3);
        }
    };
    unsigned* ctr = (unsigned*)(myws + WS_CTR + (size_t)blockIdx.x * 64);
    unsigned prevv;
    STAMPL(9);
    asm volatile("global_load_dword %0, %1, %2 sc1" : "=v"(prevv) : "v"(0u), "s"(ctr) : "memory");
    if constexpr (IS_W) { loadw(0, gw[0]); loadw(1, gw[1]); } else { loadx(0, 0, gx[0]); loadx(0, 1, gx[0]); }
    STAMPL(10);
    asm volatile("s_waitcnt vmcnt(%1)" : "+v"(prevv) : "n"(IS_W ? 32 : 16) : "memory");
    STAMPL(11);
    const unsigned prev = rfl(prevv), call = prev + 1u, fbase = prev * FSTRIDE;
    lds_st(L, L_CALL, call);
    asm volatile("s_waitcnt lgkmcnt(0)" ::: "memory");
    __builtin_amdgcn_s_barrier();
    FENCE();
    STAMPL(12);
    unsigned char* db = dws + WS_DATA + (size_t)(call & (NPAR - 1u)) * PAR_BYTES + (size_t)rank * SRC_BYTES + (size_t)q * CH;
    unsigned* dflag = (unsigned*)(dws + WS_FLAG + q * FSTR + rank * 4);
#define FMODE 1
    unsigned fm = (d & 1) ? 0x05555555u : 0x0AAAAAAAu;
    if (FMODE == 1 && !self) fm = (fm & 0x0F000000u) | (0x00111111u << (d & 3));
    if (FMODE == 2 && !self) fm = (fm & 0x0F000000u) | (0x00100000u << (d & 3)) | (0x00000808u << d);
    if (FMODE == 3 && !self) fm = (fm & 0x06000000u) | (0x00100000u << (d & 3)) | (0x00000400u << d);
    const unsigned fmask = fm | 0x08000000u | ((IS_W || self) ? 0xA0000000u : 0xF0000000u);
    if constexpr (IS_W) { loadw(2, gw[2]); wait_vm<32>(); STAMPL(13); xformw(gw[0], ow[0]); }
    else { wait_vm<8>(); STAMPL(13); xformx(0, gx[0]); wait_vm<0>(); xformx(1, gx[0]); }
    int ops = IS_W ? 32 : 0, h1 = 0, h2 = 0, h3 = 0, h4 = 0, h5 = 0, h6 = 0;
    int l1 = IS_W ? 16 : 0, l2 = IS_W ? 32 : 0, l3 = 0;
    int ia = 0, ib = 0;
    unsigned fa = 0, fb = 0;
    unsigned tt = 0, period = self ? 0u : per;
    bool wide = self;
    auto flag_try = [&](int ret) {
        if (ia != 0 && ret >= ia) { st4_sys_lane0(dflag, fa, fo); ++ops; ia = ib; fa = fb; ib = 0; }
    };
    auto flag_put = [&](unsigned v) {
        if (ia == 0) { ia = h1; fa = v; } else { ib = h1; fb = v; }
    };
    auto slot = [&](unsigned char* a, v4u& v) {
        const int n0 = wide ? h4 + 0 : h2 + 0, n1 = wide ? h6 + 0 : h3 + 0, n2 = wide ? h6 + 0 : h4 + 0;
        const unsigned due = t0 + (tt >> 3);
        for (;;) {
            const int late = (int)((unsigned)rt() - due);
            const int ret = ops - vm_count();
            flag_try(ret);
            if (late >= 0 && ret >= (late >= LATE2 ? n2 : late >= late1 ? n1 : n0)) break;
            __builtin_amdgcn_s_sleep(1);
        }
        st16_sys(a, so, v);
        h6 = h5; h5 = h4; h4 = h3; h3 = h2; h2 = h1; h1 = ++ops;
        tt += period;
    };
    auto piece = [&](int i, v4u& s0, v4u& s1, v4u& s2, v4u& s3, auto&& loads, auto&& xform) {
        if (i == NBULK + 1 && !self) {
            if (!IS_W || REBASE > 1) t0 += min(max((int)((unsigned)rt() - t0 - ((tt - per) >> 3)) - RB0, 0), RBMAX);
            if (!IS_W) tt += 6u * per - per / 2 + XDELAY;
            period = per / 2; wide = true;
        }
        const unsigned fv = fbase + ((IS_W || self || i < NBULK) ? i + 1 : 2 * i - (NBULK - 2));
        slot(db, s0);
        slot(db + 1024, s1);
        if (XMID && !IS_W && !self && i > NBULK) flag_put(fv - 1u);
        loads();
        slot(db + 2048, s2);
        xform();
        slot(db + 3072, s3);
        if ((fmask >> i) & 1u) flag_put(fv);
        db += ROW_BYTES;
    };
    auto arrived = [&](int l) { while (ops - vm_count() < l) __builtin_amdgcn_s_sleep(1); };
    STAMPL(14);
    if constexpr (IS_W) {
        auto wpiece = [&](int i, v4u (&oc)[4], v4u (&on)[4], unsigned (&gl)[16], unsigned (&gt)[16]) {
            piece(i, oc[0], oc[1], oc[2], oc[3],
                  [&]() { if (i + 3 < NCH) { loadw(i + 3, gl); ops += 16; } l3 = ops; },
                  [&]() { arrived(l1); xformw(gt, on); l1 = l2; l2 = l3; });
        };
#pragma unroll 1
        for (int i = 0; i < NCH; i += 4) {
            wpiece(i, ow[0], ow[1], gw[3], gw[1]);
            wpiece(i + 1, ow[1], ow[0], gw[0], gw[2]);
            wpiece(i + 2, ow[0], ow[1], gw[1], gw[3]);
            wpiece(i + 3, ow[1], ow[0], gw[2], gw[0]);
        }
    } else {
        auto group = [&](int g, v4u (&c)[16], v4u (&n)[16]) {
            const bool more = g + 1 < NCH / 4;
            piece(4 * g, c[0], c[4], c[8], c[12], [&]() { if (more) { loadx(g + 1, 0, n); ops += 8; l1 = ops; } }, [&]() {});
            piece(4 * g + 1, c[1], c[5], c[9], c[13], [&]() { if (more) { loadx(g + 1, 1, n); ops += 8; l2 = ops; } }, [&]() {});
            piece(4 * g + 2, c[2], c[6], c[10], c[14], [&]() {}, [&]() { arrived(l1); xformx(0, n); });
            piece(4 * g + 3, c[3], c[7], c[11], c[15], [&]() {}, [&]() { arrived(l2); xformx(1, n); });
        };
#pragma unroll 1
        for (int g = 0; g < NCH / 4; g += 2) {
            group(g, gx[0], gx[1]);
            const unsigned last = rfl(g == NCH / 4 - 2 && !self ? ~0u : 0u);
#pragma unroll
            for (int a = 0; a < 4; ++a)
#pragma unroll
                for (int c = a + 1; c < 4; ++c) swap16(gx[1][4 * a + c], gx[1][4 * c + a], last);
            group(g + 1, gx[1], gx[0]);
        }
    }
    while (ia != 0) flag_try(ops - vm_count());
    if (lane == 0) __hip_atomic_store(ctr, call, __ATOMIC_RELAXED, __HIP_MEMORY_SCOPE_AGENT);
    STAMP(9);
}

__global__ void __launch_bounds__(NTHREADS) fused_gemm_ag(Params p) {
    __shared__ __attribute__((aligned(1024))) unsigned char lds_raw[LDS_BYTES];
    LAS unsigned char* L = (LAS unsigned char*)lds_raw;
    const int tid = threadIdx.x, lane = tid & 63;
    const int wave = rfl(tid >> 6);
    const int b = blockIdx.x, xcd = b & 7, idx = b >> 3;
    const int ti = (xcd >> 1) * 4 + (idx >> 3), tj = (xcd & 1) * 8 + (idx & 7);
    unsigned xcc;
    asm volatile("s_getreg_b32 %0, hwreg(HW_REG_XCC_ID)" : "=s"(xcc));
    const int q = (idx & 4) * 4 + (xcc & 3) * 4 + (idx & 3);
    int sl = (xcc & 4) + (idx >> 3);
    if (!NOSWAP && ((idx >> 2) & 1) == (xcc & 1)) sl = sl == 1 ? 4 + (idx & 3) : sl == 4 + (idx & 3) ? 1 : sl;
    const unsigned char *px = p.x, *pw = p.w;
    unsigned char *myws = p.myws, *dws = p.dws[sl & 7];
    int rank = p.rank;
    unsigned dtab = p.dtab;
    unsigned tstart = (unsigned)rt();
    asm volatile("" : "+s"(px), "+s"(pw), "+s"(myws), "+s"(dws), "+s"(rank), "+s"(dtab), "+s"(tstart));

    if (wave < 8) {
        if (wave == 0) STAMP(0);
        lds_st(L, L_CTL + wave * 256 + lane * 4, 0u);
        float scale = p.sx[0] * p.sw[0];
        asm volatile("s_waitcnt lgkmcnt(0)" ::: "memory");
        __builtin_amdgcn_s_barrier();
        FENCE();
        scale = __builtin_bit_cast(float, rfl(__builtin_bit_cast(unsigned, scale)));
        const int wr = wave >> 1, wc = wave & 1, i = lane & 31, h = lane >> 5;
        unsigned char* otile = (unsigned char*)(p.out + (size_t)(ti * 256 + wr * 64) * NN + tj * 256 + wc * 128);
        asm volatile("" : "+s"(scale), "+s"(otile));
        const int abase = h * 8192 + (64 * wr + i) * 16;
        const int bbase = 16384 + h * 8192 + (32 * wc + i) * 16;
        v16f acc[2][4];
#pragma unroll
        for (int m = 0; m < 2; ++m)
#pragma unroll
            for (int c = 0; c < 4; ++c)
#pragma unroll
                for (int e = 0; e < 16; ++e) acc[m][c][e] = 0.f;
        unsigned spin = 0;
        for (int s = 0; s < NSB; ++s) {
            for (;;) {
                v2u f = *(volatile LAS v2u*)(L + L_FULL);
                unsigned mn = rfl(f.x < f.y ? f.x : f.y);
                if (mn >= (unsigned)(4 * s + 4)) break;
                if (expired(spin, tstart)) { lds_st(L, L_BAD, 1u); break; }
                __builtin_amdgcn_s_sleep(1);
            }
            FENCE();
            LAS unsigned char* sl = L + (s & (NSLOT - 1)) * SLOT;
            v8i a[2], bf[4];
#pragma unroll
            for (int m = 0; m < 2; ++m) {
                v4i lo = *(LAS v4i*)(sl + abase + m * 512), hi = *(LAS v4i*)(sl + abase + m * 512 + 4096);
                a[m] = __builtin_shufflevector(lo, hi, 0, 1, 2, 3, 4, 5, 6, 7);
            }
#pragma unroll
            for (int c = 0; c < 4; ++c) {
                v4i lo = *(LAS v4i*)(sl + bbase + c * 1024), hi = *(LAS v4i*)(sl + bbase + c * 1024 + 4096);
                bf[c] = __builtin_shufflevector(lo, hi, 0, 1, 2, 3, 4, 5, 6, 7);
                if (c == 3) {
                    asm volatile("s_waitcnt lgkmcnt(0)" : "+v"(bf[3]) :: "memory");
                    lds_st(L, L_PROG + wave * 4, (unsigned)(s + 1 < NSB ? s + 1 : NSB + 7 * wr));
                    FENCE();
                }
#pragma unroll
                for (int m = 0; m < 2; ++m)
                    acc[m][c] = __builtin_amdgcn_mfma_scale_f32_32x32x64_f8f6f4(a[m], bf[c], acc[m][c], 0, 0, 0, 0, 0, 0);
            }
        }
        if (wave == 0) STAMP(1);
        unsigned va;
        {
            const unsigned lb = (unsigned)(uintptr_t)L;
            unsigned vb = lb + (32 * wc + i) * 16, t0, t1;
            unsigned m = 7 * wr, blk = (7 * wr + NXB - 1) % NXB, cnt, tmp, rel;
            va = lb + L_XR + blk * CH + h * 2048 + i * 16;
            u64 now;
#define POLL_ADDR \
            "v_bfe_u32 %[t0], %[va], 11, 1\n\ts_lshl3_add_u32 %[tmp], %[m], %[fin]\n\t" \
            "v_lshl_add_u32 %[t0], %[t0], 2, %[tmp]\n\t"
#define MOVE \
            "s_add_u32 %[blk], %[blk], 1\n\ts_cmp_eq_u32 %[blk], %[nxb]\n\ts_cselect_b32 %[blk], 0, %[blk]\n\t" \
            "s_cselect_b32 %[tmp], %[back], %[fwd]\n\t" \
            "v_and_b32 %[t0], 0xf000, %[t0]\n\tv_and_b32 %[vb], 0x1fff, %[vb]\n\tv_add_u32 %[va], %[tmp], %[va]\n\t" \
            "v_lshl_add_u32 %[vb], %[t0], 1, %[vb]\n\t"
#define RD(r0, r1, adr, off, hi) \
            "ds_read_b128 v[" #r0 "], %[" #adr "] offset:" #off "\n\tds_read_b128 v[" #r1 "], %[" #adr "] offset:" #off "+" #hi "\n\t"
#define RD_A0 RD(136:139, 140:143, va, 0, 1024)
#define RD_A1 RD(144:147, 148:151, va, 512, 1024)
#define RD_B0(off) RD(152:155, 156:159, vb, off, 4096)
#define RD_B1(off) RD(160:163, 164:167, vb, off, 4096)
#define MM_(c, a, b) "v_mfma_f32_32x32x64_f8f6f4 %[" #c "], v[" #a "], v[" #b "], %[" #c "]\n\t"
            asm volatile(
                "s_mov_b32 %[cnt], 7\n"
                "1:\n\t"
                POLL_ADDR
                "ds_read_b32 %[t0], %[t0]\n\ts_waitcnt lgkmcnt(0)\n\t"
                "v_cmp_le_u32 vcc, 0x01010000, %[t0]\n\ts_cmp_eq_u64 vcc, -1\n\ts_cbranch_scc1 3f\n\t"
                "s_memrealtime %[now]\n\ts_waitcnt lgkmcnt(0)\n\t"
                "s_mov_b64 vcc, %[now]\n\ts_sub_u32 %[tmp], vcc_lo, %[ts]\n\t"
                "s_cmp_gt_u32 %[tmp], %[dl]\n\ts_cbranch_scc0 1b\n\t"
                "v_mov_b32 %[t0], %[bad]\n\tv_mov_b32 %[t1], 1\n\tds_write_b32 %[t0], %[t1]\n\t"
                "v_mov_b32 %[t0], %[prog]\n\tv_mov_b32 %[t1], 1000\n\tds_write_b32 %[t0], %[t1]\n\t"
                "s_branch 9f\n"
                "3:\n\t"
                MOVE
                RD_B0(0) RD_A0 RD_A1
                "\n4:\n\t"
                RD_B1(1024)
                "s_add_u32 %[m], %[m], 1\n\ts_add_u32 %[rel], %[m], %[nsb]\n\ts_sub_u32 %[cnt], %[cnt], 1\n\t"
                "s_cmp_eq_u32 %[cnt], 0\n\ts_cselect_b32 %[rel], 1000, %[rel]\n\t"
                "v_mov_b32 %[t0], %[prog]\n\tv_mov_b32 %[t1], %[rel]\n\t"
                "s_waitcnt lgkmcnt(4)\n\t"
                MM_(c00, 136:143, 152:159)
                "s_waitcnt lgkmcnt(2)\n\t"
                MM_(c10, 144:151, 152:159)
                "ds_write_b32 %[t0], %[t1]\n\t"
                RD_B0(2048)
                POLL_ADDR
                "s_waitcnt lgkmcnt(3)\n\t"
                MM_(c01, 136:143, 160:167)
                "ds_read_b32 %[t0], %[t0]\n\t"
                MM_(c11, 144:151, 160:167)
                RD_B1(3072)
                "s_waitcnt lgkmcnt(3)\n\t"
                MM_(c02, 136:143, 152:159)
                "s_waitcnt lgkmcnt(2)\n\t"
                "v_cmp_le_u32 vcc, 0x01010000, %[t0]\n\t"
                MM_(c12, 144:151, 152:159)
                "s_cmp_eq_u32 %[cnt], 0\n\ts_cbranch_scc1 6f\n\t"
                "s_cmp_eq_u64 vcc, -1\n\ts_cbranch_scc0 6f\n\t"
                MOVE
                RD_B0(0)
                "s_waitcnt lgkmcnt(2)\n\t"
                MM_(c03, 136:143, 160:167)
                RD_A0
                MM_(c13, 144:151, 160:167)
                RD_A1
                "s_branch 4b\n"
                "6:\n\t"
                "s_waitcnt lgkmcnt(0)\n\t"
                MM_(c03, 136:143, 160:167)
                MM_(c13, 144:151, 160:167)
                "s_cmp_eq_u32 %[cnt], 0\n\ts_cbranch_scc0 1b\n"
                "9:\n\t"
                "s_waitcnt lgkmcnt(0)\n\ts_nop 15\n\ts_nop 15"
                : [c00] "+v"(acc[0][0]), [c01] "+v"(acc[0][1]), [c02] "+v"(acc[0][2]), [c03] "+v"(acc[0][3]),
                  [c10] "+v"(acc[1][0]), [c11] "+v"(acc[1][1]), [c12] "+v"(acc[1][2]), [c13] "+v"(acc[1][3]),
                  [va] "+v"(va), [vb] "+v"(vb), [t0] "=&v"(t0), [t1] "=&v"(t1),
                  [m] "+s"(m), [blk] "+s"(blk), [cnt] "=&s"(cnt), [tmp] "=&s"(tmp), [rel] "=&s"(rel), [now] "=&s"(now)
                : [fin] "s"(lb + L_FIN), [prog] "s"(lb + L_PROG + wave * 4), [bad] "s"(lb + L_BAD), [ts] "s"(tstart),
                  [dl] "s"(DEADLINE2), [nsb] "n"(NSB), [nxb] "n"(NXB), [fwd] "s"(CH), [back] "n"(-(NXB - 1) * CH)
                : "memory", "vcc", "scc",
                  "v136", "v137", "v138", "v139", "v140", "v141", "v142", "v143", "v144", "v145", "v146", "v147",
                  "v148", "v149", "v150", "v151", "v152", "v153", "v154", "v155", "v156", "v157", "v158", "v159",
                  "v160", "v161", "v162", "v163", "v164", "v165", "v166", "v167");
        }
        if (wc == 0) STAMP(20 + wr);
        const bool bad = lds_ld(L, L_BAD) != 0u;
        const float mul = bad ? __builtin_nanf("") : scale;
        const unsigned oo = (va & 0x800u) * 32u + (va & 0x1f0u);
#pragma unroll
        for (int m = 0; m < 2; ++m)
#pragma unroll
            for (int e = 0; e < 16; ++e) {
                v4f v = {acc[m][0][e] * mul, acc[m][1][e] * mul, acc[m][2][e] * mul, acc[m][3][e] * mul};
                st16_sys(otile + (size_t)(m * 32 + (e & 3) + 8 * (e >> 2)) * NN * 4, oo, __builtin_bit_cast(v4u, v));
                if (OWIN < 32 && wr < 3) wait_vm<OWIN>();
            }
        return;
    }

    __builtin_amdgcn_s_setprio(3);
    if (wave == 10) {
        const int d = (dtab >> (4 * sl)) & 7;
        const int ord = q ^ 16;
        const unsigned per = (sl & 4) ? PERX8 : PER8;
        const unsigned t0 = tstart + (unsigned)LEAD + ord * per / 256;
        const int late1 = ((sl ^ xcc) & 4) ? FARLATE : LATE1;
        if (q < 16) send_panel<false>(px + (size_t)q * 256 * KS, L, myws, dws, rank, q, d, lane, t0, per, late1);
        else send_panel<true>(pw + (q - 16) * 256, L, myws, dws, rank, q, d, lane, t0, per, late1);
        return;
    }

    if (wave == 11) {
        __builtin_amdgcn_s_barrier();
        FENCE();
        const unsigned call = rfl(lds_ld(L, L_CALL));
        const unsigned fbase = (call - 1u) * FSTRIDE;
        const int d = lane & 7;
        const unsigned srcr = (unsigned)(rank + d) & 7u;
        const unsigned foff = WS_FLAG + (lane & 8 ? 16 + tj : ti) * FSTR + ((lane & 8 ? ti : tj) & (FREP - 1)) * FLINE + srcr * 4;
        const unsigned row0 = srcr * NCH;
        const int nmax = lane >= 8 ? 0 : d == 0 ? NCH : NBULK, amax = lane >= 8 || d == 0 ? 0 : 8;
        unsigned it = 0;
        int seen = 0, qn = 0;
        int seen2 = 0, fq = 0, glim = 2;
#define POLL(r) asm volatile("s_mov_b64 exec, %3\n\tglobal_atomic_or " r ", %0, %1, %2 sc0 sc1\n\ts_mov_b64 exec, -1" \
                             :: "v"(foff), "v"(0u), "s"(myws), "s"(wdone ? 0xffull : 0xffffull) : "memory", r)
#define TAKE(v, r, n) asm volatile("s_waitcnt vmcnt(" #n ")\n\tv_mov_b32 %0, " r : "=v"(v) :: "memory")
        bool wdone = false;
        auto process = [&](unsigned v) {
            bool dead = false;
            if ((++it & 15u) == 0u) {
                dead = rfl(__hip_atomic_load((const unsigned*)(myws + WS_TMO), __ATOMIC_RELAXED, __HIP_MEMORY_SCOPE_AGENT)) == call;
                if (!dead && (unsigned)rt() - tstart > DEADLINE) {
                    if (lane == 0) __hip_atomic_store((unsigned*)(myws + WS_TMO), call, __ATOMIC_RELAXED, __HIP_MEMORY_SCOPE_AGENT);
                    dead = true;
                }
            }
            int nx = (int)(v - fbase), nw = dpp0<0x128>(nx);
            if (wdone) nw = NCH;
            if (dead) { nx = nw = 64; lds_st(L, L_BAD, 1u); }
            int nn = min(min(nx, nw), nmax) - seen;
            nn = max(nn, 0);
            if (__ballot(nn > 0)) {
                int pre = nn;
                pre += dpp0<0x111>(pre); pre += dpp0<0x112>(pre); pre += dpp0<0x114>(pre);
                const int pos = qn + pre - nn;
                FENCE();
                for (int j = 0; j < nn; ++j) lds_st(L, L_QUEUE + (pos + j) * 4, row0 + seen + j + 1u);
                seen += nn;
                qn += __builtin_amdgcn_readlane(pre, 7);
                it = 0;
            }
            int a = min(nx - NBULK, amax);
            if (nw < NCH) a = min(a, (nw - NBULK) >> 1);
            for (;;) {
                const int n2 = max(min(a, glim) - seen2, 0);
                if (!__ballot(n2 > 0)) break;
                int pre = n2;
                pre += dpp0<0x111>(pre); pre += dpp0<0x112>(pre); pre += dpp0<0x114>(pre);
                const int pos = fq + pre - n2;
                FENCE();
                for (int j = 0; j < n2; ++j) {
                    const unsigned pr = (seen2 + j) & 1;
                    *(volatile LAS unsigned short*)(L + L_FIN + (pos + j) * 4) =
                        (unsigned short)((2 * d - 2 + pr) << 12 | (row0 + NBULK + ((seen2 + j) >> 1)));
                }
                seen2 += n2;
                fq += __builtin_amdgcn_readlane(pre, 7);
                it = 0;
                if (fq != glim * (NR - 1)) break;
                STAMP(31 + glim / 2);
                glim += 2;
            }
            wdone = qn >= NQ && fq >= NPAIR;
            return qn >= NQ && fq >= NFIN;
        };
        unsigned v;
        bool end = false;
        while (!end && (PD == 1 || qn < NQ - PFAST)) {
            POLL("v160"); TAKE(v, "v160", 0);
            end = process(v);
            if (qn >= NQ - PFAST) __builtin_amdgcn_s_sleep(POLLSLEEP2); else __builtin_amdgcn_s_sleep(POLLSLEEP);
        }
        STAMP(10);
        return;
    }

    {
        const int ld = wave - 8;
        __builtin_amdgcn_s_barrier();
        FENCE();
        const unsigned voff = lane * 16;
        const unsigned char* dbase = myws + WS_DATA + (size_t)(rfl(lds_ld(L, L_CALL)) & (NPAR - 1u)) * PAR_BYTES + (size_t)(ld ? 16 + tj : ti) * CH;
        const unsigned lb = (unsigned)(uintptr_t)L;
        int iss = 0, done = 0;
        unsigned spin = 0;
        while (done < NQ) {
            const unsigned v = lds_ld(L, lane < 16 ? L_PROG + (lane & 7) * 4 : L_QUEUE + (iss + (lane & 15)) * 4);
            int lim = iss + __builtin_ctzll(~(__ballot(v != 0u) >> 16));
            lim = min(min(lim, iss + 16), min(NQ, done + 15));
            const int st = iss >> 2;
            int fr = st + NSLOT;
#pragma unroll
            for (int j = NSLOT - 1; j >= 0; --j) if (__ballot(lane < 8 && (int)v < st + j - (NSLOT - 1))) fr = st + j;
            lim = min(lim, fr * 4);
            FENCE();
            const bool any = iss < lim;
            for (const int i0 = iss; iss < lim; ++iss) {
                const unsigned e = __builtin_amdgcn_readlane(v, 16 + iss - i0) - 1u;
                dma4k(dbase + (size_t)e * ROW_BYTES, lb + ld * 16384 + ((iss >> 2) & (NSLOT - 1)) * SLOT + (iss & 3) * CH, voff);
            }
            const int tg = iss - ((vm_count() + 3) >> 2);
            if (tg > done) {
                FENCE();
                done = tg;
                lds_st(L, L_FULL + ld * 4, (unsigned)done);
            } else if (!any) {
                if (expired(spin, tstart)) { lds_st(L, L_BAD, 1u); break; }
                __builtin_amdgcn_s_sleep(1);
            }
        }
        auto fin = [&](auto isw) {
            constexpr bool W = decltype(isw)::value;
            constexpr int NF = W ? NPAIR : NFIN;
            unsigned xa = lb + L_XR;
            iss = 0; done = 0;
            while (done < NF) {
                const unsigned v = lds_ld(L, lane < 16 ? L_PROG + (lane & 7) * 4 : L_FIN + (iss + (lane & 15)) * 4);
                int n = __builtin_ctz(~(unsigned)(__ballot((v & 0xfffu) != 0u) >> 16) | (1u << (W ? WBATCH : XBATCH)));
                int pm = (int)v;
                pm = min(pm, dpp0<0x121>(pm)); pm = min(pm, dpp0<0x122>(pm)); pm = min(pm, dpp0<0x124>(pm));
                pm = (int)rfl(pm) - NSB;
                int lim = W ? done + 7 : 2 * (pm + NXB);
                lim = lim < NF ? lim : NF;
                lim = pm < 0 ? 0 : lim - iss;
                n = n < lim ? n : lim;
                const unsigned off = (v & 0x1ffu) << 17 | (W ? (v & 0x1000u) << 6 : (v & 0x1000u) >> 1);
                FENCE();
                for (int j = 0; j < n; ++j) {
                    const unsigned vo = voff + __builtin_amdgcn_readlane(off, 16 + j);
                    if constexpr (W) {
                        const unsigned la = lb + (__builtin_amdgcn_readlane(v, 16 + j) & 0xf000u) * 2;
                        dma4k(dbase, la, vo);
                        dma4k(dbase + ROW_BYTES, la + CH, vo);
                    } else {
                        dma2k(dbase, xa, vo);
                        xa = xa == lb + L_CTL - 2048 ? lb + L_XR : xa + 2048;
                    }
                }
                if (n > 0) iss += n;
                const int vm = vm_count();
                const int tg = iss - (W ? (vm + 7) >> 3 : (vm + 1) >> 1);
                if (tg > done) {
                    FENCE();
                    const int last = tg == NF ? NFIN : tg;
                    if (done + lane < last) *(volatile LAS unsigned char*)(L + L_FIN + (done + lane) * 4 + 2 + W) = 1;
                    done = tg;
                } else if (n <= 0) {
                    if (expired(spin, tstart)) { lds_st(L, L_BAD, 1u); break; }
                    __builtin_amdgcn_s_sleep(1);
                }
            }
        };
        if (ld) fin(std::true_type{}); else fin(std::false_type{});
        STAMP(11 + 2 * ld);
    }
}

}

extern "C" void kernel_launch(int rank, int nranks, ncclComm_t comm, hipStream_t stream,
                              void* const* d_in, const int* in_sizes, int n_in,
                              void* d_out, int out_size, const int* in_dtypes, int out_dtype,
                              void* const* d_out_all, void* const* d_ws_all, size_t ws_bytes) {
    if (nranks != NR || n_in != 4 || ws_bytes < WS_NEED || in_sizes[0] != MM * KS || in_sizes[1] != KS * NN ||
        out_size != MM * NN || (unsigned)rank >= (unsigned)NR) {
        fprintf(stderr, "kernel_launch: unexpected problem geometry\n");
        return;
    }
    Params p;
    p.x = (const unsigned char*)d_in[0]; p.w = (const unsigned char*)d_in[1];
    p.sx = (const float*)d_in[2]; p.sw = (const float*)d_in[3];
    p.out = (float*)d_out; p.rank = rank;
    p.myws = (unsigned char*)d_ws_all[rank];
    const unsigned dtab[NR] = {0x14735602u, 0x73164502u, 0x13654207u, 0x74623501u, 0x16245307u, 0x75234601u, 0x15724306u, 0x75143206u};
    p.dtab = dtab[rank];
    for (int s = 0; s < NR; ++s) p.dws[s] = (unsigned char*)d_ws_all[(rank + NR - ((p.dtab >> (4 * s)) & 7)) & 7];
    hipLaunchKernelGGL(fused_gemm_ag, dim3(NWG), dim3(NTHREADS), 0, stream, p);
    HIP_CHECK(hipGetLastError());
}
```

```cpp
#include <hip/hip_runtime.h>
#include <rccl/rccl.h>
#include <cstdio>
#include <cstdint>
#include <type_traits>

#define HIP_CHECK(expr) do { hipError_t _e = (expr); if (_e != hipSuccess) { \
    fprintf(stderr, "HIP error %s at %s:%d\n", hipGetErrorString(_e), __FILE__, __LINE__); \
    return; } } while (0)

namespace {

#define LAS __attribute__((address_space(3)))
typedef int v4i __attribute__((ext_vector_type(4)));
typedef int v8i __attribute__((ext_vector_type(8)));
typedef unsigned v2u __attribute__((ext_vector_type(2)));
typedef unsigned v4u __attribute__((ext_vector_type(4)));
typedef float v4f __attribute__((ext_vector_type(4)));
typedef float v16f __attribute__((ext_vector_type(16)));
typedef unsigned long long u64;

constexpr int NR = 8;
constexpr int MM = 4096, NN = 4096, KS = 512;
constexpr int NCH = 32;
constexpr int NBULK = 28;
constexpr int NPANEL = 32;
constexpr int CH = 4096;
constexpr int NWG = 256, NTHREADS = 768;
constexpr int NQ = (NR - 1) * NBULK + NCH;
constexpr int NSB = NQ / 4;
constexpr int NPAIR = 2 * (NR - 1);
constexpr int NFIN = 4 * NPAIR;
constexpr unsigned FSTRIDE = 64;
constexpr size_t WS_CTR = 0;
#define FREP 1
#define FLINE 128
#define FSTR (FLINE * FREP)
constexpr size_t WS_FLAG = 524288;
constexpr size_t WS_TMO = 16384 + 4096;
constexpr size_t WS_DBG = 32768;
constexpr size_t WS_DATA = 1u << 20;
constexpr size_t ROW_BYTES = (size_t)NPANEL * CH;
constexpr size_t SRC_BYTES = (size_t)NCH * ROW_BYTES;
constexpr size_t PAR_BYTES = (size_t)NR * SRC_BYTES;
#define NPAR 2
constexpr size_t WS_NEED = WS_DATA + NPAR * PAR_BYTES;
constexpr unsigned DEADLINE = 40000000u;
constexpr unsigned DEADLINE2 = 60000000u;
constexpr int NSLOT = 4, SLOT = 32768;
constexpr int L_XR = NPAIR * 2 * CH;
constexpr int NXB = 11;
constexpr int L_CTL = L_XR + NXB * CH;
constexpr int L_FULL = L_CTL;
constexpr int L_PROG = L_CTL + 64;
constexpr int L_BAD = L_CTL + 128;
constexpr int L_QUEUE = L_CTL + 256;
constexpr int L_FIN = L_CTL + 1536;
constexpr int L_CALL = L_CTL + 2048;
constexpr int LDS_BYTES = L_CALL + 64 + 256;
static_assert(L_QUEUE + (NQ + 32) * 4 <= L_FIN && L_FIN + (NFIN + 32) * 4 <= L_CALL && L_CTL >= NSLOT * SLOT, "LDS map");

#define PER8 442
#define PERX8 444
#define LEAD 50
#define POLLSLEEP 2
#define POLLSLEEP2 POLLSLEEP
#define PFAST 14
#define PD 1
#define PGAP 16
#define REBASE 2
#define RB0 10
#define XDELAY 200
#define FARLATE 0
#define RBMAX 100000
#define NOSWAP 0
#define TOUCH 0
#define TDELAY 100
#define TMASK 31
#define TWAVE 11
#define TOP "global_load_dword v161, %0, %1 sc0 sc1"
#define SSCALE 1
#define CATCH 10
#define TPER per
#define SCTR 0
#define STOUCH 0
#define LATE1 100000
#define LATE2 100000
#define XBATCH 8
#define WBATCH 4
#define OWIN 63
#define XMID 1

struct Params {
    const unsigned char* x; const unsigned char* w; unsigned char* myws;
    int rank;
    unsigned dtab;
    const float* sx; const float* sw; float* out;
    unsigned char* dws[NR];
};

__device__ __forceinline__ u64 rt() { return __builtin_amdgcn_s_memrealtime(); }
__device__ __forceinline__ unsigned rfl(unsigned v) { return __builtin_amdgcn_readfirstlane(v); }
__device__ __forceinline__ unsigned lds_ld(LAS unsigned char* L, int off) { return *(volatile LAS unsigned*)(L + off); }
__device__ __forceinline__ void lds_st(LAS unsigned char* L, int off, unsigned v) { *(volatile LAS unsigned*)(L + off) = v; }
#define FENCE() asm volatile("" ::: "memory")
template <int N> __device__ __forceinline__ void wait_vm() { asm volatile("s_waitcnt vmcnt(%0)" :: "n"(N) : "memory"); }
__device__ __forceinline__ int vm_count() {
    unsigned r;
    asm volatile("s_getreg_b32 %0, hwreg(HW_REG_IB_STS)" : "=s"(r) :: "memory");
    return (int)((r & 15u) | ((r >> 18) & 48u));
}
__device__ __forceinline__ bool expired(unsigned& spin, unsigned t0) {
    return (++spin & 4095u) == 0u && (unsigned)rt() - t0 > DEADLINE2;
}
template <int CTRL> __device__ __forceinline__ int dpp0(int v) { return __builtin_amdgcn_update_dpp(0, v, CTRL, 0xf, 0xf, true); }

#define STAMPL(i) do {} while (0)
#define STAMP(i) do {} while (0)

__device__ __forceinline__ void dma4k(const unsigned char* g, unsigned ldsaddr, unsigned voff) {
    asm volatile("s_mov_b32 m0, %2\n\ts_nop 0\n\t"
                 "global_load_lds_dwordx4 %0, %1 offset:0\n\tglobal_load_lds_dwordx4 %0, %1 offset:1024\n\t"
                 "global_load_lds_dwordx4 %0, %1 offset:2048\n\tglobal_load_lds_dwordx4 %0, %1 offset:3072"
                 :: "v"(voff), "s"(g), "s"(ldsaddr) : "memory");
}
__device__ __forceinline__ void dma2k(const unsigned char* g, unsigned ldsaddr, unsigned voff) {
    asm volatile("s_mov_b32 m0, %2\n\ts_nop 0\n\t"
                 "global_load_lds_dwordx4 %0, %1 offset:0\n\tglobal_load_lds_dwordx4 %0, %1 offset:1024"
                 :: "v"(voff), "s"(g), "s"(ldsaddr) : "memory");
}
__device__ __forceinline__ void st16_sys(unsigned char* sbase, unsigned voff, v4u v) {
    asm volatile("global_store_dwordx4 %0, %1, %2 sc0 sc1\n\ts_nop 1" :: "v"(voff), "v"(v), "s"(sbase) : "memory");
}
__device__ __forceinline__ void st4_sys_lane0(unsigned* p, unsigned v, unsigned voff) {
#define FOP "global_atomic_swap %0, %1, %2 sc1"
    asm volatile("s_mov_b64 exec, %3\n\t" FOP "\n\ts_mov_b64 exec, -1"
                 :: "v"(voff), "v"(v), "s"(p), "n"((1 << FREP) - 1) : "memory");
}
__device__ __forceinline__ void ld4(unsigned& d, unsigned voff, const unsigned char* sbase) {
    asm volatile("global_load_dword %0, %1, %2" : "=v"(d) : "v"(voff), "s"(sbase) : "memory");
}
template <int OFF> __device__ __forceinline__ void ld16(v4u& d, unsigned voff, const unsigned char* sbase) {
    asm volatile("global_load_dwordx4 %0, %1, %2 offset:%3" : "=v"(d) : "v"(voff), "s"(sbase), "n"(OFF) : "memory");
}
__device__ __forceinline__ void landed(unsigned (&g)[16]) {
    asm volatile("" : "+v"(g[0]), "+v"(g[1]), "+v"(g[2]), "+v"(g[3]), "+v"(g[4]), "+v"(g[5]), "+v"(g[6]), "+v"(g[7]),
                      "+v"(g[8]), "+v"(g[9]), "+v"(g[10]), "+v"(g[11]), "+v"(g[12]), "+v"(g[13]), "+v"(g[14]), "+v"(g[15]) :: "memory");
}
__device__ __forceinline__ void landed(v4u& a, v4u& b, v4u& c, v4u& d) {
    asm volatile("" : "+v"(a), "+v"(b), "+v"(c), "+v"(d) :: "memory");
}
__device__ __forceinline__ void swap16(v4u& a, v4u& b, unsigned mask) {
    unsigned x0 = a[0], x1 = a[1], x2 = a[2], x3 = a[3], y0 = b[0], y1 = b[1], y2 = b[2], y3 = b[3];
    asm volatile("s_mov_b32 exec_lo, %8\n\ts_mov_b32 exec_hi, %8\n\tv_swap_b32 %0, %4\n\tv_swap_b32 %1, %5\n\tv_swap_b32 %2, %6\n\tv_swap_b32 %3, %7\n\t"
                 "s_mov_b64 exec, -1"
                 : "+v"(x0), "+v"(x1), "+v"(x2), "+v"(x3), "+v"(y0), "+v"(y1), "+v"(y2), "+v"(y3) : "s"(mask));
    a[0] = x0; a[1] = x1; a[2] = x2; a[3] = x3; b[0] = y0; b[1] = y1; b[2] = y2; b[3] = y3;
}

template <bool IS_W>
__device__ __forceinline__ void send_panel(const unsigned char* src, LAS unsigned char* L, unsigned char* myws, unsigned char* dws,
                                           int rank, int q, int d, int lane, unsigned t0, unsigned per, int late1) {
    const bool self = d == 0;
    const unsigned lo = IS_W ? lane * 4 : (lane >> 2) * 4 * KS + (lane & 3) * 16;
    const unsigned so = lane * 16, fo = lane * FLINE;
    unsigned gw[4][16];
    v4u ow[2][4];
    v4u gx[2][16];
    auto loadw = [&](int i, unsigned (&g)[16]) {
#pragma unroll
        for (int k = 0; k < 16; ++k) ld4(g[k], lo, src + (size_t)(i * 16 + k) * NN);
    };
    auto xformw = [&](unsigned (&g)[16], v4u (&ob)[4]) {
        landed(g);
#pragma unroll
        for (int t = 0; t < 4; ++t) {
            const unsigned a0 = g[4 * t], a1 = g[4 * t + 1], a2 = g[4 * t + 2], a3 = g[4 * t + 3];
            const unsigned x0 = __builtin_amdgcn_perm(a1, a0, 0x05010400u), x1 = __builtin_amdgcn_perm(a1, a0, 0x07030602u);
            const unsigned y0 = __builtin_amdgcn_perm(a3, a2, 0x05010400u), y1 = __builtin_amdgcn_perm(a3, a2, 0x07030602u);
            ob[0][t] = __builtin_amdgcn_perm(y0, x0, 0x05040100u);
            ob[1][t] = __builtin_amdgcn_perm(y0, x0, 0x07060302u);
            ob[2][t] = __builtin_amdgcn_perm(y1, x1, 0x05040100u);
            ob[3][t] = __builtin_amdgcn_perm(y1, x1, 0x07060302u);
        }
        landed(ob[0], ob[1], ob[2], ob[3]);
    };
    auto loadx = [&](int grp, int half, v4u (&g)[16]) {
#pragma unroll
        for (int j = 2 * half; j < 2 * half + 2; ++j) {
            const unsigned char* s = src + (size_t)j * 64 * KS + grp * 64;
            ld16<0>(g[4 * j], lo, s); ld16<KS>(g[4 * j + 1], lo, s); ld16<2 * KS>(g[4 * j + 2], lo, s); ld16<3 * KS>(g[4 * j + 3], lo, s);
        }
    };
    const bool b0 = lane & 1, b1 = lane & 2;
    auto xformx = [&](int half, v4u (&g)[16]) {
#pragma unroll
        for (int j = 2 * half; j < 2 * half + 2; ++j) {
            v4u &r0 = g[4 * j], &r1 = g[4 * j + 1], &r2 = g[4 * j + 2], &r3 = g[4 * j + 3];
            landed(r0, r1, r2, r3);
#pragma unroll
            for (int e = 0; e < 4; ++e) {
                const unsigned a0 = r0[e], a1 = r1[e], a2 = r2[e], a3 = r3[e];
                const unsigned u0 = __builtin_amdgcn_mov_dpp(a0, 0xB1, 0xf, 0xf, true), u1 = __builtin_amdgcn_mov_dpp(a1, 0xB1, 0xf, 0xf, true);
                const unsigned u2 = __builtin_amdgcn_mov_dpp(a2, 0xB1, 0xf, 0xf, true), u3 = __builtin_amdgcn_mov_dpp(a3, 0xB1, 0xf, 0xf, true);
                const unsigned n0 = b0 ? u1 : a0, n1 = b0 ? a1 : u0, n2 = b0 ? u3 : a2, n3 = b0 ? a3 : u2;
                const unsigned w0 = __builtin_amdgcn_mov_dpp(n0, 0x4E, 0xf, 0xf, true), w1 = __builtin_amdgcn_mov_dpp(n1, 0x4E, 0xf, 0xf, true);
                const unsigned w2 = __builtin_amdgcn_mov_dpp(n2, 0x4E, 0xf, 0xf, true), w3 = __builtin_amdgcn_mov_dpp(n3, 0x4E, 0xf, 0xf, true);
                r0[e] = b1 ? w2 : n0; r2[e] = b1 ? n2 : w0; r1[e] = b1 ? w3 : n1; r3[e] = b1 ? n3 : w1;
            }
            landed(r0, r1, r2, r3);
        }
    };
    unsigned* ctr = (unsigned*)(myws + WS_CTR + (size_t)blockIdx.x * 64);
    unsigned prevv;
    STAMPL(9);
    asm volatile("global_load_dword %0, %1, %2 sc1" : "=v"(prevv) : "v"(0u), "s"(ctr) : "memory");
    if constexpr (IS_W) { loadw(0, gw[0]); loadw(1, gw[1]); } else { loadx(0, 0, gx[0]); loadx(0, 1, gx[0]); }
    STAMPL(10);
    asm volatile("s_waitcnt vmcnt(%1)" : "+v"(prevv) : "n"(IS_W ? 32 : 16) : "memory");
    STAMPL(11);
    const unsigned prev = rfl(prevv), call = prev + 1u, fbase = prev * FSTRIDE;
    lds_st(L, L_CALL, call);
    asm volatile("s_waitcnt lgkmcnt(0)" ::: "memory");
    __builtin_amdgcn_s_barrier();
    FENCE();
    STAMPL(12);
    unsigned char* db = dws + WS_DATA + (size_t)(call & (NPAR - 1u)) * PAR_BYTES + (size_t)rank * SRC_BYTES + (size_t)q * CH;
    unsigned* dflag = (unsigned*)(dws + WS_FLAG + q * FSTR + rank * 4);
#define FMODE 1
    unsigned fm = (d & 1) ? 0x05555555u : 0x0AAAAAAAu;
    if (FMODE == 1 && !self) fm = (fm & 0x0F000000u) | (0x00111111u << (d & 3));
    if (FMODE == 2 && !self) fm = (fm & 0x0F000000u) | (0x00100000u << (d & 3)) | (0x00000808u << d);
    if (FMODE == 3 && !self) fm = (fm & 0x06000000u) | (0x00100000u << (d & 3)) | (0x00000400u << d);
    const unsigned fmask = fm | 0x08000000u | ((IS_W || self) ? 0xA0000000u : 0xF0000000u);
    if constexpr (IS_W) { loadw(2, gw[2]); wait_vm<32>(); STAMPL(13); xformw(gw[0], ow[0]); }
    else { wait_vm<8>(); STAMPL(13); xformx(0, gx[0]); wait_vm<0>(); xformx(1, gx[0]); }
    int ops = IS_W ? 32 : 0, h1 = 0, h2 = 0, h3 = 0, h4 = 0, h5 = 0, h6 = 0;
    int l1 = IS_W ? 16 : 0, l2 = IS_W ? 32 : 0, l3 = 0;
    int ia = 0, ib = 0;
    unsigned fa = 0, fb = 0;
    unsigned tt = 0, period = self ? 0u : per;
    bool wide = self;
    int slip = 0;
    auto flag_try = [&](int ret) {
        if (ia != 0 && ret >= ia) { st4_sys_lane0(dflag, fa, fo); ++ops; ia = ib; fa = fb; ib = 0; }
    };
    auto flag_put = [&](unsigned v) {
        if (ia == 0) { ia = h1; fa = v; } else { ib = h1; fb = v; }
    };
    auto slot = [&](unsigned char* a, v4u& v) {
        const int n0 = wide ? h4 + 0 : h2 + 0, n1 = wide ? h6 + 0 : h3 + 0, n2 = wide ? h6 + 0 : h4 + 0;
        const unsigned due = t0 + (tt >> 3);
        int late;
        for (;;) {
            late = (int)((unsigned)rt() - due);
            const int ret = ops - vm_count();
            flag_try(ret);
            if (late >= slip && ret >= (late >= LATE2 ? n2 : late >= late1 ? n1 : n0)) break;
            __builtin_amdgcn_s_sleep(1);
        }
        if (CATCH < 1000) slip = wide ? 0 : max(late - CATCH, 0);
        st16_sys(a, so, v);
        h6 = h5; h5 = h4; h4 = h3; h3 = h2; h2 = h1; h1 = ++ops;
        tt += period;
    };
    auto piece = [&](int i, v4u& s0, v4u& s1, v4u& s2, v4u& s3, auto&& loads, auto&& xform) {
        if (i == NBULK + 1 && !self) {
            if (!IS_W || REBASE > 1) t0 += min(max((int)((unsigned)rt() - t0 - ((tt - per) >> 3)) - RB0, 0), RBMAX);
            if (!IS_W) tt += 6u * TPER - per / 2 + XDELAY;
            period = TPER / 2; wide = true; slip = 0;
        }
        const unsigned fv = fbase + ((IS_W || self || i < NBULK) ? i + 1 : 2 * i - (NBULK - 2));
        slot(db, s0);
        slot(db + 1024, s1);
        if (XMID && !IS_W && !self && i > NBULK) flag_put(fv - 1u);
        loads();
        slot(db + 2048, s2);
        xform();
        slot(db + 3072, s3);
        if ((fmask >> i) & 1u) flag_put(fv);
        db += ROW_BYTES;
    };
    auto arrived = [&](int l) { while (ops - vm_count() < l) __builtin_amdgcn_s_sleep(1); };
    STAMPL(14);
    if constexpr (IS_W) {
        auto wpiece = [&](int i, v4u (&oc)[4], v4u (&on)[4], unsigned (&gl)[16], unsigned (&gt)[16]) {
            piece(i, oc[0], oc[1], oc[2], oc[3],
                  [&]() { if (i + 3 < NCH) { loadw(i + 3, gl); ops += 16; } l3 = ops; },
                  [&]() { arrived(l1); xformw(gt, on); l1 = l2; l2 = l3; });
        };
#pragma unroll 1
        for (int i = 0; i < NCH; i += 4) {
            wpiece(i, ow[0], ow[1], gw[3], gw[1]);
            wpiece(i + 1, ow[1], ow[0], gw[0], gw[2]);
            wpiece(i + 2, ow[0], ow[1], gw[1], gw[3]);
            wpiece(i + 3, ow[1], ow[0], gw[2], gw[0]);
        }
    } else {
        auto group = [&](int g, v4u (&c)[16], v4u (&n)[16]) {
            const bool more = g + 1 < NCH / 4;
            piece(4 * g, c[0], c[4], c[8], c[12], [&]() { if (more) { loadx(g + 1, 0, n); ops += 8; l1 = ops; } }, [&]() {});
            piece(4 * g + 1, c[1], c[5], c[9], c[13], [&]() { if (more) { loadx(g + 1, 1, n); ops += 8; l2 = ops; } }, [&]() {});
            piece(4 * g + 2, c[2], c[6], c[10], c[14], [&]() {}, [&]() { arrived(l1); xformx(0, n); });
            piece(4 * g + 3, c[3], c[7], c[11], c[15], [&]() {}, [&]() { arrived(l2); xformx(1, n); });
        };
#pragma unroll 1
        for (int g = 0; g < NCH / 4; g += 2) {
            group(g, gx[0], gx[1]);
            const unsigned last = rfl(g == NCH / 4 - 2 && !self ? ~0u : 0u);
#pragma unroll
            for (int a = 0; a < 4; ++a)
#pragma unroll
                for (int c = a + 1; c < 4; ++c) swap16(gx[1][4 * a + c], gx[1][4 * c + a], last);
            group(g + 1, gx[1], gx[0]);
        }
    }
    while (ia != 0) flag_try(ops - vm_count());
    if (lane == 0) __hip_atomic_store(ctr, call, __ATOMIC_RELAXED, __HIP_MEMORY_SCOPE_AGENT);
    STAMP(9);
}

__global__ void __launch_bounds__(NTHREADS) fused_gemm_ag(Params p) {
    __shared__ __attribute__((aligned(1024))) unsigned char lds_raw[LDS_BYTES];
    LAS unsigned char* L = (LAS unsigned char*)lds_raw;
    const int tid = threadIdx.x, lane = tid & 63;
    const int wave = rfl(tid >> 6);
    const int b = blockIdx.x, xcd = b & 7, idx = b >> 3;
    const int ti = (xcd >> 1) * 4 + (idx >> 3), tj = (xcd & 1) * 8 + (idx & 7);
    unsigned xcc;
    asm volatile("s_getreg_b32 %0, hwreg(HW_REG_XCC_ID)" : "=s"(xcc));
    const int q = (idx & 4) * 4 + (xcc & 3) * 4 + (idx & 3);
    int sl = (xcc & 4) + (idx >> 3);
    if (!NOSWAP && ((idx >> 2) & 1) == (xcc & 1)) sl = sl == 1 ? 4 + (idx & 3) : sl == 4 + (idx & 3) ? 1 : sl;
    const unsigned char *px = p.x, *pw = p.w;
    unsigned char *myws = p.myws, *dws = p.dws[sl & 7];
    int rank = p.rank;
    unsigned dtab = p.dtab;
    unsigned tstart = (unsigned)rt();
    asm volatile("" : "+s"(px), "+s"(pw), "+s"(myws), "+s"(dws), "+s"(rank), "+s"(dtab), "+s"(tstart));

    if (wave < 8) {
        if (wave == 0) STAMP(0);
        lds_st(L, L_CTL + wave * 256 + lane * 4, 0u);
        asm volatile("s_waitcnt lgkmcnt(0)" ::: "memory");
        __builtin_amdgcn_s_barrier();
        FENCE();
        float scale, scale2;
        asm volatile("s_load_dword %0, %2, 0x0\n\ts_load_dword %1, %3, 0x0\n\ts_waitcnt lgkmcnt(0)"
                     : "=&s"(scale), "=&s"(scale2) : "s"(p.sx), "s"(p.sw) : "memory");
        scale *= scale2;
        scale = __builtin_bit_cast(float, rfl(__builtin_bit_cast(unsigned, scale)));
        const int wr = wave >> 1, wc = wave & 1, i = lane & 31, h = lane >> 5;
        unsigned char* otile = (unsigned char*)(p.out + (size_t)(ti * 256 + wr * 64) * NN + tj * 256 + wc * 128);
        asm volatile("" : "+s"(scale), "+s"(otile));
        const int abase = h * 8192 + (64 * wr + i) * 16;
        const int bbase = 16384 + h * 8192 + (32 * wc + i) * 16;
        v16f acc[2][4];
#pragma unroll
        for (int m = 0; m < 2; ++m)
#pragma unroll
            for (int c = 0; c < 4; ++c)
#pragma unroll
                for (int e = 0; e < 16; ++e) acc[m][c][e] = 0.f;
        unsigned spin = 0;
        for (int s = 0; s < NSB; ++s) {
            for (;;) {
                v2u f = *(volatile LAS v2u*)(L + L_FULL);
                unsigned mn = rfl(f.x < f.y ? f.x : f.y);
                if (mn >= (unsigned)(4 * s + 4)) break;
                if (expired(spin, tstart)) { lds_st(L, L_BAD, 1u); break; }
                __builtin_amdgcn_s_sleep(1);
            }
            FENCE();
            LAS unsigned char* sl = L + (s & (NSLOT - 1)) * SLOT;
            v8i a[2], bf[4];
#pragma unroll
            for (int m = 0; m < 2; ++m) {
                v4i lo = *(LAS v4i*)(sl + abase + m * 512), hi = *(LAS v4i*)(sl + abase + m * 512 + 4096);
                a[m] = __builtin_shufflevector(lo, hi, 0, 1, 2, 3, 4, 5, 6, 7);
            }
#pragma unroll
            for (int c = 0; c < 4; ++c) {
                v4i lo = *(LAS v4i*)(sl + bbase + c * 1024), hi = *(LAS v4i*)(sl + bbase + c * 1024 + 4096);
                bf[c] = __builtin_shufflevector(lo, hi, 0, 1, 2, 3, 4, 5, 6, 7);
                if (c == 3) {
                    asm volatile("s_waitcnt lgkmcnt(0)" : "+v"(bf[3]) :: "memory");
                    lds_st(L, L_PROG + wave * 4, (unsigned)(s + 1 < NSB ? s + 1 : NSB + 7 * wr));
                    FENCE();
                }
#pragma unroll
                for (int m = 0; m < 2; ++m)
                    acc[m][c] = __builtin_amdgcn_mfma_scale_f32_32x32x64_f8f6f4(a[m], bf[c], acc[m][c], 0, 0, 0, 0, 0, 0);
            }
        }
        if (wave == 0) STAMP(1);
        unsigned va;
        {
            const unsigned lb = (unsigned)(uintptr_t)L;
            unsigned vb = lb + (32 * wc + i) * 16, t0, t1;
            unsigned m = 7 * wr, blk = (7 * wr + NXB - 1) % NXB, cnt, tmp, rel;
            va = lb + L_XR + blk * CH + h * 2048 + i * 16;
            u64 now;
#define POLL_ADDR \
            "v_bfe_u32 %[t0], %[va], 11, 1\n\ts_lshl3_add_u32 %[tmp], %[m], %[fin]\n\t" \
            "v_lshl_add_u32 %[t0], %[t0], 2, %[tmp]\n\t"
#define MOVE \
            "s_add_u32 %[blk], %[blk], 1\n\ts_cmp_eq_u32 %[blk], %[nxb]\n\ts_cselect_b32 %[blk], 0, %[blk]\n\t" \
            "s_cselect_b32 %[tmp], %[back], %[fwd]\n\t" \
            "v_and_b32 %[t0], 0xf000, %[t0]\n\tv_and_b32 %[vb], 0x1fff, %[vb]\n\tv_add_u32 %[va], %[tmp], %[va]\n\t" \
            "v_lshl_add_u32 %[vb], %[t0], 1, %[vb]\n\t"
#define RD(r0, r1, adr, off, hi) \
            "ds_read_b128 v[" #r0 "], %[" #adr "] offset:" #off "\n\tds_read_b128 v[" #r1 "], %[" #adr "] offset:" #off "+" #hi "\n\t"
#define RD_A0 RD(136:139, 140:143, va, 0, 1024)
#define RD_A1 RD(144:147, 148:151, va, 512, 1024)
#define RD_B0(off) RD(152:155, 156:159, vb, off, 4096)
#define RD_B1(off) RD(160:163, 164:167, vb, off, 4096)
#define MM_(c, a, b) "v_mfma_f32_32x32x64_f8f6f4 %[" #c "], v[" #a "], v[" #b "], %[" #c "]\n\t"
            asm volatile(
                "s_mov_b32 %[cnt], 7\n"
                "1:\n\t"
                POLL_ADDR
                "ds_read_b32 %[t0], %[t0]\n\ts_waitcnt lgkmcnt(0)\n\t"
                "v_cmp_le_u32 vcc, 0x01010000, %[t0]\n\ts_cmp_eq_u64 vcc, -1\n\ts_cbranch_scc1 3f\n\t"
                "s_memrealtime %[now]\n\ts_waitcnt lgkmcnt(0)\n\t"
                "s_mov_b64 vcc, %[now]\n\ts_sub_u32 %[tmp], vcc_lo, %[ts]\n\t"
                "s_cmp_gt_u32 %[tmp], %[dl]\n\ts_cbranch_scc0 1b\n\t"
                "v_mov_b32 %[t0], %[bad]\n\tv_mov_b32 %[t1], 1\n\tds_write_b32 %[t0], %[t1]\n\t"
                "v_mov_b32 %[t0], %[prog]\n\tv_mov_b32 %[t1], 1000\n\tds_write_b32 %[t0], %[t1]\n\t"
                "s_branch 9f\n"
                "3:\n\t"
                MOVE
                RD_B0(0) RD_A0 RD_A1
                "\n4:\n\t"
                RD_B1(1024)
                "s_add_u32 %[m], %[m], 1\n\ts_add_u32 %[rel], %[m], %[nsb]\n\ts_sub_u32 %[cnt], %[cnt], 1\n\t"
                "s_cmp_eq_u32 %[cnt], 0\n\ts_cselect_b32 %[rel], 1000, %[rel]\n\t"
                "v_mov_b32 %[t0], %[prog]\n\tv_mov_b32 %[t1], %[rel]\n\t"
                "s_waitcnt lgkmcnt(4)\n\t"
                MM_(c00, 136:143, 152:159)
                "s_waitcnt lgkmcnt(2)\n\t"
                MM_(c10, 144:151, 152:159)
                "ds_write_b32 %[t0], %[t1]\n\t"
                RD_B0(2048)
                POLL_ADDR
                "s_waitcnt lgkmcnt(3)\n\t"
                MM_(c01, 136:143, 160:167)
                "ds_read_b32 %[t0], %[t0]\n\t"
                MM_(c11, 144:151, 160:167)
                RD_B1(3072)
                "s_waitcnt lgkmcnt(3)\n\t"
                MM_(c02, 136:143, 152:159)
                "s_waitcnt lgkmcnt(2)\n\t"
                "v_cmp_le_u32 vcc, 0x01010000, %[t0]\n\t"
                MM_(c12, 144:151, 152:159)
                "s_cmp_eq_u32 %[cnt], 0\n\ts_cbranch_scc1 6f\n\t"
                "s_cmp_eq_u64 vcc, -1\n\ts_cbranch_scc0 6f\n\t"
                MOVE
                RD_B0(0)
                "s_waitcnt lgkmcnt(2)\n\t"
                MM_(c03, 136:143, 160:167)
                RD_A0
                MM_(c13, 144:151, 160:167)
                RD_A1
                "s_branch 4b\n"
                "6:\n\t"
                "s_waitcnt lgkmcnt(0)\n\t"
                MM_(c03, 136:143, 160:167)
                MM_(c13, 144:151, 160:167)
                "s_cmp_eq_u32 %[cnt], 0\n\ts_cbranch_scc0 1b\n"
                "9:\n\t"
                "s_waitcnt lgkmcnt(0)\n\ts_nop 15\n\ts_nop 15"
                : [c00] "+v"(acc[0][0]), [c01] "+v"(acc[0][1]), [c02] "+v"(acc[0][2]), [c03] "+v"(acc[0][3]),
                  [c10] "+v"(acc[1][0]), [c11] "+v"(acc[1][1]), [c12] "+v"(acc[1][2]), [c13] "+v"(acc[1][3]),
                  [va] "+v"(va), [vb] "+v"(vb), [t0] "=&v"(t0), [t1] "=&v"(t1),
                  [m] "+s"(m), [blk] "+s"(blk), [cnt] "=&s"(cnt), [tmp] "=&s"(tmp), [rel] "=&s"(rel), [now] "=&s"(now)
                : [fin] "s"(lb + L_FIN), [prog] "s"(lb + L_PROG + wave * 4), [bad] "s"(lb + L_BAD), [ts] "s"(tstart),
                  [dl] "s"(DEADLINE2), [nsb] "n"(NSB), [nxb] "n"(NXB), [fwd] "s"(CH), [back] "n"(-(NXB - 1) * CH)
                : "memory", "vcc", "scc",
                  "v136", "v137", "v138", "v139", "v140", "v141", "v142", "v143", "v144", "v145", "v146", "v147",
                  "v148", "v149", "v150", "v151", "v152", "v153", "v154", "v155", "v156", "v157", "v158", "v159",
                  "v160", "v161", "v162", "v163", "v164", "v165", "v166", "v167");
        }
        if (wc == 0) STAMP(20 + wr);
        const bool bad = lds_ld(L, L_BAD) != 0u;
        const float mul = bad ? __builtin_nanf("") : scale;
        const unsigned oo = (va & 0x800u) * 32u + (va & 0x1f0u);
#pragma unroll
        for (int m = 0; m < 2; ++m)
#pragma unroll
            for (int e = 0; e < 16; ++e) {
                v4f v = {acc[m][0][e] * mul, acc[m][1][e] * mul, acc[m][2][e] * mul, acc[m][3][e] * mul};
                st16_sys(otile + (size_t)(m * 32 + (e & 3) + 8 * (e >> 2)) * NN * 4, oo, __builtin_bit_cast(v4u, v));
                if (OWIN < 32 && wr < 3) wait_vm<OWIN>();
            }
        return;
    }

    __builtin_amdgcn_s_setprio(3);
    if (wave == 10) {
        const int d = (dtab >> (4 * sl)) & 7;
        const int ord = q ^ 16;
        const unsigned per = (sl & 4) ? PERX8 : PER8;
        const unsigned t0 = tstart + (unsigned)(LEAD) + ord * per / 256;
        const int late1 = ((sl ^ xcc) & 4) ? FARLATE : LATE1;
        if (q < 16) send_panel<false>(px + (size_t)q * 256 * KS, L, myws, dws, rank, q, d, lane, t0, per, late1);
        else send_panel<true>(pw + (q - 16) * 256, L, myws, dws, rank, q, d, lane, t0, per, late1);
        return;
    }

    if (wave == 11) {
        __builtin_amdgcn_s_barrier();
        FENCE();
        const unsigned call = rfl(lds_ld(L, L_CALL));
        const unsigned fbase = (call - 1u) * FSTRIDE;
        const int d = lane & 7;
        const unsigned srcr = (unsigned)(rank + d) & 7u;
        const unsigned foff = WS_FLAG + (lane & 8 ? 16 + tj : ti) * FSTR + ((lane & 8 ? ti : tj) & (FREP - 1)) * FLINE + srcr * 4;
        const unsigned row0 = srcr * NCH;
        const int nmax = lane >= 8 ? 0 : d == 0 ? NCH : NBULK, amax = lane >= 8 || d == 0 ? 0 : 8;
        unsigned it = 0;
        int seen = 0, qn = 0;
        int seen2 = 0, fq = 0, glim = 2;
#define POLL(r) asm volatile("s_mov_b64 exec, %3\n\tglobal_atomic_or " r ", %0, %1, %2 sc0 sc1\n\ts_mov_b64 exec, -1" \
                             :: "v"(foff), "v"(0u), "s"(myws), "s"(wdone ? 0xffull : 0xffffull) : "memory", r)
#define TAKE(v, r, n) asm volatile("s_waitcnt vmcnt(" #n ")\n\tv_mov_b32 %0, " r : "=v"(v) :: "memory")
        bool wdone = false;
        auto process = [&](unsigned v) {
            bool dead = false;
            if ((++it & 15u) == 0u) {
                dead = rfl(__hip_atomic_load((const unsigned*)(myws + WS_TMO), __ATOMIC_RELAXED, __HIP_MEMORY_SCOPE_AGENT)) == call;
                if (!dead && (unsigned)rt() - tstart > DEADLINE) {
                    if (lane == 0) __hip_atomic_store((unsigned*)(myws + WS_TMO), call, __ATOMIC_RELAXED, __HIP_MEMORY_SCOPE_AGENT);
                    dead = true;
                }
            }
            int nx = (int)(v - fbase), nw = dpp0<0x128>(nx);
            if (wdone) nw = NCH;
            if (dead) { nx = nw = 64; lds_st(L, L_BAD, 1u); }
            int nn = min(min(nx, nw), nmax) - seen;
            nn = max(nn, 0);
            if (__ballot(nn > 0)) {
                int pre = nn;
                pre += dpp0<0x111>(pre); pre += dpp0<0x112>(pre); pre += dpp0<0x114>(pre);
                const int pos = qn + pre - nn;
                FENCE();
                for (int j = 0; j < nn; ++j) lds_st(L, L_QUEUE + (pos + j) * 4, row0 + seen + j + 1u);
                seen += nn;
                qn += __builtin_amdgcn_readlane(pre, 7);
                it = 0;
            }
            int a = min(nx - NBULK, amax);
            if (nw < NCH) a = min(a, (nw - NBULK) >> 1);
            for (;;) {
                const int n2 = max(min(a, glim) - seen2, 0);
                if (!__ballot(n2 > 0)) break;
                int pre = n2;
                pre += dpp0<0x111>(pre); pre += dpp0<0x112>(pre); pre += dpp0<0x114>(pre);
                const int pos = fq + pre - n2;
                FENCE();
                for (int j = 0; j < n2; ++j) {
                    const unsigned pr = (seen2 + j) & 1;
                    *(volatile LAS unsigned short*)(L + L_FIN + (pos + j) * 4) =
                        (unsigned short)((2 * d - 2 + pr) << 12 | (row0 + NBULK + ((seen2 + j) >> 1)));
                }
                seen2 += n2;
                fq += __builtin_amdgcn_readlane(pre, 7);
                it = 0;
                if (fq != glim * (NR - 1)) break;
                STAMP(31 + glim / 2);
                glim += 2;
            }
            wdone = qn >= NQ && fq >= NPAIR;
            return qn >= NQ && fq >= NFIN;
        };
        unsigned v;
        bool end = false;
        while (!end && (PD == 1 || qn < NQ - PFAST)) {
            POLL("v160"); TAKE(v, "v160", 0);
            end = process(v);
            if (qn >= NQ - PFAST) __builtin_amdgcn_s_sleep(POLLSLEEP2); else __builtin_amdgcn_s_sleep(POLLSLEEP);
        }
        STAMP(10);
        return;
    }

    {
        const int ld = wave - 8;
        __builtin_amdgcn_s_barrier();
        FENCE();
        const unsigned voff = lane * 16;
        const unsigned char* dbase = myws + WS_DATA + (size_t)(rfl(lds_ld(L, L_CALL)) & (NPAR - 1u)) * PAR_BYTES + (size_t)(ld ? 16 + tj : ti) * CH;
        const unsigned lb = (unsigned)(uintptr_t)L;
        int iss = 0, done = 0;
        unsigned spin = 0;
        while (done < NQ) {
            const unsigned v = lds_ld(L, lane < 16 ? L_PROG + (lane & 7) * 4 : L_QUEUE + (iss + (lane & 15)) * 4);
            int lim = iss + __builtin_ctzll(~(__ballot(v != 0u) >> 16));
            lim = min(min(lim, iss + 16), min(NQ, done + 15));
            const int st = iss >> 2;
            int fr = st + NSLOT;
#pragma unroll
            for (int j = NSLOT - 1; j >= 0; --j) if (__ballot(lane < 8 && (int)v < st + j - (NSLOT - 1))) fr = st + j;
            lim = min(lim, fr * 4);
            FENCE();
            const bool any = iss < lim;
            for (const int i0 = iss; iss < lim; ++iss) {
                const unsigned e = __builtin_amdgcn_readlane(v, 16 + iss - i0) - 1u;
                dma4k(dbase + (size_t)e * ROW_BYTES, lb + ld * 16384 + ((iss >> 2) & (NSLOT - 1)) * SLOT + (iss & 3) * CH, voff);
            }
            const int tg = iss - ((vm_count() + 3) >> 2);
            if (tg > done) {
                FENCE();
                done = tg;
                lds_st(L, L_FULL + ld * 4, (unsigned)done);
            } else if (!any) {
                if (expired(spin, tstart)) { lds_st(L, L_BAD, 1u); break; }
                __builtin_amdgcn_s_sleep(1);
            }
        }
        auto fin = [&](auto isw) {
            constexpr bool W = decltype(isw)::value;
            constexpr int NF = W ? NPAIR : NFIN;
            unsigned xa = lb + L_XR;
            iss = 0; done = 0;
            while (done < NF) {
                const unsigned v = lds_ld(L, lane < 16 ? L_PROG + (lane & 7) * 4 : L_FIN + (iss + (lane & 15)) * 4);
                int n = __builtin_ctz(~(unsigned)(__ballot((v & 0xfffu) != 0u) >> 16) | (1u << (W ? WBATCH : XBATCH)));
                int pm = (int)v;
                pm = min(pm, dpp0<0x121>(pm)); pm = min(pm, dpp0<0x122>(pm)); pm = min(pm, dpp0<0x124>(pm));
                pm = (int)rfl(pm) - NSB;
                int lim = W ? done + 7 : 2 * (pm + NXB);
                lim = lim < NF ? lim : NF;
                lim = pm < 0 ? 0 : lim - iss;
                n = n < lim ? n : lim;
                const unsigned off = (v & 0x1ffu) << 17 | (W ? (v & 0x1000u) << 6 : (v & 0x1000u) >> 1);
                FENCE();
                for (int j = 0; j < n; ++j) {
                    const unsigned vo = voff + __builtin_amdgcn_readlane(off, 16 + j);
                    if constexpr (W) {
                        const unsigned la = lb + (__builtin_amdgcn_readlane(v, 16 + j) & 0xf000u) * 2;
                        dma4k(dbase, la, vo);
                        dma4k(dbase + ROW_BYTES, la + CH, vo);
                    } else {
                        dma2k(dbase, xa, vo);
                        xa = xa == lb + L_CTL - 2048 ? lb + L_XR : xa + 2048;
                    }
                }
                if (n > 0) iss += n;
                const int vm = vm_count();
                const int tg = iss - (W ? (vm + 7) >> 3 : (vm + 1) >> 1);
                if (tg > done) {
                    FENCE();
                    const int last = tg == NF ? NFIN : tg;
                    if (done + lane < last) *(volatile LAS unsigned char*)(L + L_FIN + (done + lane) * 4 + 2 + W) = 1;
                    done = tg;
                } else if (n <= 0) {
                    if (expired(spin, tstart)) { lds_st(L, L_BAD, 1u); break; }
                    __builtin_amdgcn_s_sleep(1);
                }
            }
        };
        if (ld) fin(std::true_type{}); else fin(std::false_type{});
        STAMP(11 + 2 * ld);
    }
}

}

extern "C" void kernel_launch(int rank, int nranks, ncclComm_t comm, hipStream_t stream,
                              void* const* d_in, const int* in_sizes, int n_in,
                              void* d_out, int out_size, const int* in_dtypes, int out_dtype,
                              void* const* d_out_all, void* const* d_ws_all, size_t ws_bytes) {
    if (nranks != NR || n_in != 4 || ws_bytes < WS_NEED || in_sizes[0] != MM * KS || in_sizes[1] != KS * NN ||
        out_size != MM * NN || (unsigned)rank >= (unsigned)NR) {
        fprintf(stderr, "kernel_launch: unexpected problem geometry\n");
        return;
    }
    Params p;
    p.x = (const unsigned char*)d_in[0]; p.w = (const unsigned char*)d_in[1];
    p.sx = (const float*)d_in[2]; p.sw = (const float*)d_in[3];
    p.out = (float*)d_out; p.rank = rank;
    p.myws = (unsigned char*)d_ws_all[rank];
    const unsigned dtab[NR] = {0x14735602u, 0x73164502u, 0x13654207u, 0x74623501u, 0x16245307u, 0x75234601u, 0x15724306u, 0x75143206u};
    p.dtab = dtab[rank];
    for (int s = 0; s < NR; ++s) p.dws[s] = (unsigned char*)d_ws_all[(rank + NR - ((p.dtab >> (4 * s)) & 7)) & 7];
    hipLaunchKernelGGL(fused_gemm_ag, dim3(NWG), dim3(NTHREADS), 0, stream, p);
    HIP_CHECK(hipGetLastError());
}
```

```cpp
#include <hip/hip_runtime.h>
#include <rccl/rccl.h>
#include <cstdio>
#include <cstdint>
#include <type_traits>

#define HIP_CHECK(expr) do { hipError_t _e = (expr); if (_e != hipSuccess) { \
    fprintf(stderr, "HIP error %s at %s:%d\n", hipGetErrorString(_e), __FILE__, __LINE__); \
    return; } } while (0)

namespace {

#define LAS __attribute__((address_space(3)))
typedef int v4i __attribute__((ext_vector_type(4)));
typedef int v8i __attribute__((ext_vector_type(8)));
typedef unsigned v2u __attribute__((ext_vector_type(2)));
typedef unsigned v4u __attribute__((ext_vector_type(4)));
typedef float v4f __attribute__((ext_vector_type(4)));
typedef float v16f __attribute__((ext_vector_type(16)));
typedef unsigned long long u64;

constexpr int NR = 8;
constexpr int MM = 4096, NN = 4096, KS = 512;
constexpr int NCH = 32;
constexpr int NBULK = 28;
constexpr int NPANEL = 32;
constexpr int CH = 4096;
constexpr int NWG = 256, NTHREADS = 768;
constexpr int NQ = (NR - 1) * NBULK + NCH;
constexpr int NSB = NQ / 4;
constexpr int NPAIR = 2 * (NR - 1);
constexpr int NFIN = 4 * NPAIR;
constexpr unsigned FSTRIDE = 64;
constexpr size_t WS_CTR = 0;
#define FREP 1
#define FLINE 128
#define FSTR (FLINE * FREP)
constexpr size_t WS_FLAG = 524288;
constexpr size_t WS_TMO = 16384 + 4096;
constexpr size_t WS_DBG = 32768;
constexpr size_t WS_DATA = 1u << 20;
constexpr size_t ROW_BYTES = (size_t)NPANEL * CH;
constexpr size_t SRC_BYTES = (size_t)NCH * ROW_BYTES;
constexpr size_t PAR_BYTES = (size_t)NR * SRC_BYTES;
#define NPAR 2
constexpr size_t WS_NEED = WS_DATA + NPAR * PAR_BYTES;
constexpr unsigned DEADLINE = 40000000u;
constexpr unsigned DEADLINE2 = 60000000u;
constexpr int NSLOT = 4, SLOT = 32768;
constexpr int L_XR = NPAIR * 2 * CH;
constexpr int NXB = 11;
constexpr int L_CTL = L_XR + NXB * CH;
constexpr int L_FULL = L_CTL;
constexpr int L_PROG = L_CTL + 64;
constexpr int L_BAD = L_CTL + 128;
constexpr int L_TI = L_CTL + 192;
constexpr int L_VOTE = L_CTL + 196;
constexpr int L_QUEUE = L_CTL + 256;
constexpr int L_FIN = L_CTL + 1536;
constexpr int L_CALL = L_CTL + 2048;
constexpr int LDS_BYTES = L_CALL + 64 + 256;
static_assert(L_QUEUE + (NQ + 32) * 4 <= L_FIN && L_FIN + (NFIN + 32) * 4 <= L_CALL && L_CTL >= NSLOT * SLOT, "LDS map");

#define PER8 442
#define PERX8 444
#define LEAD 50
#define POLLSLEEP 2
#define POLLSLEEP2 POLLSLEEP
#define PFAST 14
#define PD 1
#define PGAP 16
#define REBASE 2
#define RB0 10
#define XDELAY 200
#define FARLATE 0
#define RBMAX 100000
#define NOSWAP 0
#define TOUCH 0
#define TDELAY 100
#define TMASK 31
#define TWAVE 11
#define TOP "global_load_dword v161, %0, %1 sc0 sc1"
#define SSCALE 1
#define CATCH 10
#define TPER per
#define SCTR 0
#define STOUCH 0
#define LATE1 100000
#define LATE2 100000
#define XBATCH 8
#define WBATCH 4
#define OWIN 63
#define VSTART 200
#define VPOLL 450
#define VIDX 0
#define VCLAMP 10
#define VMODE 4
#define VOP "global_load_dword %2, %3, %5 sc0 sc1"
#define VFRESH 16384
#define VREP 4
#define OSL 0
#define OFREE 0
#define OGRP 3
#define SPEC 0
#define ELEADX 0
#define ELEADW 0
#define ZFIRST 4
#define TWIN 0
#define XMID 1

struct Params {
    const unsigned char* x; const unsigned char* w; unsigned char* myws;
    int rank;
    unsigned dtab;
    const float* sx; const float* sw; float* out;
    unsigned char* dws[NR];
};

__device__ __forceinline__ u64 rt() { return __builtin_amdgcn_s_memrealtime(); }
__device__ __forceinline__ unsigned rfl(unsigned v) { return __builtin_amdgcn_readfirstlane(v); }
__device__ __forceinline__ unsigned lds_ld(LAS unsigned char* L, int off) { return *(volatile LAS unsigned*)(L + off); }
__device__ __forceinline__ void lds_st(LAS unsigned char* L, int off, unsigned v) { *(volatile LAS unsigned*)(L + off) = v; }
#define FENCE() asm volatile("" ::: "memory")
template <int N> __device__ __forceinline__ void wait_vm() { asm volatile("s_waitcnt vmcnt(%0)" :: "n"(N) : "memory"); }
__device__ __forceinline__ int vm_count() {
    unsigned r;
    asm volatile("s_getreg_b32 %0, hwreg(HW_REG_IB_STS)" : "=s"(r) :: "memory");
    return (int)((r & 15u) | ((r >> 18) & 48u));
}
__device__ __forceinline__ bool expired(unsigned& spin, unsigned t0) {
    return (++spin & 4095u) == 0u && (unsigned)rt() - t0 > DEADLINE2;
}
template <int CTRL> __device__ __forceinline__ int dpp0(int v) { return __builtin_amdgcn_update_dpp(0, v, CTRL, 0xf, 0xf, true); }

#define STAMPL(i) do {} while (0)
#define STAMP(i) do {} while (0)

__device__ __forceinline__ void dma4k(const unsigned char* g, unsigned ldsaddr, unsigned voff) {
    asm volatile("s_mov_b32 m0, %2\n\ts_nop 0\n\t"
                 "global_load_lds_dwordx4 %0, %1 offset:0\n\tglobal_load_lds_dwordx4 %0, %1 offset:1024\n\t"
                 "global_load_lds_dwordx4 %0, %1 offset:2048\n\tglobal_load_lds_dwordx4 %0, %1 offset:3072"
                 :: "v"(voff), "s"(g), "s"(ldsaddr) : "memory");
}
__device__ __forceinline__ void dma2k(const unsigned char* g, unsigned ldsaddr, unsigned voff) {
    asm volatile("s_mov_b32 m0, %2\n\ts_nop 0\n\t"
                 "global_load_lds_dwordx4 %0, %1 offset:0\n\tglobal_load_lds_dwordx4 %0, %1 offset:1024"
                 :: "v"(voff), "s"(g), "s"(ldsaddr) : "memory");
}
__device__ __forceinline__ void dma2k_sys(const unsigned char* g, unsigned ldsaddr, unsigned voff) {
    asm volatile("s_mov_b32 m0, %2\n\ts_nop 0\n\t"
                 "global_load_lds_dwordx4 %0, %1 offset:0 sc0 sc1\n\tglobal_load_lds_dwordx4 %0, %1 offset:1024 sc0 sc1"
                 :: "v"(voff), "s"(g), "s"(ldsaddr) : "memory");
}
template <int K> __device__ __forceinline__ void lds_ld2(v2u& d, unsigned a) {
    asm volatile("ds_read2st64_b32 %0, %1 offset0:%2 offset1:%3" : "=v"(d) : "v"(a), "n"(4 * K), "n"(4 * K + 4) : "memory");
}
__device__ __forceinline__ unsigned min3(unsigned a, v2u b) { return min(a, min(b[0], b[1])); }
__device__ __forceinline__ void st16_sys(unsigned char* sbase, unsigned voff, v4u v) {
    asm volatile("global_store_dwordx4 %0, %1, %2 sc0 sc1\n\ts_nop 1" :: "v"(voff), "v"(v), "s"(sbase) : "memory");
}
__device__ __forceinline__ void st4_sys_lane0(unsigned* p, unsigned v, unsigned voff) {
#define FOP "global_atomic_swap %0, %1, %2 sc1"
    asm volatile("s_mov_b64 exec, %3\n\t" FOP "\n\ts_mov_b64 exec, -1"
                 :: "v"(voff), "v"(v), "s"(p), "n"((1 << FREP) - 1) : "memory");
}
__device__ __forceinline__ void ld4(unsigned& d, unsigned voff, const unsigned char* sbase) {
    asm volatile("global_load_dword %0, %1, %2" : "=v"(d) : "v"(voff), "s"(sbase) : "memory");
}
template <int OFF> __device__ __forceinline__ void ld16(v4u& d, unsigned voff, const unsigned char* sbase) {
    asm volatile("global_load_dwordx4 %0, %1, %2 offset:%3" : "=v"(d) : "v"(voff), "s"(sbase), "n"(OFF) : "memory");
}
__device__ __forceinline__ void landed(unsigned (&g)[16]) {
    asm volatile("" : "+v"(g[0]), "+v"(g[1]), "+v"(g[2]), "+v"(g[3]), "+v"(g[4]), "+v"(g[5]), "+v"(g[6]), "+v"(g[7]),
                      "+v"(g[8]), "+v"(g[9]), "+v"(g[10]), "+v"(g[11]), "+v"(g[12]), "+v"(g[13]), "+v"(g[14]), "+v"(g[15]) :: "memory");
}
__device__ __forceinline__ void landed(v4u& a, v4u& b, v4u& c, v4u& d) {
    asm volatile("" : "+v"(a), "+v"(b), "+v"(c), "+v"(d) :: "memory");
}
__device__ __forceinline__ void swap16(v4u& a, v4u& b, unsigned mask) {
    unsigned x0 = a[0], x1 = a[1], x2 = a[2], x3 = a[3], y0 = b[0], y1 = b[1], y2 = b[2], y3 = b[3];
    asm volatile("s_mov_b32 exec_lo, %8\n\ts_mov_b32 exec_hi, %8\n\tv_swap_b32 %0, %4\n\tv_swap_b32 %1, %5\n\tv_swap_b32 %2, %6\n\tv_swap_b32 %3, %7\n\t"
                 "s_mov_b64 exec, -1"
                 : "+v"(x0), "+v"(x1), "+v"(x2), "+v"(x3), "+v"(y0), "+v"(y1), "+v"(y2), "+v"(y3) : "s"(mask));
    a[0] = x0; a[1] = x1; a[2] = x2; a[3] = x3; b[0] = y0; b[1] = y1; b[2] = y2; b[3] = y3;
}

__device__ __forceinline__ void nonzero(v4u& a, v4u& b, unsigned on) {
    unsigned x0 = a[0], x1 = a[1], x2 = a[2], x3 = a[3], y0 = b[0], y1 = b[1], y2 = b[2], y3 = b[3];
#define NZ(r) "v_cmp_eq_u32 vcc, 0, " r "\n\tv_cndmask_b32 " r ", " r ", %9, vcc\n\t"
    asm volatile("s_cmp_eq_u32 %8, 0\n\ts_cbranch_scc1 1f\n\t"
                 NZ("%0") NZ("%1") NZ("%2") NZ("%3") NZ("%4") NZ("%5") NZ("%6") NZ("%7") "\n1:"
                 : "+v"(x0), "+v"(x1), "+v"(x2), "+v"(x3), "+v"(y0), "+v"(y1), "+v"(y2), "+v"(y3) : "s"(on), "v"(0x80u) : "vcc", "scc");
    a[0] = x0; a[1] = x1; a[2] = x2; a[3] = x3; b[0] = y0; b[1] = y1; b[2] = y2; b[3] = y3;
}

template <bool IS_W>
__device__ __forceinline__ void send_panel(const unsigned char* src, LAS unsigned char* L, unsigned char* myws, unsigned char* dws,
                                           int rank, int q, int d, int lane, unsigned t0, unsigned per, int late1) {
    const bool self = d == 0;
    constexpr bool SP = SPEC & (IS_W ? 2 : 1);
    const unsigned lo = IS_W ? lane * 4 : (lane >> 2) * 4 * KS + (lane & 3) * 16;
    const unsigned so = lane * 16, fo = lane * FLINE;
    unsigned gw[4][16];
    v4u ow[2][4];
    v4u gx[2][16];
    auto loadw = [&](int i, unsigned (&g)[16]) {
#pragma unroll
        for (int k = 0; k < 16; ++k) ld4(g[k], lo, src + (size_t)(i * 16 + k) * NN);
    };
    auto xformw = [&](unsigned (&g)[16], v4u (&ob)[4]) {
        landed(g);
#pragma unroll
        for (int t = 0; t < 4; ++t) {
            const unsigned a0 = g[4 * t], a1 = g[4 * t + 1], a2 = g[4 * t + 2], a3 = g[4 * t + 3];
            const unsigned x0 = __builtin_amdgcn_perm(a1, a0, 0x05010400u), x1 = __builtin_amdgcn_perm(a1, a0, 0x07030602u);
            const unsigned y0 = __builtin_amdgcn_perm(a3, a2, 0x05010400u), y1 = __builtin_amdgcn_perm(a3, a2, 0x07030602u);
            ob[0][t] = __builtin_amdgcn_perm(y0, x0, 0x05040100u);
            ob[1][t] = __builtin_amdgcn_perm(y0, x0, 0x07060302u);
            ob[2][t] = __builtin_amdgcn_perm(y1, x1, 0x05040100u);
            ob[3][t] = __builtin_amdgcn_perm(y1, x1, 0x07060302u);
        }
        landed(ob[0], ob[1], ob[2], ob[3]);
    };
    auto loadx = [&](int grp, int half, v4u (&g)[16]) {
#pragma unroll
        for (int j = 2 * half; j < 2 * half + 2; ++j) {
            const unsigned char* s = src + (size_t)((j & 1) * 64 + (j >> 1) * 256) * KS + grp * 64;
            ld16<0>(g[4 * j], lo, s); ld16<KS>(g[4 * j + 1], lo, s); ld16<2 * KS>(g[4 * j + 2], lo, s); ld16<3 * KS>(g[4 * j + 3], lo, s);
        }
    };
    const bool b0 = lane & 1, b1 = lane & 2;
    auto xformx = [&](int half, v4u (&g)[16]) {
#pragma unroll
        for (int j = 2 * half; j < 2 * half + 2; ++j) {
            v4u &r0 = g[4 * j], &r1 = g[4 * j + 1], &r2 = g[4 * j + 2], &r3 = g[4 * j + 3];
            landed(r0, r1, r2, r3);
#pragma unroll
            for (int e = 0; e < 4; ++e) {
                const unsigned a0 = r0[e], a1 = r1[e], a2 = r2[e], a3 = r3[e];
                const unsigned u0 = __builtin_amdgcn_mov_dpp(a0, 0xB1, 0xf, 0xf, true), u1 = __builtin_amdgcn_mov_dpp(a1, 0xB1, 0xf, 0xf, true);
                const unsigned u2 = __builtin_amdgcn_mov_dpp(a2, 0xB1, 0xf, 0xf, true), u3 = __builtin_amdgcn_mov_dpp(a3, 0xB1, 0xf, 0xf, true);
                const unsigned n0 = b0 ? u1 : a0, n1 = b0 ? a1 : u0, n2 = b0 ? u3 : a2, n3 = b0 ? a3 : u2;
                const unsigned w0 = __builtin_amdgcn_mov_dpp(n0, 0x4E, 0xf, 0xf, true), w1 = __builtin_amdgcn_mov_dpp(n1, 0x4E, 0xf, 0xf, true);
                const unsigned w2 = __builtin_amdgcn_mov_dpp(n2, 0x4E, 0xf, 0xf, true), w3 = __builtin_amdgcn_mov_dpp(n3, 0x4E, 0xf, 0xf, true);
                r0[e] = b1 ? w2 : n0; r2[e] = b1 ? n2 : w0; r1[e] = b1 ? w3 : n1; r3[e] = b1 ? n3 : w1;
            }
            landed(r0, r1, r2, r3);
        }
    };
    unsigned* ctr = (unsigned*)(myws + WS_CTR + (size_t)blockIdx.x * 64);
    unsigned prevv;
    STAMPL(9);
    asm volatile("global_load_dword %0, %1, %2 sc1" : "=v"(prevv) : "v"(0u), "s"(ctr) : "memory");
    if constexpr (IS_W) { loadw(0, gw[0]); loadw(1, gw[1]); } else { loadx(0, 0, gx[0]); loadx(0, 1, gx[0]); }
    STAMPL(10);
    asm volatile("s_waitcnt vmcnt(%1)" : "+v"(prevv) : "n"(IS_W ? 32 : 16) : "memory");
    STAMPL(11);
    const unsigned prev = rfl(prevv), call = prev + 1u, fbase = prev * FSTRIDE;
    lds_st(L, L_CALL, call);
    asm volatile("s_waitcnt lgkmcnt(0)" ::: "memory");
    __builtin_amdgcn_s_barrier();
    FENCE();
    STAMPL(12);
    unsigned char* db = dws + WS_DATA + (size_t)(call & (NPAR - 1u)) * PAR_BYTES + (size_t)rank * SRC_BYTES + (size_t)q * CH;
    unsigned* dflag = (unsigned*)(dws + WS_FLAG + q * FSTR + rank * 4);
#define FMODE 1
    unsigned fm = (d & 1) ? 0x05555555u : 0x0AAAAAAAu;
    if (FMODE == 1 && !self) fm = (fm & 0x0F000000u) | (0x00111111u << (d & 3));
    if (FMODE == 2 && !self) fm = (fm & 0x0F000000u) | (0x00100000u << (d & 3)) | (0x00000808u << d);
    if (FMODE == 3 && !self) fm = (fm & 0x06000000u) | (0x00100000u << (d & 3)) | (0x00000400u << d);
    const unsigned fmask = fm | 0x08000000u | (self ? 0xA0000000u : SP ? (IS_W ? 0x20000000u : 0x10000000u) : IS_W ? 0xA0000000u : 0xF0000000u);
    constexpr int NSPEC = IS_W ? NCH - 1 : NBULK + 1;
    unsigned char* zb = myws + WS_DATA + (size_t)((call + 1u) & (NPAR - 1u)) * PAR_BYTES + (size_t)((rank - d) & 7) * SRC_BYTES +
                        (size_t)NSPEC * ROW_BYTES + (size_t)q * CH;
    int hs = 0;
    if constexpr (IS_W) { loadw(2, gw[2]); wait_vm<32>(); STAMPL(13); xformw(gw[0], ow[0]); }
    else { wait_vm<8>(); STAMPL(13); xformx(0, gx[0]); wait_vm<0>(); xformx(1, gx[0]); }
    int ops = IS_W ? 32 : 0, h1 = 0, h2 = 0, h3 = 0, h4 = 0, h5 = 0, h6 = 0;
    int l1 = IS_W ? 16 : 0, l2 = IS_W ? 32 : 0, l3 = 0;
    int ia = 0, ib = 0;
    unsigned fa = 0, fb = 0;
    unsigned tt = 0, period = self ? 0u : per;
    bool wide = self;
    int slip = 0;
    auto flag_try = [&](int ret) {
        if (ia != 0 && ret >= ia) { st4_sys_lane0(dflag, fa, fo); ++ops; ia = ib; fa = fb; ib = 0; }
    };
    auto flag_put = [&](unsigned v) {
        if (ia == 0) { ia = h1; fa = v; } else { ib = h1; fb = v; }
    };
    auto slot = [&](unsigned char* a, v4u& v) {
        int n0 = wide ? h4 + 0 : h2 + 0, n1 = wide ? h6 + 0 : h3 + 0, n2 = wide ? h6 + 0 : h4 + 0;
        if (TWIN == 1 && !IS_W && wide) n0 = n1;
        if (TWIN == 2 && !IS_W && wide && !self) n0 = n1 = 0;
        const unsigned due = t0 + (tt >> 3);
        int late;
        for (;;) {
            late = (int)((unsigned)rt() - due);
            const int ret = ops - vm_count();
            flag_try(ret);
            if (late >= slip && ret >= (late >= LATE2 ? n2 : late >= late1 ? n1 : n0)) break;
            __builtin_amdgcn_s_sleep(1);
        }
        if (CATCH < 1000) slip = wide ? 0 : max(late - CATCH, 0);
        st16_sys(a, so, v);
        h6 = h5; h5 = h4; h4 = h3; h3 = h2; h2 = h1; h1 = ++ops;
        tt += period;
    };
    auto piece = [&](int i, v4u& s0, v4u& s1, v4u& s2, v4u& s3, auto&& loads, auto&& xform) {
        if (i == NBULK + 1 && !self) {
            if (!IS_W || REBASE > 1) t0 += min(max((int)((unsigned)rt() - t0 - ((tt - per) >> 3)) - RB0, 0), RBMAX);
            if (!IS_W) tt += 6u * TPER - per / 2 + XDELAY;
            period = TPER / 2; wide = true; slip = 0;
        }
        const unsigned fv = fbase + ((IS_W || self || i < NBULK) ? i + 1 : 2 * i - (NBULK - 2));
        const bool spec = SP && !self && i >= NSPEC;
        if (spec && i == NSPEC) hs = h1;
        if (SP) { const unsigned on = rfl(i > NBULK); nonzero(s0, s1, on); nonzero(s2, s3, on); }
        auto early = [&](int j) {
            if (!spec) return;
            const int n = 4 * (i - NSPEC) + j + 1 + ELEADX;
            if (IS_W ? j + ELEADW == 3 : ((n & 1) == 0 && n <= 12) || (i == NSPEC && j == 0 && n >= 2)) {
                const unsigned v = IS_W ? fv : fbase + NBULK + 2 + (n >> 1);
                if (ia == 0) { ia = hs; fa = v; } else { ib = hs; fb = v; }
                flag_try(ops - vm_count());
            }
        };
        slot(db, s0);
        early(0);
        slot(db + 1024, s1);
        early(1);
        if (!SP && XMID && !IS_W && !self && i > NBULK) flag_put(fv - 1u);
        if (SP && !self && (unsigned)(i - ZFIRST) < 4u * (NCH - NSPEC)) {
            st16_sys(zb + (size_t)((i - ZFIRST) >> 2) * ROW_BYTES + ((i - ZFIRST) & 3) * 1024, so, v4u{0u, 0u, 0u, 0u});
            ++ops;
        }
        loads();
        slot(db + 2048, s2);
        early(2);
        xform();
        slot(db + 3072, s3);
        early(3);
        if ((fmask >> i) & 1u) flag_put(fv);
        db += ROW_BYTES;
    };
    auto arrived = [&](int l) { while (ops - vm_count() < l) __builtin_amdgcn_s_sleep(1); };
    STAMPL(14);
    if constexpr (IS_W) {
        auto wpiece = [&](int i, v4u (&oc)[4], v4u (&on)[4], unsigned (&gl)[16], unsigned (&gt)[16]) {
            piece(i, oc[0], oc[1], oc[2], oc[3],
                  [&]() { if (i + 3 < NCH) { loadw(i + 3, gl); ops += 16; } l3 = ops; },
                  [&]() { arrived(l1); xformw(gt, on); l1 = l2; l2 = l3; });
        };
#pragma unroll 1
        for (int i = 0; i < NCH; i += 4) {
            wpiece(i, ow[0], ow[1], gw[3], gw[1]);
            wpiece(i + 1, ow[1], ow[0], gw[0], gw[2]);
            wpiece(i + 2, ow[0], ow[1], gw[1], gw[3]);
            wpiece(i + 3, ow[1], ow[0], gw[2], gw[0]);
        }
    } else {
        auto group = [&](int g, v4u (&c)[16], v4u (&n)[16]) {
            const bool more = g + 1 < NCH / 4;
            piece(4 * g, c[0], c[4], c[8], c[12], [&]() { if (more) { loadx(g + 1, 0, n); ops += 8; l1 = ops; } }, [&]() {});
            piece(4 * g + 1, c[1], c[5], c[9], c[13], [&]() { if (more) { loadx(g + 1, 1, n); ops += 8; l2 = ops; } }, [&]() {});
            piece(4 * g + 2, c[2], c[6], c[10], c[14], [&]() {}, [&]() { arrived(l1); xformx(0, n); });
            piece(4 * g + 3, c[3], c[7], c[11], c[15], [&]() {}, [&]() { arrived(l2); xformx(1, n); });
        };
#pragma unroll 1
        for (int g = 0; g < NCH / 4; g += 2) {
            group(g, gx[0], gx[1]);
            const unsigned last = rfl(g == NCH / 4 - 2 && !self ? ~0u : 0u);
#pragma unroll
            for (int a = 0; a < 4; ++a)
#pragma unroll
                for (int c = a + 1; c < 4; ++c) swap16(gx[1][4 * a + c], gx[1][4 * c + a], last);
            group(g + 1, gx[1], gx[0]);
        }
    }
    while (ia != 0) flag_try(ops - vm_count());
    if (lane == 0) __hip_atomic_store(ctr, call, __ATOMIC_RELAXED, __HIP_MEMORY_SCOPE_AGENT);
    STAMP(9);
}

__global__ void __launch_bounds__(NTHREADS) fused_gemm_ag(Params p) {
    __shared__ __attribute__((aligned(1024))) unsigned char lds_raw[LDS_BYTES];
    LAS unsigned char* L = (LAS unsigned char*)lds_raw;
    const int tid = threadIdx.x, lane = tid & 63;
    const int wave = rfl(tid >> 6);
    const int b = blockIdx.x, idx = b >> 3;
    unsigned xcc;
    asm volatile("s_getreg_b32 %0, hwreg(HW_REG_XCC_ID)" : "=s"(xcc));
    const int tj = (xcc & 1) * 8 + (idx & 7);
    const int q = (idx & 4) * 4 + (xcc & 3) * 4 + (idx & 3);
    int sl = (xcc & 4) + (idx >> 3);
    if (!NOSWAP && ((idx >> 2) & 1) == (xcc & 1)) sl = sl == 1 ? 4 + (idx & 3) : sl == 4 + (idx & 3) ? 1 : sl;
    const unsigned char *px = p.x, *pw = p.w;
    unsigned char *myws = p.myws, *dws = p.dws[sl & 7];
    int rank = p.rank;
    unsigned dtab = p.dtab;
    unsigned tstart = (unsigned)rt();
    asm volatile("" : "+s"(px), "+s"(pw), "+s"(myws), "+s"(dws), "+s"(rank), "+s"(dtab), "+s"(tstart));

    if (wave < 8) {
        if (wave == 0) STAMP(0);
        lds_st(L, L_CTL + wave * 256 + lane * 4, 0u);
        asm volatile("s_waitcnt lgkmcnt(0)" ::: "memory");
        __builtin_amdgcn_s_barrier();
        FENCE();
        float scale, scale2;
        asm volatile("s_load_dword %0, %2, 0x0\n\ts_load_dword %1, %3, 0x0\n\ts_waitcnt lgkmcnt(0)"
                     : "=&s"(scale), "=&s"(scale2) : "s"(p.sx), "s"(p.sw) : "memory");
        scale *= scale2;
        scale = __builtin_bit_cast(float, rfl(__builtin_bit_cast(unsigned, scale)));
        const int wr = wave >> 1, wc = wave & 1, i = lane & 31, h = lane >> 5;
        unsigned char* otile = (unsigned char*)p.out;
        if (idx == VIDX) {
            const unsigned pg = (xcc & 7) * 4 + (xcc & 3);
            const unsigned char* a = otile + (size_t)pg * (2u << 20) + wave * 1024;
            while ((unsigned)rt() - tstart < VSTART) __builtin_amdgcn_s_sleep(4);
            int best[2] = {1 << 20, 1 << 20};
#pragma unroll 1
            for (int r = 0; r <= VREP; ++r)
#pragma unroll
                for (int k = 0; k < 2; ++k) {
                    u64 c0, c1;
                    unsigned z;
                    asm volatile("s_memtime %0\n\ts_waitcnt lgkmcnt(0)\n\ts_load_dword %2, %3, %4\n\ts_waitcnt lgkmcnt(0)\n\t"
                                 "s_memtime %1\n\ts_waitcnt lgkmcnt(0)"
                                 : "=&s"(c0), "=&s"(c1), "=&s"(z) : "s"(a), "s"(k * 8192u + r * 16384u) : "memory");
                    if (r > 0) best[k] = min(best[k], (int)(unsigned)(c1 - c0));
                }
            int df = min(max((best[0] - best[1]) >> 4, -VCLAMP), VCLAMP);
            if (((xcc >> 2) ^ pg) & 1) df = -df;
            unsigned all = 0;
            if (lane == 0) all = __hip_atomic_fetch_add((LAS unsigned*)(L + L_VOTE), 0x1000u + 16u + df, __ATOMIC_RELAXED, __HIP_MEMORY_SCOPE_WORKGROUP);
            all = rfl(all) + 0x1000u + 16u + df;
            if (all >= 0x8000u) {
                const unsigned vote = rfl(lds_ld(L, L_CALL)) << 8 | (all & 255u);
                asm volatile("s_mov_b64 exec, 0xffff\n\tglobal_atomic_swap %0, %1, %2 sc1\n\ts_mov_b64 exec, -1"
                             :: "v"((unsigned)WS_FLAG + (16 + lane) * FSTR + 64 + (xcc & 7) * 4), "v"(vote), "s"(myws) : "memory");
                STAMP(7);
            }
        }
        otile += ((size_t)((wr & 1) * 64 + (wr >> 1) * 256) * NN + tj * 256 + wc * 128) * 4;
        asm volatile("" : "+s"(scale), "+s"(otile));
        const int abase = h * 8192 + (64 * wr + i) * 16;
        const int bbase = 16384 + h * 8192 + (32 * wc + i) * 16;
        v16f acc[2][4];
#pragma unroll
        for (int m = 0; m < 2; ++m)
#pragma unroll
            for (int c = 0; c < 4; ++c)
#pragma unroll
                for (int e = 0; e < 16; ++e) acc[m][c][e] = 0.f;
        unsigned spin = 0;
        for (int s = 0; s < NSB; ++s) {
            for (;;) {
                v2u f = *(volatile LAS v2u*)(L + L_FULL);
                unsigned mn = rfl(f.x < f.y ? f.x : f.y);
                if (mn >= (unsigned)(4 * s + 4)) break;
                if (expired(spin, tstart)) { lds_st(L, L_BAD, 1u); break; }
                __builtin_amdgcn_s_sleep(1);
            }
            FENCE();
            LAS unsigned char* sl = L + (s & (NSLOT - 1)) * SLOT;
            v8i a[2], bf[4];
#pragma unroll
            for (int m = 0; m < 2; ++m) {
                v4i lo = *(LAS v4i*)(sl + abase + m * 512), hi = *(LAS v4i*)(sl + abase + m * 512 + 4096);
                a[m] = __builtin_shufflevector(lo, hi, 0, 1, 2, 3, 4, 5, 6, 7);
            }
#pragma unroll
            for (int c = 0; c < 4; ++c) {
                v4i lo = *(LAS v4i*)(sl + bbase + c * 1024), hi = *(LAS v4i*)(sl + bbase + c * 1024 + 4096);
                bf[c] = __builtin_shufflevector(lo, hi, 0, 1, 2, 3, 4, 5, 6, 7);
                if (c == 3) {
                    asm volatile("s_waitcnt lgkmcnt(0)" : "+v"(bf[3]) :: "memory");
                    lds_st(L, L_PROG + wave * 4, (unsigned)(s + 1 < NSB ? s + 1 : NSB + 7 * wr));
                    FENCE();
                }
#pragma unroll
                for (int m = 0; m < 2; ++m)
                    acc[m][c] = __builtin_amdgcn_mfma_scale_f32_32x32x64_f8f6f4(a[m], bf[c], acc[m][c], 0, 0, 0, 0, 0, 0);
            }
        }
        if (wave == 0) STAMP(1);
        {
            const unsigned ti = (rfl(lds_ld(L, L_TI)) - 1u) & 15u;
            otile += (size_t)((ti >> 1) * 512 + (ti & 1) * 128) * NN * 4;
            asm volatile("" : "+s"(otile));
        }
        unsigned va;
        {
            const unsigned lb = (unsigned)(uintptr_t)L;
            unsigned vb = lb + (32 * wc + i) * 16, t0, t1;
            unsigned m = 7 * wr, blk = (7 * wr + NXB - 1) % NXB, cnt, tmp, rel;
            va = lb + L_XR + blk * CH + h * 2048 + i * 16;
            u64 now;
#define POLL_ADDR \
            "v_bfe_u32 %[t0], %[va], 11, 1\n\ts_lshl3_add_u32 %[tmp], %[m], %[fin]\n\t" \
            "v_lshl_add_u32 %[t0], %[t0], 2, %[tmp]\n\t"
#define MOVE \
            "s_add_u32 %[blk], %[blk], 1\n\ts_cmp_eq_u32 %[blk], %[nxb]\n\ts_cselect_b32 %[blk], 0, %[blk]\n\t" \
            "s_cselect_b32 %[tmp], %[back], %[fwd]\n\t" \
            "v_and_b32 %[t0], 0xf000, %[t0]\n\tv_and_b32 %[vb], 0x1fff, %[vb]\n\tv_add_u32 %[va], %[tmp], %[va]\n\t" \
            "v_lshl_add_u32 %[vb], %[t0], 1, %[vb]\n\t"
#define RD(r0, r1, adr, off, hi) \
            "ds_read_b128 v[" #r0 "], %[" #adr "] offset:" #off "\n\tds_read_b128 v[" #r1 "], %[" #adr "] offset:" #off "+" #hi "\n\t"
#define RD_A0 RD(136:139, 140:143, va, 0, 1024)
#define RD_A1 RD(144:147, 148:151, va, 512, 1024)
#define RD_B0(off) RD(152:155, 156:159, vb, off, 4096)
#define RD_B1(off) RD(160:163, 164:167, vb, off, 4096)
#define MM_(c, a, b) "v_mfma_f32_32x32x64_f8f6f4 %[" #c "], v[" #a "], v[" #b "], %[" #c "]\n\t"
            asm volatile(
                "s_mov_b32 %[cnt], 7\n"
                "1:\n\t"
                POLL_ADDR
                "ds_read_b32 %[t0], %[t0]\n\ts_waitcnt lgkmcnt(0)\n\t"
                "v_cmp_le_u32 vcc, 0x01010000, %[t0]\n\ts_cmp_eq_u64 vcc, -1\n\ts_cbranch_scc1 3f\n\t"
                "s_memrealtime %[now]\n\ts_waitcnt lgkmcnt(0)\n\t"
                "s_mov_b64 vcc, %[now]\n\ts_sub_u32 %[tmp], vcc_lo, %[ts]\n\t"
                "s_cmp_gt_u32 %[tmp], %[dl]\n\ts_cbranch_scc0 1b\n\t"
                "v_mov_b32 %[t0], %[bad]\n\tv_mov_b32 %[t1], 1\n\tds_write_b32 %[t0], %[t1]\n\t"
                "v_mov_b32 %[t0], %[prog]\n\tv_mov_b32 %[t1], 1000\n\tds_write_b32 %[t0], %[t1]\n\t"
                "s_branch 9f\n"
                "3:\n\t"
                MOVE
                RD_B0(0) RD_A0 RD_A1
                "\n4:\n\t"
                RD_B1(1024)
                "s_add_u32 %[m], %[m], 1\n\ts_add_u32 %[rel], %[m], %[nsb]\n\ts_sub_u32 %[cnt], %[cnt], 1\n\t"
                "s_cmp_eq_u32 %[cnt], 0\n\ts_cselect_b32 %[rel], 1000, %[rel]\n\t"
                "v_mov_b32 %[t0], %[prog]\n\tv_mov_b32 %[t1], %[rel]\n\t"
                "s_waitcnt lgkmcnt(4)\n\t"
                MM_(c00, 136:143, 152:159)
                "s_waitcnt lgkmcnt(2)\n\t"
                MM_(c10, 144:151, 152:159)
                "ds_write_b32 %[t0], %[t1]\n\t"
                RD_B0(2048)
                POLL_ADDR
                "s_waitcnt lgkmcnt(3)\n\t"
                MM_(c01, 136:143, 160:167)
                "ds_read_b32 %[t0], %[t0]\n\t"
                MM_(c11, 144:151, 160:167)
                RD_B1(3072)
                "s_waitcnt lgkmcnt(3)\n\t"
                MM_(c02, 136:143, 152:159)
                "s_waitcnt lgkmcnt(2)\n\t"
                "v_cmp_le_u32 vcc, 0x01010000, %[t0]\n\t"
                MM_(c12, 144:151, 152:159)
                "s_cmp_eq_u32 %[cnt], 0\n\ts_cbranch_scc1 6f\n\t"
                "s_cmp_eq_u64 vcc, -1\n\ts_cbranch_scc0 6f\n\t"
                MOVE
                RD_B0(0)
                "s_waitcnt lgkmcnt(2)\n\t"
                MM_(c03, 136:143, 160:167)
                RD_A0
                MM_(c13, 144:151, 160:167)
                RD_A1
                "s_branch 4b\n"
                "6:\n\t"
                "s_waitcnt lgkmcnt(0)\n\t"
                MM_(c03, 136:143, 160:167)
                MM_(c13, 144:151, 160:167)
                "s_cmp_eq_u32 %[cnt], 0\n\ts_cbranch_scc0 1b\n"
                "9:\n\t"
                "s_waitcnt lgkmcnt(0)\n\ts_nop 15\n\ts_nop 15"
                : [c00] "+v"(acc[0][0]), [c01] "+v"(acc[0][1]), [c02] "+v"(acc[0][2]), [c03] "+v"(acc[0][3]),
                  [c10] "+v"(acc[1][0]), [c11] "+v"(acc[1][1]), [c12] "+v"(acc[1][2]), [c13] "+v"(acc[1][3]),
                  [va] "+v"(va), [vb] "+v"(vb), [t0] "=&v"(t0), [t1] "=&v"(t1),
                  [m] "+s"(m), [blk] "+s"(blk), [cnt] "=&s"(cnt), [tmp] "=&s"(tmp), [rel] "=&s"(rel), [now] "=&s"(now)
                : [fin] "s"(lb + L_FIN), [prog] "s"(lb + L_PROG + wave * 4), [bad] "s"(lb + L_BAD), [ts] "s"(tstart),
                  [dl] "s"(DEADLINE2), [nsb] "n"(NSB), [nxb] "n"(NXB), [fwd] "s"(CH), [back] "n"(-(NXB - 1) * CH)
                : "memory", "vcc", "scc",
                  "v136", "v137", "v138", "v139", "v140", "v141", "v142", "v143", "v144", "v145", "v146", "v147",
                  "v148", "v149", "v150", "v151", "v152", "v153", "v154", "v155", "v156", "v157", "v158", "v159",
                  "v160", "v161", "v162", "v163", "v164", "v165", "v166", "v167");
        }
        if (wc == 0) STAMP(20 + wr);
        const bool bad = lds_ld(L, L_BAD) != 0u;
        const float mul = bad ? __builtin_nanf("") : scale;
        const unsigned oo = (va & 0x800u) * 32u + (va & 0x1f0u);
#pragma unroll
        for (int m = 0; m < 2; ++m)
#pragma unroll
            for (int e = 0; e < 16; ++e) {
                v4f v = {acc[m][0][e] * mul, acc[m][1][e] * mul, acc[m][2][e] * mul, acc[m][3][e] * mul};
                st16_sys(otile + (size_t)(m * 32 + (e & 3) + 8 * (e >> 2)) * NN * 4, oo, __builtin_bit_cast(v4u, v));
                if (OWIN < 32 && wr < 3) wait_vm<OWIN>();
                if (OSL > 0 && m * 16 + e >= OFREE && wr < OGRP) __builtin_amdgcn_s_sleep(OSL);
            }
        return;
    }

    __builtin_amdgcn_s_setprio(3);
    if (wave == 10) {
        const int d = (dtab >> (4 * sl)) & 7;
        const int ord = q ^ 16;
        const unsigned per = (sl & 4) ? PERX8 : PER8;
        const unsigned t0 = tstart + (unsigned)(LEAD) + ord * per / 256;
        const int late1 = ((sl ^ xcc) & 4) ? FARLATE : LATE1;
        if (q < 16) send_panel<false>(px + (size_t)((q >> 1) * 512 + (q & 1) * 128) * KS, L, myws, dws, rank, q, d, lane, t0, per, late1);
        else send_panel<true>(pw + (q - 16) * 256, L, myws, dws, rank, q, d, lane, t0, per, late1);
        return;
    }

    if (wave == 11) {
        __builtin_amdgcn_s_barrier();
        FENCE();
        const unsigned call = rfl(lds_ld(L, L_CALL));
        const unsigned fbase = (call - 1u) * FSTRIDE;
        int ti;
        {
            while ((unsigned)rt() - tstart < VPOLL) __builtin_amdgcn_s_sleep(8);
            unsigned v;
            for (;;) {
                asm volatile("s_mov_b64 exec, 0xff\n\tglobal_atomic_or %0, %1, %2, %3 sc0 sc1\n\ts_mov_b64 exec, -1\n\ts_waitcnt vmcnt(0)"
                             : "=&v"(v) : "v"((unsigned)WS_FLAG + (16 + tj) * FSTR + 64 + lane * 4), "v"(0u), "s"(myws) : "memory");
                if ((__ballot((v ^ call << 8) < 256u) & 0xffu) == 0xffu) break;
                if ((unsigned)rt() - tstart > DEADLINE) { lds_st(L, L_BAD, 1u); break; }
                __builtin_amdgcn_s_sleep(8);
            }
            int sum = v & 255u;
            sum += dpp0<0x111>(sum); sum += dpp0<0x112>(sum); sum += dpp0<0x114>(sum);
            unsigned c = __builtin_amdgcn_readlane(sum, 7) > 8 * 128;
            ti = (xcc & 2) * 4 + (idx >> 3) * 2 + ((xcc >> 2 ^ xcc ^ c) & 1);
            lds_st(L, L_TI, ti + 1);
            STAMP(8);
        }
        const int d = lane & 7;
        const unsigned srcr = (unsigned)(rank + d) & 7u;
        const unsigned foff = WS_FLAG + (lane & 8 ? 16 + tj : ti) * FSTR + ((lane & 8 ? ti : tj) & (FREP - 1)) * FLINE + srcr * 4;
        const unsigned row0 = srcr * NCH;
        const int nmax = lane >= 8 ? 0 : d == 0 ? NCH : NBULK, amax = lane >= 8 || d == 0 ? 0 : 8;
        unsigned it = 0;
        int seen = 0, qn = 0;
        int seen2 = 0, fq = 0, glim = 2;
#define POLL(r) asm volatile("s_mov_b64 exec, %3\n\tglobal_atomic_or " r ", %0, %1, %2 sc0 sc1\n\ts_mov_b64 exec, -1" \
                             :: "v"(foff), "v"(0u), "s"(myws), "s"(wdone ? 0xffull : 0xffffull) : "memory", r)
#define TAKE(v, r, n) asm volatile("s_waitcnt vmcnt(" #n ")\n\tv_mov_b32 %0, " r : "=v"(v) :: "memory")
        bool wdone = false;
        auto process = [&](unsigned v) {
            bool dead = false;
            if ((++it & 15u) == 0u) {
                dead = rfl(__hip_atomic_load((const unsigned*)(myws + WS_TMO), __ATOMIC_RELAXED, __HIP_MEMORY_SCOPE_AGENT)) == call;
                if (!dead && (unsigned)rt() - tstart > DEADLINE) {
                    if (lane == 0) __hip_atomic_store((unsigned*)(myws + WS_TMO), call, __ATOMIC_RELAXED, __HIP_MEMORY_SCOPE_AGENT);
                    dead = true;
                }
            }
            int nx = (int)(v - fbase), nw = dpp0<0x128>(nx);
            if (wdone) nw = NCH;
            if (dead) { nx = nw = 64; lds_st(L, L_BAD, 1u); }
            int nn = min(min(nx, nw), nmax) - seen;
            nn = max(nn, 0);
            if (__ballot(nn > 0)) {
                int pre = nn;
                pre += dpp0<0x111>(pre); pre += dpp0<0x112>(pre); pre += dpp0<0x114>(pre);
                const int pos = qn + pre - nn;
                FENCE();
                for (int j = 0; j < nn; ++j) lds_st(L, L_QUEUE + (pos + j) * 4, row0 + seen + j + 1u);
                seen += nn;
                qn += __builtin_amdgcn_readlane(pre, 7);
                it = 0;
            }
            int a = min(nx - NBULK, amax);
            if (nw < NCH) a = min(a, (nw - NBULK) >> 1);
            for (;;) {
                const int n2 = max(min(a, glim) - seen2, 0);
                if (!__ballot(n2 > 0)) break;
                int pre = n2;
                pre += dpp0<0x111>(pre); pre += dpp0<0x112>(pre); pre += dpp0<0x114>(pre);
                const int pos = fq + pre - n2;
                FENCE();
                for (int j = 0; j < n2; ++j) {
                    const unsigned pr = (seen2 + j) & 1;
                    *(volatile LAS unsigned short*)(L + L_FIN + (pos + j) * 4) =
                        (unsigned short)((2 * d - 2 + pr) << 12 | (row0 + NBULK + ((seen2 + j) >> 1)));
                }
                seen2 += n2;
                fq += __builtin_amdgcn_readlane(pre, 7);
                it = 0;
                if (fq != glim * (NR - 1)) break;
                STAMP(31 + glim / 2);
                glim += 2;
            }
            wdone = qn >= NQ && fq >= NPAIR;
            return qn >= NQ && fq >= NFIN;
        };
        unsigned v;
        bool end = false;
        while (!end && (PD == 1 || qn < NQ - PFAST)) {
            POLL("v160"); TAKE(v, "v160", 0);
            end = process(v);
            if (qn >= NQ - PFAST) __builtin_amdgcn_s_sleep(POLLSLEEP2); else __builtin_amdgcn_s_sleep(POLLSLEEP);
        }
        STAMP(10);
        return;
    }

    {
        const int ld = wave - 8;
        __builtin_amdgcn_s_barrier();
        FENCE();
        const unsigned voff = lane * 16;
        unsigned spin = 0, ti;
        while ((ti = rfl(lds_ld(L, L_TI))) == 0u) {
            if (expired(spin, tstart)) { lds_st(L, L_BAD, 1u); break; }
            __builtin_amdgcn_s_sleep(4);
        }
        ti = (ti - 1u) & 15u;
        const unsigned char* dbase = myws + WS_DATA + (size_t)(rfl(lds_ld(L, L_CALL)) & (NPAR - 1u)) * PAR_BYTES + (size_t)(ld ? 16 + tj : ti) * CH;
        const unsigned lb = (unsigned)(uintptr_t)L;
        int iss = 0, done = 0;
        while (done < NQ) {
            const unsigned v = lds_ld(L, lane < 16 ? L_PROG + (lane & 7) * 4 : L_QUEUE + (iss + (lane & 15)) * 4);
            int lim = iss + __builtin_ctzll(~(__ballot(v != 0u) >> 16));
            lim = min(min(lim, iss + 16), min(NQ, done + 15));
            const int st = iss >> 2;
            int fr = st + NSLOT;
#pragma unroll
            for (int j = NSLOT - 1; j >= 0; --j) if (__ballot(lane < 8 && (int)v < st + j - (NSLOT - 1))) fr = st + j;
            lim = min(lim, fr * 4);
            FENCE();
            const bool any = iss < lim;
            for (const int i0 = iss; iss < lim; ++iss) {
                const unsigned e = __builtin_amdgcn_readlane(v, 16 + iss - i0) - 1u;
                dma4k(dbase + (size_t)e * ROW_BYTES, lb + ld * 16384 + ((iss >> 2) & (NSLOT - 1)) * SLOT + (iss & 3) * CH, voff);
            }
            const int tg = iss - ((vm_count() + 3) >> 2);
            if (tg > done) {
                FENCE();
                done = tg;
                lds_st(L, L_FULL + ld * 4, (unsigned)done);
            } else if (!any) {
                if (expired(spin, tstart)) { lds_st(L, L_BAD, 1u); break; }
                __builtin_amdgcn_s_sleep(1);
            }
        }
        auto fin = [&](auto isw) {
            constexpr bool W = decltype(isw)::value;
            constexpr int NF = W ? NPAIR : NFIN;
            unsigned xa = lb + L_XR;
            unsigned xd = xa + NPAIR * 2048;
            const unsigned co = voff + (lane >> 4) * 4;
            iss = 0; done = 0;
            while (done < NF) {
                const unsigned v = lds_ld(L, lane < 16 ? L_PROG + (lane & 7) * 4 : L_FIN + ((lane < 32 ? iss : done) + (lane & 15)) * 4);
                int n = __builtin_ctz(~(unsigned)(__ballot((v & 0xfffu) != 0u) >> 16) | (1u << (W ? WBATCH : XBATCH)));
                int pm = (int)v;
                pm = min(pm, dpp0<0x121>(pm)); pm = min(pm, dpp0<0x122>(pm)); pm = min(pm, dpp0<0x124>(pm));
                pm = (int)rfl(pm) - NSB;
                int lim = W ? done + 7 : 2 * (pm + NXB);
                lim = lim < NF ? lim : NF;
                lim = pm < 0 ? 0 : lim - iss;
                n = n < lim ? n : lim;
                const unsigned off = (v & 0x1ffu) << 17 | (W ? (v & 0x1000u) << 6 : (v & 0x1000u) >> 1);
                FENCE();
                for (int j = 0; j < n; ++j) {
                    const unsigned vo = voff + __builtin_amdgcn_readlane(off, 16 + j);
                    if constexpr (W) {
                        const unsigned la = lb + (__builtin_amdgcn_readlane(v, 16 + j) & 0xf000u) * 2;
                        dma4k(dbase, la, vo);
                        dma4k(dbase + ROW_BYTES, la + CH, vo);
                    } else {
                        dma2k(dbase, xa, vo);
                        xa = xa == lb + L_CTL - 2048 ? lb + L_XR : xa + 2048;
                    }
                }
                if (n > 0) iss += n;
                const int vm = vm_count();
                int tg = iss - (W ? (vm + 7) >> 3 : (vm + 1) >> 1);
                if (tg > done) {
                    FENCE();
#define NOVAL 0
                    if ((NOVAL | ~SPEC) & (W ? 2 : 1)) {}
                    else if (!W && done < NPAIR) tg = min(tg, NPAIR);
                    else {
                        tg = min(tg, done + 4);
                        if (!W) tg = min(tg, done + (int)((lb + L_CTL - xd) >> 11));
                        v2u t[8];
                        unsigned mn = 1u;
                        if constexpr (W) {
#pragma unroll
                            for (int j = 0; j < 4; ++j) {
                                const unsigned a = co + lb + (__builtin_amdgcn_readlane(v, 32 + min(j, tg - done - 1)) & 0xf000u) * 2;
                                lds_ld2<4>(t[2 * j], a); lds_ld2<6>(t[2 * j + 1], a);
                            }
                            asm volatile("s_waitcnt lgkmcnt(0)" : "+v"(t[0]), "+v"(t[1]), "+v"(t[2]), "+v"(t[3]),
                                         "+v"(t[4]), "+v"(t[5]), "+v"(t[6]), "+v"(t[7]) :: "memory");
#pragma unroll
                            for (int j = 0; j < 8; ++j) mn = min3(mn, t[j]);
                        } else {
                            const unsigned a = co + xd;
                            lds_ld2<0>(t[0], a); lds_ld2<2>(t[1], a); lds_ld2<4>(t[2], a); lds_ld2<6>(t[3], a);
                            asm volatile("s_waitcnt lgkmcnt(0)" : "+v"(t[0]), "+v"(t[1]), "+v"(t[2]), "+v"(t[3]) :: "memory");
                            mn = min(t[0][0], t[0][1]);
                            if (tg > done + 1) { mn = min3(mn, t[1]); FENCE();
                                if (tg > done + 2) { mn = min3(mn, t[2]); FENCE();
                                    if (tg > done + 3) { mn = min3(mn, t[3]); FENCE(); } } }
                        }
                        if (__ballot(mn == 0u)) {
                            for (int j = 0; j < tg - done; ++j) {
                                const unsigned la = W ? lb + (__builtin_amdgcn_readlane(v, 32 + j) & 0xf000u) * 2 + CH : xd + j * 2048;
                                const unsigned vo = voff + __builtin_amdgcn_readlane(off, 32 + j);
                                for (;;) {
                                    lds_ld2<0>(t[0], co + la); lds_ld2<W ? 2 : 0>(t[1], co + la);
                                    asm volatile("s_waitcnt lgkmcnt(0)" : "+v"(t[0]), "+v"(t[1]) :: "memory");
                                    if (!__ballot(min3(min(t[0][0], t[0][1]), t[1]) == 0u) || rfl(lds_ld(L, L_BAD))) break;
                                    if ((unsigned)rt() - tstart > DEADLINE2) { lds_st(L, L_BAD, 1u); break; }
                                    dma2k_sys(W ? dbase + ROW_BYTES : dbase, la, vo);
                                    if (W) dma2k_sys(dbase + ROW_BYTES + 2048, la + 2048, vo);
                                    wait_vm<0>();
                                }
                            }
                        }
                        if (!W) { xd += (tg - done) * 2048; xd = xd == lb + L_CTL ? lb + L_XR : xd; }
                        FENCE();
                    }
                    const int last = tg == NF ? NFIN : tg;
                    if (done + lane < last) *(volatile LAS unsigned char*)(L + L_FIN + (done + lane) * 4 + 2 + W) = 1;
                    done = tg;
                } else if (n <= 0) {
                    if (expired(spin, tstart)) { lds_st(L, L_BAD, 1u); break; }
                    __builtin_amdgcn_s_sleep(1);
                }
            }
        };
        if (ld) fin(std::true_type{}); else fin(std::false_type{});
        STAMP(11 + 2 * ld);
    }
}

}

extern "C" void kernel_launch(int rank, int nranks, ncclComm_t comm, hipStream_t stream,
                              void* const* d_in, const int* in_sizes, int n_in,
                              void* d_out, int out_size, const int* in_dtypes, int out_dtype,
                              void* const* d_out_all, void* const* d_ws_all, size_t ws_bytes) {
    if (nranks != NR || n_in != 4 || ws_bytes < WS_NEED || in_sizes[0] != MM * KS || in_sizes[1] != KS * NN ||
        out_size != MM * NN || (unsigned)rank >= (unsigned)NR) {
        fprintf(stderr, "kernel_launch: unexpected problem geometry\n");
        return;
    }
    Params p;
    p.x = (const unsigned char*)d_in[0]; p.w = (const unsigned char*)d_in[1];
    p.sx = (const float*)d_in[2]; p.sw = (const float*)d_in[3];
    p.out = (float*)d_out; p.rank = rank;
    p.myws = (unsigned char*)d_ws_all[rank];
    const unsigned dtab[NR] = {0x14735602u, 0x73164502u, 0x13654207u, 0x74623501u, 0x16245307u, 0x75234601u, 0x15724306u, 0x75143206u};
    p.dtab = dtab[rank];
    for (int s = 0; s < NR; ++s) p.dws[s] = (unsigned char*)d_ws_all[(rank + NR - ((p.dtab >> (4 * s)) & 7)) & 7];
    fprintf(stderr, "PTRS rank %d out %p ws %p x %p w %p ws0 %p ws7 %p\n", rank, d_out, d_ws_all[rank], d_in[0], d_in[1], d_ws_all[0], d_ws_all[7]);
    hipLaunchKernelGGL(fused_gemm_ag, dim3(NWG), dim3(NTHREADS), 0, stream, p);
    HIP_CHECK(hipGetLastError());
}
```

```cpp
#include <hip/hip_runtime.h>
#include <rccl/rccl.h>
#include <cstdio>
#include <cstdint>
#include <type_traits>

#define HIP_CHECK(expr) do { hipError_t _e = (expr); if (_e != hipSuccess) { \
    fprintf(stderr, "HIP error %s at %s:%d\n", hipGetErrorString(_e), __FILE__, __LINE__); \
    return; } } while (0)

namespace {

#define LAS __attribute__((address_space(3)))
typedef int v4i __attribute__((ext_vector_type(4)));
typedef int v8i __attribute__((ext_vector_type(8)));
typedef unsigned v2u __attribute__((ext_vector_type(2)));
typedef unsigned v4u __attribute__((ext_vector_type(4)));
typedef float v4f __attribute__((ext_vector_type(4)));
typedef float v16f __attribute__((ext_vector_type(16)));
typedef unsigned long long u64;

constexpr int NR = 8;
constexpr int MM = 4096, NN = 4096, KS = 512;
constexpr int NCH = 32;
constexpr int NBULK = 28;
constexpr int NPANEL = 32;
constexpr int CH = 4096;
constexpr int NWG = 256, NTHREADS = 768;
constexpr int NQ = (NR - 1) * NBULK + NCH;
constexpr int NSB = NQ / 4;
constexpr int NPAIR = 2 * (NR - 1);
constexpr int NFIN = 4 * NPAIR;
constexpr unsigned FSTRIDE = 64;
constexpr size_t WS_CTR = 0;
constexpr int FSTR = 128;
constexpr size_t WS_FLAG = 524288;
constexpr size_t WS_TMO = 16384 + 4096;
constexpr size_t WS_ACC = 24576;
constexpr size_t WS_DATA = 2u << 20;
constexpr size_t PAN_BYTES = (size_t)NCH * CH;
constexpr size_t SRC_BYTES = (size_t)NPANEL * PAN_BYTES;
constexpr size_t PAR_BYTES = (size_t)NR * SRC_BYTES;
constexpr unsigned NPAR = 2;
constexpr size_t WS_NEED = WS_DATA + NPAR * PAR_BYTES;
constexpr unsigned DEADLINE = 40000000u;
constexpr unsigned DEADLINE2 = 60000000u;
constexpr int NSLOT = 4, SLOT = 32768;
constexpr int L_XR = NPAIR * 2 * CH;
constexpr int NXB = 11;
constexpr int L_CTL = L_XR + NXB * CH;
constexpr int L_FULL = L_CTL;
constexpr int L_PROG = L_CTL + 64;
constexpr int L_BAD = L_CTL + 128;
constexpr int L_TI = L_CTL + 192;
constexpr int L_VOTE = L_CTL + 196;
constexpr int L_QUEUE = L_CTL + 256;
constexpr int L_FIN = L_CTL + 1536;
constexpr int L_CALL = L_CTL + 2048;
constexpr int LDS_BYTES = L_CALL + 64;
static_assert(L_QUEUE + (NQ + 32) * 4 <= L_FIN && L_FIN + (NFIN + 32) * 4 <= L_CALL && L_CTL >= NSLOT * SLOT, "LDS map");

constexpr unsigned PER8 = 438;
constexpr unsigned PERX8 = 439;
constexpr unsigned LEAD = 0;
constexpr unsigned XDELAY = 200;
constexpr int CATCH = 20;
constexpr int RBI = 2, RB2 = 5;
constexpr int RBC = 160;
constexpr unsigned VSTART = 200;
constexpr unsigned VPOLL = 450;
constexpr int XBATCH = 8, WBATCH = 4;

struct Params {
    const unsigned char* x; const unsigned char* w; unsigned char* myws;
    int rank;
    unsigned dtab;
    const float* sx; const float* sw; float* out;
    unsigned char* dws[NR];
};

__device__ __forceinline__ int ppos(int q) { return (q & 12) * 2 + (q >> 4) * 4 + (q & 3); }
__device__ __forceinline__ u64 rt() { return __builtin_amdgcn_s_memrealtime(); }
__device__ __forceinline__ unsigned rfl(unsigned v) { return __builtin_amdgcn_readfirstlane(v); }
__device__ __forceinline__ unsigned lds_ld(LAS unsigned char* L, int off) { return *(volatile LAS unsigned*)(L + off); }
__device__ __forceinline__ void lds_st(LAS unsigned char* L, int off, unsigned v) { *(volatile LAS unsigned*)(L + off) = v; }
#define FENCE() asm volatile("" ::: "memory")
template <int N> __device__ __forceinline__ void wait_vm() { asm volatile("s_waitcnt vmcnt(%0)" :: "n"(N) : "memory"); }
__device__ __forceinline__ int vm_count() {
    unsigned r;
    asm volatile("s_getreg_b32 %0, hwreg(HW_REG_IB_STS)" : "=s"(r) :: "memory");
    return (int)((r & 15u) | ((r >> 18) & 48u));
}
__device__ __forceinline__ bool expired(unsigned& spin, unsigned t0) {
    return (++spin & 4095u) == 0u && (unsigned)rt() - t0 > DEADLINE2;
}
template <int CTRL> __device__ __forceinline__ int dpp0(int v) { return __builtin_amdgcn_update_dpp(0, v, CTRL, 0xf, 0xf, true); }

__device__ __forceinline__ void dma4k(const unsigned char* g, unsigned ldsaddr, unsigned voff) {
    asm volatile("s_mov_b32 m0, %2\n\ts_nop 0\n\t"
                 "global_load_lds_dwordx4 %0, %1 offset:0\n\tglobal_load_lds_dwordx4 %0, %1 offset:1024\n\t"
                 "global_load_lds_dwordx4 %0, %1 offset:2048\n\tglobal_load_lds_dwordx4 %0, %1 offset:3072"
                 :: "v"(voff), "s"(g), "s"(ldsaddr) : "memory");
}
__device__ __forceinline__ void dma2k(const unsigned char* g, unsigned ldsaddr, unsigned voff) {
    asm volatile("s_mov_b32 m0, %2\n\ts_nop 0\n\t"
                 "global_load_lds_dwordx4 %0, %1 offset:0\n\tglobal_load_lds_dwordx4 %0, %1 offset:1024"
                 :: "v"(voff), "s"(g), "s"(ldsaddr) : "memory");
}
__device__ __forceinline__ void st16_sys(unsigned char* sbase, unsigned voff, v4u v) {
    asm volatile("global_store_dwordx4 %0, %1, %2 sc0 sc1\n\ts_nop 1" :: "v"(voff), "v"(v), "s"(sbase) : "memory");
}
__device__ __forceinline__ void st4_sys_lane0(unsigned* p, unsigned v) {
    asm volatile("s_mov_b64 exec, 1\n\tglobal_atomic_swap %0, %1, %2 sc1\n\ts_mov_b64 exec, -1" :: "v"(0u), "v"(v), "s"(p) : "memory");
}
__device__ __forceinline__ void ld4(unsigned& d, unsigned voff, const unsigned char* sbase) {
    asm volatile("global_load_dword %0, %1, %2" : "=v"(d) : "v"(voff), "s"(sbase) : "memory");
}
template <int OFF> __device__ __forceinline__ void ld16(v4u& d, unsigned voff, const unsigned char* sbase) {
    asm volatile("global_load_dwordx4 %0, %1, %2 offset:%3" : "=v"(d) : "v"(voff), "s"(sbase), "n"(OFF) : "memory");
}
__device__ __forceinline__ void landed(unsigned (&g)[16]) {
    asm volatile("" : "+v"(g[0]), "+v"(g[1]), "+v"(g[2]), "+v"(g[3]), "+v"(g[4]), "+v"(g[5]), "+v"(g[6]), "+v"(g[7]),
                      "+v"(g[8]), "+v"(g[9]), "+v"(g[10]), "+v"(g[11]), "+v"(g[12]), "+v"(g[13]), "+v"(g[14]), "+v"(g[15]) :: "memory");
}
__device__ __forceinline__ void landed(v4u& a, v4u& b, v4u& c, v4u& d) {
    asm volatile("" : "+v"(a), "+v"(b), "+v"(c), "+v"(d) :: "memory");
}
__device__ __forceinline__ void swap16(v4u& a, v4u& b, unsigned mask) {
    unsigned x0 = a[0], x1 = a[1], x2 = a[2], x3 = a[3], y0 = b[0], y1 = b[1], y2 = b[2], y3 = b[3];
    asm volatile("s_mov_b32 exec_lo, %8\n\ts_mov_b32 exec_hi, %8\n\tv_swap_b32 %0, %4\n\tv_swap_b32 %1, %5\n\tv_swap_b32 %2, %6\n\tv_swap_b32 %3, %7\n\t"
                 "s_mov_b64 exec, -1"
                 : "+v"(x0), "+v"(x1), "+v"(x2), "+v"(x3), "+v"(y0), "+v"(y1), "+v"(y2), "+v"(y3) : "s"(mask));
    a[0] = x0; a[1] = x1; a[2] = x2; a[3] = x3; b[0] = y0; b[1] = y1; b[2] = y2; b[3] = y3;
}

template <bool IS_W>
__device__ __forceinline__ void send_panel(const unsigned char* src, LAS unsigned char* L, unsigned char* myws, unsigned char* dws,
                                           int rank, int q, int d, int lane, unsigned t0, unsigned per, int late1, unsigned org) {
    const bool self = d == 0;
    const unsigned lo = IS_W ? lane * 4 : (lane >> 2) * 4 * KS + (lane & 3) * 16;
    const unsigned so = lane * 16;
    unsigned gw[4][16];
    v4u ow[2][4];
    v4u gx[2][16];
    auto loadw = [&](int i, unsigned (&g)[16]) {
#pragma unroll
        for (int k = 0; k < 16; ++k) ld4(g[k], lo, src + (size_t)(i * 16 + k) * NN);
    };
    auto xformw = [&](unsigned (&g)[16], v4u (&ob)[4]) {
        landed(g);
#pragma unroll
        for (int t = 0; t < 4; ++t) {
            const unsigned a0 = g[4 * t], a1 = g[4 * t + 1], a2 = g[4 * t + 2], a3 = g[4 * t + 3];
            const unsigned x0 = __builtin_amdgcn_perm(a1, a0, 0x05010400u), x1 = __builtin_amdgcn_perm(a1, a0, 0x07030602u);
            const unsigned y0 = __builtin_amdgcn_perm(a3, a2, 0x05010400u), y1 = __builtin_amdgcn_perm(a3, a2, 0x07030602u);
            ob[0][t] = __builtin_amdgcn_perm(y0, x0, 0x05040100u);
            ob[1][t] = __builtin_amdgcn_perm(y0, x0, 0x07060302u);
            ob[2][t] = __builtin_amdgcn_perm(y1, x1, 0x05040100u);
            ob[3][t] = __builtin_amdgcn_perm(y1, x1, 0x07060302u);
        }
        landed(ob[0], ob[1], ob[2], ob[3]);
    };
    auto loadx = [&](int grp, int half, v4u (&g)[16]) {
#pragma unroll
        for (int j = 2 * half; j < 2 * half + 2; ++j) {
            const unsigned char* s = src + (size_t)((j & 1) * 64 + (j >> 1) * 256) * KS + grp * 64;
            ld16<0>(g[4 * j], lo, s); ld16<KS>(g[4 * j + 1], lo, s); ld16<2 * KS>(g[4 * j + 2], lo, s); ld16<3 * KS>(g[4 * j + 3], lo, s);
        }
    };
    const bool b0 = lane & 1, b1 = lane & 2;
    auto xformx = [&](int half, v4u (&g)[16]) {
#pragma unroll
        for (int j = 2 * half; j < 2 * half + 2; ++j) {
            v4u &r0 = g[4 * j], &r1 = g[4 * j + 1], &r2 = g[4 * j + 2], &r3 = g[4 * j + 3];
            landed(r0, r1, r2, r3);
#pragma unroll
            for (int e = 0; e < 4; ++e) {
                const unsigned a0 = r0[e], a1 = r1[e], a2 = r2[e], a3 = r3[e];
                const unsigned u0 = __builtin_amdgcn_mov_dpp(a0, 0xB1, 0xf, 0xf, true), u1 = __builtin_amdgcn_mov_dpp(a1, 0xB1, 0xf, 0xf, true);
                const unsigned u2 = __builtin_amdgcn_mov_dpp(a2, 0xB1, 0xf, 0xf, true), u3 = __builtin_amdgcn_mov_dpp(a3, 0xB1, 0xf, 0xf, true);
                const unsigned n0 = b0 ? u1 : a0, n1 = b0 ? a1 : u0, n2 = b0 ? u3 : a2, n3 = b0 ? a3 : u2;
                const unsigned w0 = __builtin_amdgcn_mov_dpp(n0, 0x4E, 0xf, 0xf, true), w1 = __builtin_amdgcn_mov_dpp(n1, 0x4E, 0xf, 0xf, true);
                const unsigned w2 = __builtin_amdgcn_mov_dpp(n2, 0x4E, 0xf, 0xf, true), w3 = __builtin_amdgcn_mov_dpp(n3, 0x4E, 0xf, 0xf, true);
                r0[e] = b1 ? w2 : n0; r2[e] = b1 ? n2 : w0; r1[e] = b1 ? w3 : n1; r3[e] = b1 ? n3 : w1;
            }
            landed(r0, r1, r2, r3);
        }
    };
    unsigned* ctr = (unsigned*)(myws + WS_CTR + (size_t)blockIdx.x * 64);
    unsigned prevv;
    asm volatile("global_load_dword %0, %1, %2 sc1" : "=v"(prevv) : "v"(0u), "s"(ctr) : "memory");
    if constexpr (IS_W) { loadw(0, gw[0]); loadw(1, gw[1]); } else { loadx(0, 0, gx[0]); loadx(0, 1, gx[0]); }
    asm volatile("s_waitcnt vmcnt(%1)" : "+v"(prevv) : "n"(IS_W ? 32 : 16) : "memory");
    const unsigned prev = rfl(prevv), call = prev + 1u, fbase = prev * FSTRIDE;
    lds_st(L, L_CALL, call);
    asm volatile("s_waitcnt lgkmcnt(0)" ::: "memory");
    __builtin_amdgcn_s_barrier();
    FENCE();
    unsigned char* const db0 = dws + WS_DATA + (size_t)(call & (NPAR - 1u)) * PAR_BYTES + (size_t)rank * SRC_BYTES + (size_t)ppos(q) * PAN_BYTES;
    unsigned char* db = db0 + q * CH;
    unsigned char* acc = myws + WS_ACC + (call & 1u) * 1024 + d * 128;
    unsigned* dflag = (unsigned*)(dws + WS_FLAG + q * FSTR + rank * 4);
    unsigned fm = (d & 1) ? 0x05555555u : 0x0AAAAAAAu;
    if (!self) fm = (fm & 0x0F000000u) | (0x00111111u << (d & 3));
    const unsigned fmask = fm | 0x08000000u | ((IS_W || self) ? 0xA0000000u : 0xF0000000u);
    if constexpr (IS_W) { loadw(2, gw[2]); wait_vm<32>(); xformw(gw[0], ow[0]); }
    else { wait_vm<8>(); xformx(0, gx[0]); wait_vm<0>(); xformx(1, gx[0]); }
    int ops = IS_W ? 32 : 0, h1 = 0, h2 = 0, h3 = 0, h4 = 0, h5 = 0, h6 = 0;
    int l1 = IS_W ? 16 : 0, l2 = IS_W ? 32 : 0, l3 = 0;
    int ia = 0, ib = 0;
    unsigned fa = 0, fb = 0;
    unsigned tt = 0, period = self ? 0u : per;
    bool wide = self;
    int slip = 0;
    auto flag_try = [&](int ret) {
        if (ia != 0 && ret >= ia) { st4_sys_lane0(dflag, fa); ++ops; ia = ib; fa = fb; ib = 0; }
    };
    auto flag_put = [&](unsigned v) {
        if (ia == 0) { ia = h1; fa = v; } else { ib = h1; fb = v; }
    };
    auto slot = [&](unsigned char* a, v4u& v) {
        const int n0 = wide ? h4 + 0 : h2 + 0, n1 = wide ? h6 + 0 : h3 + 0;
        const unsigned due = t0 + (tt >> 3);
        int late;
        for (;;) {
            late = (int)((unsigned)rt() - due);
            const int ret = ops - vm_count();
            flag_try(ret);
            if (late >= slip && ret >= (late >= late1 ? n1 : n0)) break;
            __builtin_amdgcn_s_sleep(1);
        }
        slip = wide ? 0 : max(late - CATCH, 0);
        st16_sys(a, so, v);
        h6 = h5; h5 = h4; h4 = h3; h3 = h2; h2 = h1; h1 = ++ops;
        tt += period;
    };
    auto piece = [&](int i, v4u& s0, v4u& s1, v4u& s2, v4u& s3, auto&& loads, auto&& xform) {
        if (i == RBI && !self) {
            t0 += max((int)((unsigned)rt() - t0 - ((tt - per) >> 3)) - 10, 0); slip = 0;
            org = (t0 - org) & 0xfffffu;
            asm volatile("s_mov_b64 exec, 1\n\tglobal_atomic_add %0, %1, %2 sc1\n\ts_mov_b64 exec, -1"
                         :: "v"(0u), "v"(org | 1u << 26), "s"(acc) : "memory");
            ++ops;
            if (q == 0) { st4_sys_lane0((unsigned*)((u64)acc ^ 1024u), 0u); ++ops; }
        }
        if (i == RB2 && !self) {
            unsigned sum;
            asm volatile("s_load_dword %0, %1, 0x0 glc\n\ts_waitcnt lgkmcnt(0)" : "=s"(sum) : "s"(acc) : "memory");
            const int dl = (int)((((sum & 0x3ffffffu) >> 5) - org) << 12) >> 12;
            if (sum >> 26 == NPANEL) period = per + min(max(dl, -RBC), RBC) / 8;
        }
        if (i == RB2 + 16 && !self) period = per;
        if (i == NBULK + 1 && !self) {
            t0 += max((int)((unsigned)rt() - t0 - ((tt - per) >> 3)) - 10, 0);
            if (!IS_W) tt += 6u * per - per / 2 + XDELAY;
            period = per / 2; wide = true; slip = 0;
        }
        const unsigned fv = fbase + ((IS_W || self || i < NBULK) ? i + 1 : 2 * i - (NBULK - 2));
        slot(db, s0);
        slot(db + 1024, s1);
        if (!IS_W && !self && i > NBULK) flag_put(fv - 1u);
        loads();
        slot(db + 2048, s2);
        xform();
        slot(db + 3072, s3);
        if ((fmask >> i) & 1u) flag_put(fv);
        db = db0 + ((i + 1) ^ q) * CH;
    };
    auto arrived = [&](int l) { while (ops - vm_count() < l) __builtin_amdgcn_s_sleep(1); };
    if constexpr (IS_W) {
        auto wpiece = [&](int i, v4u (&oc)[4], v4u (&on)[4], unsigned (&gl)[16], unsigned (&gt)[16]) {
            piece(i, oc[0], oc[1], oc[2], oc[3],
                  [&]() { if (i + 3 < NCH) { loadw(i + 3, gl); ops += 16; } l3 = ops; },
                  [&]() { arrived(l1); xformw(gt, on); l1 = l2; l2 = l3; });
        };
#pragma unroll 1
        for (int i = 0; i < NCH; i += 4) {
            wpiece(i, ow[0], ow[1], gw[3], gw[1]);
            wpiece(i + 1, ow[1], ow[0], gw[0], gw[2]);
            wpiece(i + 2, ow[0], ow[1], gw[1], gw[3]);
            wpiece(i + 3, ow[1], ow[0], gw[2], gw[0]);
        }
    } else {
        auto group = [&](int g, v4u (&c)[16], v4u (&n)[16]) {
            const bool more = g + 1 < NCH / 4;
            piece(4 * g, c[0], c[4], c[8], c[12], [&]() { if (more) { loadx(g + 1, 0, n); ops += 8; l1 = ops; } }, [&]() {});
            piece(4 * g + 1, c[1], c[5], c[9], c[13], [&]() { if (more) { loadx(g + 1, 1, n); ops += 8; l2 = ops; } }, [&]() {});
            piece(4 * g + 2, c[2], c[6], c[10], c[14], [&]() {}, [&]() { arrived(l1); xformx(0, n); });
            piece(4 * g + 3, c[3], c[7], c[11], c[15], [&]() {}, [&]() { arrived(l2); xformx(1, n); });
        };
#pragma unroll 1
        for (int g = 0; g < NCH / 4; g += 2) {
            group(g, gx[0], gx[1]);
            const unsigned last = rfl(g == NCH / 4 - 2 && !self ? ~0u : 0u);
#pragma unroll
            for (int a = 0; a < 4; ++a)
#pragma unroll
                for (int c = a + 1; c < 4; ++c) swap16(gx[1][4 * a + c], gx[1][4 * c + a], last);
            group(g + 1, gx[1], gx[0]);
        }
    }
    while (ia != 0) flag_try(ops - vm_count());
    if (lane == 0) __hip_atomic_store(ctr, call, __ATOMIC_RELAXED, __HIP_MEMORY_SCOPE_AGENT);
}

__global__ void __launch_bounds__(NTHREADS) fused_gemm_ag(Params p) {
    __shared__ __attribute__((aligned(1024))) unsigned char lds_raw[LDS_BYTES];
    LAS unsigned char* L = (LAS unsigned char*)lds_raw;
    const int tid = threadIdx.x, lane = tid & 63;
    const int wave = rfl(tid >> 6);
    const int b = blockIdx.x, idx = b >> 3;
    unsigned xcc;
    asm volatile("s_getreg_b32 %0, hwreg(HW_REG_XCC_ID)" : "=s"(xcc));
    const int tj = (xcc & 1) * 8 + (idx & 7);
    const int q = (idx & 4) * 4 + (xcc & 3) * 4 + (idx & 3);
    int sl = (xcc & 4) + (idx >> 3);
    if (((idx >> 2) & 1) == (xcc & 1)) sl = sl == 1 ? 4 + (idx & 3) : sl == 4 + (idx & 3) ? 1 : sl;
    const unsigned char *px = p.x, *pw = p.w;
    unsigned char *myws = p.myws, *dws = p.dws[sl & 7];
    int rank = p.rank;
    unsigned dtab = p.dtab;
    unsigned tstart = (unsigned)rt();
    asm volatile("" : "+s"(px), "+s"(pw), "+s"(myws), "+s"(dws), "+s"(rank), "+s"(dtab), "+s"(tstart));

    if (wave < 8) {
        lds_st(L, L_CTL + wave * 256 + lane * 4, 0u);
        asm volatile("s_waitcnt lgkmcnt(0)" ::: "memory");
        __builtin_amdgcn_s_barrier();
        FENCE();
        float scale, scale2;
        asm volatile("s_load_dword %0, %2, 0x0\n\ts_load_dword %1, %3, 0x0\n\ts_waitcnt lgkmcnt(0)"
                     : "=&s"(scale), "=&s"(scale2) : "s"(p.sx), "s"(p.sw) : "memory");
        scale *= scale2;
        scale = __builtin_bit_cast(float, rfl(__builtin_bit_cast(unsigned, scale)));
        const int wr = wave >> 1, wc = wave & 1, i = lane & 31, h = lane >> 5;
        unsigned char* otile = (unsigned char*)p.out;
        if (idx == 0) {
            const unsigned pg = (xcc & 7) * 4 + (wave & 3);
            const unsigned char* a = otile + (size_t)pg * (2u << 20) + wave * 1024;
            while ((unsigned)rt() - tstart < VSTART) __builtin_amdgcn_s_sleep(4);
            int best[2] = {1 << 20, 1 << 20};
#pragma unroll 1
            for (int r = 0; r <= 4; ++r)
#pragma unroll
                for (int k = 0; k < 2; ++k) {
                    u64 c0, c1;
                    unsigned z;
                    asm volatile("s_memtime %0\n\ts_waitcnt lgkmcnt(0)\n\ts_load_dword %2, %3, %4\n\ts_waitcnt lgkmcnt(0)\n\t"
                                 "s_memtime %1\n\ts_waitcnt lgkmcnt(0)"
                                 : "=&s"(c0), "=&s"(c1), "=&s"(z) : "s"(a), "s"(k * 8192u + r * 16384u) : "memory");
                    if (r > 0) best[k] = min(best[k], (int)(unsigned)(c1 - c0));
                }
            int df = min(max((best[0] - best[1]) >> 4, -10), 10);
            if (((xcc >> 2) ^ pg) & 1) df = -df;
            unsigned all = 0;
            if (lane == 0) all = __hip_atomic_fetch_add((LAS unsigned*)(L + L_VOTE), 0x1000u + 16u + df, __ATOMIC_RELAXED, __HIP_MEMORY_SCOPE_WORKGROUP);
            all = rfl(all) + 0x1000u + 16u + df;
            if (all >= 0x8000u) {
                const unsigned vote = rfl(lds_ld(L, L_CALL)) << 8 | (all & 255u);
                asm volatile("s_mov_b64 exec, 0xffff\n\tglobal_atomic_swap %0, %1, %2 sc1\n\ts_mov_b64 exec, -1"
                             :: "v"((unsigned)WS_FLAG + (16 + lane) * FSTR + 64 + (xcc & 7) * 4), "v"(vote), "s"(myws) : "memory");
            }
        }
        otile += ((size_t)((wr & 1) * 64 + (wr >> 1) * 256) * NN + tj * 256 + wc * 128) * 4;
        asm volatile("" : "+s"(scale), "+s"(otile));
        const int abase = h * 8192 + (64 * wr + i) * 16;
        const int bbase = 16384 + h * 8192 + (32 * wc + i) * 16;
        v16f acc[2][4];
#pragma unroll
        for (int m = 0; m < 2; ++m)
#pragma unroll
            for (int c = 0; c < 4; ++c)
#pragma unroll
                for (int e = 0; e < 16; ++e) acc[m][c][e] = 0.f;
        unsigned spin = 0;
        for (int s = 0; s < NSB; ++s) {
            for (;;) {
                v2u f = *(volatile LAS v2u*)(L + L_FULL);
                unsigned mn = rfl(f.x < f.y ? f.x : f.y);
                if (mn >= (unsigned)(4 * s + 4)) break;
                if (expired(spin, tstart)) { lds_st(L, L_BAD, 1u); break; }
                __builtin_amdgcn_s_sleep(1);
            }
            FENCE();
            LAS unsigned char* sl = L + (s & (NSLOT - 1)) * SLOT;
            v8i a[2], bf[4];
#pragma unroll
            for (int m = 0; m < 2; ++m) {
                v4i lo = *(LAS v4i*)(sl + abase + m * 512), hi = *(LAS v4i*)(sl + abase + m * 512 + 4096);
                a[m] = __builtin_shufflevector(lo, hi, 0, 1, 2, 3, 4, 5, 6, 7);
            }
#pragma unroll
            for (int c = 0; c < 4; ++c) {
                v4i lo = *(LAS v4i*)(sl + bbase + c * 1024), hi = *(LAS v4i*)(sl + bbase + c * 1024 + 4096);
                bf[c] = __builtin_shufflevector(lo, hi, 0, 1, 2, 3, 4, 5, 6, 7);
                if (c == 3) {
                    asm volatile("s_waitcnt lgkmcnt(0)" : "+v"(bf[3]) :: "memory");
                    lds_st(L, L_PROG + wave * 4, (unsigned)(s + 1 < NSB ? s + 1 : NSB + 7 * wr));
                    FENCE();
                }
#pragma unroll
                for (int m = 0; m < 2; ++m)
                    acc[m][c] = __builtin_amdgcn_mfma_scale_f32_32x32x64_f8f6f4(a[m], bf[c], acc[m][c], 0, 0, 0, 0, 0, 0);
            }
        }
        {
            const unsigned ti = (rfl(lds_ld(L, L_TI)) - 1u) & 15u;
            otile += (size_t)((ti >> 1) * 512 + (ti & 1) * 128) * NN * 4;
            asm volatile("" : "+s"(otile));
        }
        unsigned va;
        {
            const unsigned lb = (unsigned)(uintptr_t)L;
            unsigned vb = lb + (32 * wc + i) * 16, t0, t1;
            unsigned m = 7 * wr, blk = (7 * wr + NXB - 1) % NXB, cnt, tmp, rel;
            va = lb + L_XR + blk * CH + h * 2048 + i * 16;
            u64 now;
#define POLL_ADDR \
            "v_bfe_u32 %[t0], %[va], 11, 1\n\ts_lshl3_add_u32 %[tmp], %[m], %[fin]\n\t" \
            "v_lshl_add_u32 %[t0], %[t0], 2, %[tmp]\n\t"
#define MOVE \
            "s_add_u32 %[blk], %[blk], 1\n\ts_cmp_eq_u32 %[blk], %[nxb]\n\ts_cselect_b32 %[blk], 0, %[blk]\n\t" \
            "s_cselect_b32 %[tmp], %[back], %[fwd]\n\t" \
            "v_and_b32 %[t0], 0xf000, %[t0]\n\tv_and_b32 %[vb], 0x1fff, %[vb]\n\tv_add_u32 %[va], %[tmp], %[va]\n\t" \
            "v_lshl_add_u32 %[vb], %[t0], 1, %[vb]\n\t"
#define RD(r0, r1, adr, off, hi) \
            "ds_read_b128 v[" #r0 "], %[" #adr "] offset:" #off "\n\tds_read_b128 v[" #r1 "], %[" #adr "] offset:" #off "+" #hi "\n\t"
#define RD_A0 RD(136:139, 140:143, va, 0, 1024)
#define RD_A1 RD(144:147, 148:151, va, 512, 1024)
#define RD_B0(off) RD(152:155, 156:159, vb, off, 4096)
#define RD_B1(off) RD(160:163, 164:167, vb, off, 4096)
#define MM_(c, a, b) "v_mfma_f32_32x32x64_f8f6f4 %[" #c "], v[" #a "], v[" #b "], %[" #c "]\n\t"
            asm volatile(
                "s_mov_b32 %[cnt], 7\n"
                "1:\n\t"
                POLL_ADDR
                "ds_read_b32 %[t0], %[t0]\n\ts_waitcnt lgkmcnt(0)\n\t"
                "v_cmp_le_u32 vcc, 0x01010000, %[t0]\n\ts_cmp_eq_u64 vcc, -1\n\ts_cbranch_scc1 3f\n\t"
                "s_memrealtime %[now]\n\ts_waitcnt lgkmcnt(0)\n\t"
                "s_mov_b64 vcc, %[now]\n\ts_sub_u32 %[tmp], vcc_lo, %[ts]\n\t"
                "s_cmp_gt_u32 %[tmp], %[dl]\n\ts_cbranch_scc0 1b\n\t"
                "v_mov_b32 %[t0], %[bad]\n\tv_mov_b32 %[t1], 1\n\tds_write_b32 %[t0], %[t1]\n\t"
                "v_mov_b32 %[t0], %[prog]\n\tv_mov_b32 %[t1], 1000\n\tds_write_b32 %[t0], %[t1]\n\t"
                "s_branch 9f\n"
                "3:\n\t"
                MOVE
                RD_B0(0) RD_A0 RD_A1
                "\n4:\n\t"
                RD_B1(1024)
                "s_add_u32 %[m], %[m], 1\n\ts_add_u32 %[rel], %[m], %[nsb]\n\ts_sub_u32 %[cnt], %[cnt], 1\n\t"
                "s_cmp_eq_u32 %[cnt], 0\n\ts_cselect_b32 %[rel], 1000, %[rel]\n\t"
                "v_mov_b32 %[t0], %[prog]\n\tv_mov_b32 %[t1], %[rel]\n\t"
                "s_waitcnt lgkmcnt(4)\n\t"
                MM_(c00, 136:143, 152:159)
                "s_waitcnt lgkmcnt(2)\n\t"
                MM_(c10, 144:151, 152:159)
                "ds_write_b32 %[t0], %[t1]\n\t"
                RD_B0(2048)
                POLL_ADDR
                "s_waitcnt lgkmcnt(3)\n\t"
                MM_(c01, 136:143, 160:167)
                "ds_read_b32 %[t0], %[t0]\n\t"
                MM_(c11, 144:151, 160:167)
                RD_B1(3072)
                "s_waitcnt lgkmcnt(3)\n\t"
                MM_(c02, 136:143, 152:159)
                "s_waitcnt lgkmcnt(2)\n\t"
                "v_cmp_le_u32 vcc, 0x01010000, %[t0]\n\t"
                MM_(c12, 144:151, 152:159)
                "s_cmp_eq_u32 %[cnt], 0\n\ts_cbranch_scc1 6f\n\t"
                "s_cmp_eq_u64 vcc, -1\n\ts_cbranch_scc0 6f\n\t"
                MOVE
                RD_B0(0)
                "s_waitcnt lgkmcnt(2)\n\t"
                MM_(c03, 136:143, 160:167)
                RD_A0
                MM_(c13, 144:151, 160:167)
                RD_A1
                "s_branch 4b\n"
                "6:\n\t"
                "s_waitcnt lgkmcnt(0)\n\t"
                MM_(c03, 136:143, 160:167)
                MM_(c13, 144:151, 160:167)
                "s_cmp_eq_u32 %[cnt], 0\n\ts_cbranch_scc0 1b\n"
                "9:\n\t"
                "s_waitcnt lgkmcnt(0)\n\ts_nop 15\n\ts_nop 15"
                : [c00] "+v"(acc[0][0]), [c01] "+v"(acc[0][1]), [c02] "+v"(acc[0][2]), [c03] "+v"(acc[0][3]),
                  [c10] "+v"(acc[1][0]), [c11] "+v"(acc[1][1]), [c12] "+v"(acc[1][2]), [c13] "+v"(acc[1][3]),
                  [va] "+v"(va), [vb] "+v"(vb), [t0] "=&v"(t0), [t1] "=&v"(t1),
                  [m] "+s"(m), [blk] "+s"(blk), [cnt] "=&s"(cnt), [tmp] "=&s"(tmp), [rel] "=&s"(rel), [now] "=&s"(now)
                : [fin] "s"(lb + L_FIN), [prog] "s"(lb + L_PROG + wave * 4), [bad] "s"(lb + L_BAD), [ts] "s"(tstart),
                  [dl] "s"(DEADLINE2), [nsb] "n"(NSB), [nxb] "n"(NXB), [fwd] "s"(CH), [back] "n"(-(NXB - 1) * CH)
                : "memory", "vcc", "scc",
                  "v136", "v137", "v138", "v139", "v140", "v141", "v142", "v143", "v144", "v145", "v146", "v147",
                  "v148", "v149", "v150", "v151", "v152", "v153", "v154", "v155", "v156", "v157", "v158", "v159",
                  "v160", "v161", "v162", "v163", "v164", "v165", "v166", "v167");
        }
        const bool bad = lds_ld(L, L_BAD) != 0u;
        const float mul = bad ? __builtin_nanf("") : scale;
        const unsigned oo = (va & 0x800u) * 32u + (va & 0x1f0u);
#pragma unroll
        for (int m = 0; m < 2; ++m)
#pragma unroll
            for (int e = 0; e < 16; ++e) {
                v4f v = {acc[m][0][e] * mul, acc[m][1][e] * mul, acc[m][2][e] * mul, acc[m][3][e] * mul};
                st16_sys(otile + (size_t)(m * 32 + (e & 3) + 8 * (e >> 2)) * NN * 4, oo, __builtin_bit_cast(v4u, v));
            }
        return;
    }

    __builtin_amdgcn_s_setprio(3);
    if (wave == 10) {
        const int d = (dtab >> (4 * sl)) & 7;
        const int ord = q ^ 16;
        const unsigned per = (sl & 4) ? PERX8 : PER8;
        const unsigned t0 = tstart + (unsigned)(LEAD) + ord * per / 256;
        const int late1 = ((sl ^ xcc) & 4) ? 0 : 1 << 30;
        if (q < 16) send_panel<false>(px + (size_t)((q >> 1) * 512 + (q & 1) * 128) * KS, L, myws, dws, rank, q, d, lane, t0, per, late1, ord * per / 256);
        else send_panel<true>(pw + (q - 16) * 256, L, myws, dws, rank, q, d, lane, t0, per, late1, ord * per / 256);
        return;
    }

    if (wave == 11) {
        __builtin_amdgcn_s_barrier();
        FENCE();
        const unsigned call = rfl(lds_ld(L, L_CALL));
        const unsigned fbase = (call - 1u) * FSTRIDE;
        int ti;
        {
            while ((unsigned)rt() - tstart < VPOLL) __builtin_amdgcn_s_sleep(8);
            const int k = (xcc & 2) * 2 + (idx >> 3);
            unsigned v;
            for (;;) {
                asm volatile("s_mov_b64 exec, 1\n\tglobal_atomic_or %0, %1, %2, %3 sc0 sc1\n\ts_mov_b64 exec, -1\n\ts_waitcnt vmcnt(0)"
                             : "=&v"(v) : "v"((unsigned)WS_FLAG + (16 + tj) * FSTR + 64 + k * 4), "v"(0u), "s"(myws) : "memory");
                v = rfl(v);
                if ((v ^ call << 8) < 256u) break;
                if ((unsigned)rt() - tstart > DEADLINE) { lds_st(L, L_BAD, 1u); break; }
                __builtin_amdgcn_s_sleep(8);
            }
            unsigned c = (v & 255u) > 128u;
            ti = 2 * k + ((xcc >> 2 ^ xcc ^ c) & 1);
            lds_st(L, L_TI, ti + 1);
        }
        const int d = lane & 7;
        const unsigned srcr = (unsigned)(rank + d) & 7u;
        const unsigned foff = WS_FLAG + (lane & 8 ? 16 + tj : ti) * FSTR + srcr * 4;
        const unsigned row0 = srcr * NCH;
        const int nmax = lane >= 8 ? 0 : d == 0 ? NCH : NBULK, amax = lane >= 8 || d == 0 ? 0 : 8;
        unsigned it = 0;
        int seen = 0, qn = 0;
        int seen2 = 0, fq = 0, glim = 2;
#define POLL(r) asm volatile("s_mov_b64 exec, %3\n\tglobal_atomic_or " r ", %0, %1, %2 sc0 sc1\n\ts_mov_b64 exec, -1" \
                             :: "v"(foff), "v"(0u), "s"(myws), "s"(wdone ? 0xffull : 0xffffull) : "memory", r)
#define TAKE(v, r, n) asm volatile("s_waitcnt vmcnt(" #n ")\n\tv_mov_b32 %0, " r : "=v"(v) :: "memory")
        bool wdone = false;
        auto process = [&](unsigned v) {
            bool dead = false;
            if ((++it & 15u) == 0u) {
                dead = rfl(__hip_atomic_load((const unsigned*)(myws + WS_TMO), __ATOMIC_RELAXED, __HIP_MEMORY_SCOPE_AGENT)) == call;
                if (!dead && (unsigned)rt() - tstart > DEADLINE) {
                    if (lane == 0) __hip_atomic_store((unsigned*)(myws + WS_TMO), call, __ATOMIC_RELAXED, __HIP_MEMORY_SCOPE_AGENT);
                    dead = true;
                }
            }
            int nx = (int)(v - fbase), nw = dpp0<0x128>(nx);
            if (wdone) nw = NCH;
            if (dead) { nx = nw = 64; lds_st(L, L_BAD, 1u); }
            int nn = min(min(nx, nw), nmax) - seen;
            nn = max(nn, 0);
            if (__ballot(nn > 0)) {
                int pre = nn;
                pre += dpp0<0x111>(pre); pre += dpp0<0x112>(pre); pre += dpp0<0x114>(pre);
                const int pos = qn + pre - nn;
                FENCE();
                for (int j = 0; j < nn; ++j) lds_st(L, L_QUEUE + (pos + j) * 4, row0 + seen + j + 1u);
                seen += nn;
                qn += __builtin_amdgcn_readlane(pre, 7);
                it = 0;
            }
            int a = min(nx - NBULK, amax);
            if (nw < NCH) a = min(a, (nw - NBULK) >> 1);
            for (;;) {
                const int n2 = max(min(a, glim) - seen2, 0);
                if (!__ballot(n2 > 0)) break;
                int pre = n2;
                pre += dpp0<0x111>(pre); pre += dpp0<0x112>(pre); pre += dpp0<0x114>(pre);
                const int pos = fq + pre - n2;
                FENCE();
                for (int j = 0; j < n2; ++j) {
                    const unsigned pr = (seen2 + j) & 1;
                    *(volatile LAS unsigned short*)(L + L_FIN + (pos + j) * 4) =
                        (unsigned short)((2 * d - 2 + pr) << 12 | (row0 + NBULK + ((seen2 + j) >> 1)));
                }
                seen2 += n2;
                fq += __builtin_amdgcn_readlane(pre, 7);
                it = 0;
                if (fq != glim * (NR - 1)) break;
                glim += 2;
            }
            wdone = qn >= NQ && fq >= NPAIR;
            return qn >= NQ && fq >= NFIN;
        };
        unsigned v;
        bool end = false;
        while (!end) {
            POLL("v160"); TAKE(v, "v160", 0);
            end = process(v);
            __builtin_amdgcn_s_sleep(2);
        }
        return;
    }

    {
        const int ld = wave - 8;
        __builtin_amdgcn_s_barrier();
        FENCE();
        const unsigned voff = lane * 16;
        unsigned spin = 0, ti;
        while ((ti = rfl(lds_ld(L, L_TI))) == 0u) {
            if (expired(spin, tstart)) { lds_st(L, L_BAD, 1u); break; }
            __builtin_amdgcn_s_sleep(4);
        }
        ti = (ti - 1u) & 15u;
        const unsigned char* dbase = myws + WS_DATA + (size_t)(rfl(lds_ld(L, L_CALL)) & (NPAR - 1u)) * PAR_BYTES +
                                     (size_t)ppos(ld ? 16 + tj : ti) * PAN_BYTES;
        const unsigned sw = ld ? 16 + tj : ti;
        const unsigned lb = (unsigned)(uintptr_t)L;
        int iss = 0, done = 0;
        while (done < NQ) {
            const unsigned v = lds_ld(L, lane < 16 ? L_PROG + (lane & 7) * 4 : L_QUEUE + (iss + (lane & 15)) * 4);
            int lim = iss + __builtin_ctzll(~(__ballot(v != 0u) >> 16));
            lim = min(min(lim, iss + 16), min(NQ, done + 15));
            const int st = iss >> 2;
            int fr = st + NSLOT;
#pragma unroll
            for (int j = NSLOT - 1; j >= 0; --j) if (__ballot(lane < 8 && (int)v < st + j - (NSLOT - 1))) fr = st + j;
            lim = min(lim, fr * 4);
            FENCE();
            const bool any = iss < lim;
            for (const int i0 = iss; iss < lim; ++iss) {
                const unsigned e = __builtin_amdgcn_readlane(v, 16 + iss - i0) - 1u;
                dma4k(dbase + ((e >> 5 << 22) + ((e & 31u) ^ sw) * CH), lb + ld * 16384 + ((iss >> 2) & (NSLOT - 1)) * SLOT + (iss & 3) * CH, voff);
            }
            const int tg = iss - ((vm_count() + 3) >> 2);
            if (tg > done) {
                FENCE();
                done = tg;
                lds_st(L, L_FULL + ld * 4, (unsigned)done);
            } else if (!any) {
                if (expired(spin, tstart)) { lds_st(L, L_BAD, 1u); break; }
                __builtin_amdgcn_s_sleep(1);
            }
        }
        auto fin = [&](auto isw) {
            constexpr bool W = decltype(isw)::value;
            constexpr int NF = W ? NPAIR : NFIN;
            unsigned xa = lb + L_XR;
            iss = 0; done = 0;
            while (done < NF) {
                const unsigned v = lds_ld(L, lane < 16 ? L_PROG + (lane & 7) * 4 : L_FIN + (iss + (lane & 15)) * 4);
                int n = __builtin_ctz(~(unsigned)(__ballot((v & 0xfffu) != 0u) >> 16) | (1u << (W ? WBATCH : XBATCH)));
                int pm = (int)v;
                pm = min(pm, dpp0<0x121>(pm)); pm = min(pm, dpp0<0x122>(pm)); pm = min(pm, dpp0<0x124>(pm));
                pm = (int)rfl(pm) - NSB;
                int lim = W ? done + 7 : 2 * (pm + NXB);
                lim = lim < NF ? lim : NF;
                lim = pm < 0 ? 0 : lim - iss;
                n = n < lim ? n : lim;
                const unsigned off = (v & 0xe0u) << 17 | (W ? (((v & 31u) | (v & 0x1000u) >> 11) ^ sw) << 12 : ((v & 31u) ^ sw) << 12 | (v & 0x1000u) >> 1);
                FENCE();
                for (int j = 0; j < n; ++j) {
                    const unsigned vo = voff + __builtin_amdgcn_readlane(off, 16 + j);
                    if constexpr (W) {
                        const unsigned la = lb + (__builtin_amdgcn_readlane(v, 16 + j) & 0xf000u) * 2;
                        dma4k(dbase, la, vo);
                        dma4k(dbase, la + CH, vo ^ CH);
                    } else {
                        dma2k(dbase, xa, vo);
                        xa = xa == lb + L_CTL - 2048 ? lb + L_XR : xa + 2048;
                    }
                }
                if (n > 0) iss += n;
                const int vm = vm_count();
                const int tg = iss - (W ? (vm + 7) >> 3 : (vm + 1) >> 1);
                if (tg > done) {
                    FENCE();
                    const int last = tg == NF ? NFIN : tg;
                    if (done + lane < last) *(volatile LAS unsigned char*)(L + L_FIN + (done + lane) * 4 + 2 + W) = 1;
                    done = tg;
                } else if (n <= 0) {
                    if (expired(spin, tstart)) { lds_st(L, L_BAD, 1u); break; }
                    __builtin_amdgcn_s_sleep(1);
                }
            }
        };
        if (ld) fin(std::true_type{}); else fin(std::false_type{});
    }
}

}

extern "C" void kernel_launch(int rank, int nranks, ncclComm_t comm, hipStream_t stream,
                              void* const* d_in, const int* in_sizes, int n_in,
                              void* d_out, int out_size, const int* in_dtypes, int out_dtype,
                              void* const* d_out_all, void* const* d_ws_all, size_t ws_bytes) {
    if (nranks != NR || n_in != 4 || ws_bytes < WS_NEED || in_sizes[0] != MM * KS || in_sizes[1] != KS * NN ||
        out_size != MM * NN || (unsigned)rank >= (unsigned)NR) {
        fprintf(stderr, "kernel_launch: unexpected problem geometry\n");
        return;
    }
    Params p;
    p.x = (const unsigned char*)d_in[0]; p.w = (const unsigned char*)d_in[1];
    p.sx = (const float*)d_in[2]; p.sw = (const float*)d_in[3];
    p.out = (float*)d_out; p.rank = rank;
    p.myws = (unsigned char*)d_ws_all[rank];
    const unsigned dtab[NR] = {0x14735602u, 0x73164502u, 0x13654207u, 0x74623501u, 0x16245307u, 0x75234601u, 0x15724306u, 0x75143206u};
    p.dtab = dtab[rank];
    for (int s = 0; s < NR; ++s) p.dws[s] = (unsigned char*)d_ws_all[(rank + NR - ((p.dtab >> (4 * s)) & 7)) & 7];
    hipLaunchKernelGGL(fused_gemm_ag, dim3(NWG), dim3(NTHREADS), 0, stream, p);
    HIP_CHECK(hipGetLastError());
}
```
